```python
import numpy as np

kernel_name = "gpu_gemm_rs_m4096_k4096_n4096_f32_qe4m3"


ABSMAX_RTOL = 0.09

COMPUTE_DTYPE = "float32"
EPILOGUE = "quant_e4m3"
IN_QUANT = None
OUT_QUANT = "e4m3"
IN_DTYPES = {"x": "float32", "w_mat": "float32"}
WORLD_SIZE = 8

def setup_inputs():

    x = np.random.randn(4096, 4096).astype(np.float32)
    w_mat = (np.random.randn(4096, 4096) / np.sqrt(4096)).astype(np.float32)
    return {"x": x, "w_mat": w_mat}

def _snap_fp8(x, mant_bits, max_abs, min_normal):
    a = np.abs(x.astype(np.float64))
    _m, _e = np.frexp(a)
    q = np.where(a >= min_normal, np.ldexp(1.0, _e - 1 - mant_bits),
                 min_normal * 2.0 ** -mant_bits)
    snapped = np.minimum(np.round(a / q) * q, max_abs)
    return (np.sign(x) * snapped).astype(np.float32)
def _quant_dequant(y, amax):
    scale = amax / 448.0
    q = _snap_fp8(y / scale, 3, 448.0, 2.0 ** -6)
    return (q * scale).astype(np.float32)
def _epilogue(y):
    return _quant_dequant(y, np.abs(y).max())

def reference(x, w_mat):
    y = (x @ w_mat).astype(np.float32)
    return _epilogue(y)
```

```cpp
#include <hip/hip_runtime.h>
#include <rccl/rccl.h>
#include <cstdio>

#define HIP_CHECK(expr) do { hipError_t _e = (expr); if (_e != hipSuccess) { \
    fprintf(stderr, "HIP error %s at %s:%d\n", hipGetErrorString(_e), __FILE__, __LINE__); \
    return; } } while (0)

namespace {

constexpr int NR = 8;
constexpr int NN = 4096;
constexpr int KS = 512;
constexpr int NWG = 256;
constexpr int NTHR = 512;

constexpr unsigned OFF_CALL = 0;
constexpr unsigned OFF_TMO = 32768;
constexpr unsigned OFF_PD = OFF_TMO + 256;
constexpr unsigned OFF_AMX = OFF_TMO + 512;
constexpr unsigned OFF_ACT = OFF_TMO + 768;
constexpr unsigned OFF_AG = OFF_TMO + 1024;
constexpr unsigned OFF_XF = OFF_TMO + 4096;
constexpr unsigned OFF_WF = OFF_XF + 2048;
constexpr unsigned OFF_PF = OFF_TMO + 8192;
constexpr unsigned OFF_DBG = 65536;
constexpr unsigned OFF_XS = 1u << 20;
constexpr unsigned OFF_WS = OFF_XS + (8u << 20);
constexpr unsigned OFF_PS = OFF_WS + (8u << 20);
constexpr unsigned WS_NEED = OFF_PS + (8u << 20);

constexpr unsigned long long DEADLINE = 40000000ull;
constexpr int AUX_SYS = 17;

constexpr int STAGE_B = 32768;
constexpr int NSTAGE = 4;
constexpr int LDS_MISC = NSTAGE * STAGE_B;
constexpr int LDS_B = LDS_MISC + 256;

typedef __bf16 bf16x8 __attribute__((ext_vector_type(8)));
typedef __bf16 bf16x2 __attribute__((ext_vector_type(2)));
typedef float f32x2 __attribute__((ext_vector_type(2)));
typedef float f32x4 __attribute__((ext_vector_type(4)));
typedef unsigned u32x4 __attribute__((ext_vector_type(4)));
typedef unsigned long long u64;

struct Params {
  const float* x;
  const float* w;
  unsigned char* out;
  char* ws[NR];
  int rank;
};

__device__ __forceinline__ unsigned pack2(float lo, float hi) {
  f32x2 v = {lo, hi};
  return __builtin_bit_cast(unsigned, __builtin_convertvector(v, bf16x2));
}
__device__ __forceinline__ float bf_lo(unsigned u) { return __builtin_bit_cast(float, u << 16); }
__device__ __forceinline__ float bf_hi(unsigned u) { return __builtin_bit_cast(float, u & 0xffff0000u); }

__device__ __forceinline__ __amdgpu_buffer_rsrc_t rsrc_of(const void* p, unsigned bytes) {
  return __builtin_amdgcn_make_buffer_rsrc(const_cast<void*>(p), 0, bytes, 0x00020000);
}
__device__ __forceinline__ char* uniform_ptr(char* p) {
  const u64 a = (u64)p;
  const unsigned lo = __builtin_amdgcn_readfirstlane((unsigned)a), hi = __builtin_amdgcn_readfirstlane((unsigned)(a >> 32));
  return (char*)(((u64)hi << 32) | lo);
}
#define WAIT_V(n) asm volatile("s_waitcnt vmcnt(" #n ")" ::: "memory")
#define WAIT_L0 asm volatile("s_waitcnt lgkmcnt(0)" ::: "memory")

#define STAMP_AT(k) do { } while (0)

__device__ __forceinline__ void wait_words(const unsigned* p, bool need, unsigned want, unsigned* tmo,
                                           unsigned call, int lane) {
  const unsigned* q = lane == 63 ? tmo : p;
  const bool act = need || lane == 63;
  const u64 t0 = __builtin_amdgcn_s_memrealtime();
  for (;;) {
    unsigned v = 0;
    if (act) v = __hip_atomic_load(q, __ATOMIC_RELAXED, __HIP_MEMORY_SCOPE_SYSTEM);
    if (__all(!need || lane == 63 || (int)(v - want) >= 0)) break;
    if (__any(lane == 63 && v == call)) break;
    if (__builtin_amdgcn_s_memrealtime() - t0 > DEADLINE) {
      if (lane == 0) __hip_atomic_store(tmo, call, __ATOMIC_RELAXED, __HIP_MEMORY_SCOPE_AGENT);
      break;
    }
    __builtin_amdgcn_s_sleep(2);
  }
}

__global__ void __launch_bounds__(NTHR, 2) gemm_rs_kernel(Params P) {
  __shared__ __attribute__((aligned(16))) char lds[LDS_B];
  const int tid = threadIdx.x;
  const int lane = tid & 63;
  const int wv = __builtin_amdgcn_readfirstlane(tid >> 6);
  const int l15 = lane & 15, q4 = lane >> 4;
  const int rank = P.rank;
  const int ri = rank >> 2, rj = (rank >> 1) & 1, rh = rank & 1;
  char* const me = P.ws[rank];
  unsigned* const tmo = (unsigned*)(me + OFF_TMO);
  unsigned* const misc = (unsigned*)(lds + LDS_MISC);
  STAMP_AT(0);

  const int b = blockIdx.x, xcd = b & 7, idx = b >> 3;
  const int rb = 4 * (idx & 3) + (xcd & 3), cb = 2 * (idx >> 2) + (xcd >> 2);
  const int owner = rb >> 1;
  const bool producer = owner == rank;

  auto draw_call = [&]() {
    if (tid < 16) misc[tid] = 0;
    if (tid == 0)
      misc[16] = __hip_atomic_fetch_add((unsigned*)(me + OFF_CALL + b * 128), 1u, __ATOMIC_RELAXED,
                                        __HIP_MEMORY_SCOPE_AGENT) + 1u;
  };
  if (!producer) draw_call();

  if (producer) {
    const int pidx = cb * 2 + (rb & 1);
    const int cls = pidx & 3, slot = pidx >> 2;
    const int ca = cls >> 1, hh = cls & 1;
    int dx0 = ca * 4 + hh, dx1 = ca * 4 + 2 + hh;
    int dw0 = ca * 2 + hh, dw1 = 4 + ca * 2 + hh;
    int cx0 = rank, cx1 = rank, cw0 = rank, cw1 = rank;
    if (dx0 == (rank ^ 1)) { dx0 = rank; cx0 = rank ^ 1; }
    if (dx1 == (rank ^ 1)) { dx1 = rank; cx1 = rank ^ 1; }
    if (dw0 == (rank ^ 1)) { dw0 = rank; cw0 = rank ^ 1; }
    if (dw1 == (rank ^ 1)) { dw1 = rank; cw1 = rank ^ 1; }
    const auto rdx0 = rsrc_of(uniform_ptr(P.ws[dx0 & 7]), WS_NEED), rdx1 = rsrc_of(uniform_ptr(P.ws[dx1 & 7]), WS_NEED);
    const auto rdw0 = rsrc_of(uniform_ptr(P.ws[dw0 & 7]), WS_NEED), rdw1 = rsrc_of(uniform_ptr(P.ws[dw1 & 7]), WS_NEED);
    const auto rx = rsrc_of(P.x, 4096u * KS * 4u);
    const auto rw = rsrc_of(P.w, (unsigned)KS * NN * 4u);
    f32x4 v[8];
    auto load_item = [&](int n) {
      const int id = (n & 3) * 8 + slot, blk = id >> 1, kh = id & 1;
      const int kb = 256 * hh + 128 * kh;
      if (n < 4) {
        const int row = 512 * (blk >> 1) + 256 * ca + 128 * (blk & 1) + 16 * wv + l15;
        const unsigned o = (unsigned)(row * KS + kb + 4 * q4) * 4u;
#pragma unroll
        for (int e = 0; e < 8; ++e)
          v[e] = __builtin_bit_cast(f32x4, __builtin_amdgcn_raw_buffer_load_b128(rx, o + e * 64, 0, 0));
      } else {
        const int col = 2048 * ca + 128 * blk + 64 * (wv & 1) + 4 * l15;
        const int k0 = kb + 32 * (wv >> 1) + 4 * q4;
        const unsigned o = (unsigned)(k0 * NN + col) * 4u;
#pragma unroll
        for (int e = 0; e < 8; ++e)
          v[e] = __builtin_bit_cast(f32x4, __builtin_amdgcn_raw_buffer_load_b128(
                     rw, o + (unsigned)((e & 3) + 16 * (e >> 2)) * NN * 4u, 0, 0));
      }
    };
    load_item(0);
    draw_call();
    __syncthreads();
    const unsigned call = misc[16];
#pragma unroll
    for (int n = 0; n < 8; ++n) {
      const int id = (n & 3) * 8 + slot, blk = id >> 1, kh = id & 1;
      u32x4 f[4];
      unsigned fo[4];
      if (n < 4) {
#pragma unroll
        for (int ks = 0; ks < 4; ++ks) {
          f[ks] = u32x4{pack2(v[2 * ks][0], v[2 * ks][1]), pack2(v[2 * ks][2], v[2 * ks][3]),
                        pack2(v[2 * ks + 1][0], v[2 * ks + 1][1]), pack2(v[2 * ks + 1][2], v[2 * ks + 1][3])};
          fo[ks] = (unsigned)(((kh * 4 + ks) * 8 + wv) * 1024 + lane * 16);
        }
      } else {
#pragma unroll
        for (int u = 0; u < 4; ++u) {
          f[u] = u32x4{pack2(v[0][u], v[1][u]), pack2(v[2][u], v[3][u]), pack2(v[4][u], v[5][u]),
                       pack2(v[6][u], v[7][u])};
          fo[u] = (unsigned)(((kh * 4 + (wv >> 1)) * 8 + (wv & 1) * 4 + u) * 1024 + lane * 16);
        }
      }
      const int c0 = n < 4 ? cx0 : cw0, c1 = n < 4 ? cx1 : cw1;
      const auto r0 = n < 4 ? rdx0 : rdw0, r1 = n < 4 ? rdx1 : rdw1;
      const unsigned sb = n < 4 ? OFF_XS : OFF_WS;
      const unsigned s0 = sb + (unsigned)(c0 * 16 + blk) * 65536u, s1 = sb + (unsigned)(c1 * 16 + blk) * 65536u;
#pragma unroll
      for (int u = 0; u < 4; ++u) {
        __builtin_amdgcn_raw_buffer_store_b128(f[u], r0, fo[u], s0, AUX_SYS);
        __builtin_amdgcn_raw_buffer_store_b128(f[u], r1, fo[u], s1, AUX_SYS);
      }
      if (n < 7) {
        load_item(n + 1);
        WAIT_V(8);
      } else {
        if (tid == 0) __hip_atomic_fetch_add((unsigned*)(me + OFF_PD), 1u, __ATOMIC_RELAXED, __HIP_MEMORY_SCOPE_AGENT);
        WAIT_V(0);
      }
      unsigned t = 0;
      if (lane == 0) t = __hip_atomic_fetch_add(misc + n, 1u, __ATOMIC_RELAXED, __HIP_MEMORY_SCOPE_WORKGROUP);
      if (__builtin_amdgcn_readfirstlane(t) == 7 && lane == 0) {
        const unsigned fb = (n < 4 ? OFF_XF : OFF_WF) + blk * 128 + kh * 4;
        __builtin_amdgcn_raw_buffer_store_b32(call, r0, 0, fb + c0 * 8, AUX_SYS);
        __builtin_amdgcn_raw_buffer_store_b32(call, r1, 0, fb + c1 * 8, AUX_SYS);
      }
    }
  }
  __syncthreads();
  const unsigned call = misc[16];
  STAMP_AT(1);

  if (wv == 0) {
    const unsigned* fp = lane < 16 ? (const unsigned*)(me + OFF_XF + rb * 128) + lane
                                   : (const unsigned*)(me + OFF_WF + cb * 128) + (lane & 15);
    wait_words(fp, lane < 32, call, tmo, call, lane);
    __builtin_amdgcn_fence(__ATOMIC_ACQUIRE, "");
    WAIT_V(0);
  }
  __syncthreads();
  STAMP_AT(2);

  f32x4 acc[2][4];
#pragma unroll
  for (int mi = 0; mi < 2; ++mi)
#pragma unroll
    for (int u = 0; u < 4; ++u) acc[mi][u] = f32x4{0.f, 0.f, 0.f, 0.f};
  {
    const int wm = wv >> 1, wn = wv & 1;
    const char* gx = me + OFF_XS + rb * 65536 + wv * 2048 + lane * 16;
    const char* gw = me + OFF_WS + cb * 65536 + wv * 2048 + lane * 16;
    auto stage = [&](int s) {
      const unsigned go = (unsigned)(s >> 2) * (16u * 65536u) + (unsigned)(s & 3) * 16384u;
      char* l = lds + (s & (NSTAGE - 1)) * STAGE_B + wv * 2048;
      __builtin_amdgcn_global_load_lds((const unsigned*)(gx + go), (unsigned*)l, 16, 0, 0);
      __builtin_amdgcn_global_load_lds((const unsigned*)(gx + go + 1024), (unsigned*)(l + 1024), 16, 0, 0);
      __builtin_amdgcn_global_load_lds((const unsigned*)(gw + go), (unsigned*)(l + 16384), 16, 0, 0);
      __builtin_amdgcn_global_load_lds((const unsigned*)(gw + go + 1024), (unsigned*)(l + 16384 + 1024), 16, 0, 0);
    };
    auto compute = [&](int s) {
      const char* l = lds + (s & (NSTAGE - 1)) * STAGE_B + lane * 16;
#pragma unroll
      for (int ks = 0; ks < 2; ++ks) {
        bf16x8 xf[2], wf[4];
#pragma unroll
        for (int mi = 0; mi < 2; ++mi) xf[mi] = *(const bf16x8*)(l + (ks * 8 + 2 * wm + mi) * 1024);
#pragma unroll
        for (int u = 0; u < 4; ++u) wf[u] = *(const bf16x8*)(l + 16384 + (ks * 8 + 4 * wn + u) * 1024);
#pragma unroll
        for (int mi = 0; mi < 2; ++mi)
#pragma unroll
          for (int u = 0; u < 4; ++u)
            acc[mi][u] = __builtin_amdgcn_mfma_f32_16x16x32_bf16(wf[u], xf[mi], acc[mi][u], 0, 0, 0);
      }
    };
    stage(0); stage(1); stage(2);
    for (int s = 0; s < 29; ++s) {
      WAIT_V(8);
      WAIT_L0;
      __builtin_amdgcn_s_barrier();
      stage(s + 3);
      compute(s);
    }
    WAIT_V(8); WAIT_L0; __builtin_amdgcn_s_barrier(); compute(29);
    WAIT_V(4); WAIT_L0; __builtin_amdgcn_s_barrier(); compute(30);
    WAIT_V(0); WAIT_L0; __builtin_amdgcn_s_barrier(); compute(31);
  }
  STAMP_AT(3);

  if ((owner | 1) != (rank | 1)) {
    if (wv == 0) wait_words((const unsigned*)(me + OFF_PD), lane == 0, call * 32u, tmo, call, lane);
    __syncthreads();
  }
  STAMP_AT(4);
  {
    const int wm = wv >> 1, wn = wv & 1;
    const int pos = ((ri * 2 + (rb & 1)) * 2 + rj) * 16 + cb;
    char* const wo = P.ws[owner & 7];
    const auto rp = rsrc_of(wo + OFF_PS + (unsigned)(pos * 2 + rh) * 32768u, 32768u);
#pragma unroll
    for (int mi = 0; mi < 2; ++mi)
#pragma unroll
      for (int s = 0; s < 2; ++s) {
        const u32x4 o = {pack2(acc[mi][0][2 * s], acc[mi][1][2 * s]), pack2(acc[mi][2][2 * s], acc[mi][3][2 * s]),
                         pack2(acc[mi][0][2 * s + 1], acc[mi][1][2 * s + 1]),
                         pack2(acc[mi][2][2 * s + 1], acc[mi][3][2 * s + 1])};
        __builtin_amdgcn_raw_buffer_store_b128(o, rp, (unsigned)((((2 * wm + mi) * 2 + wn) * 2 + s) * 1024 + lane * 16), 0, AUX_SYS);
      }
    WAIT_V(0);
    __syncthreads();
    if (tid == 0)
      __hip_atomic_store((unsigned*)(wo + OFF_PF + pos * 128 + rh * 4), call, __ATOMIC_RELAXED, __HIP_MEMORY_SCOPE_SYSTEM);
  }
  STAMP_AT(5);

  const int unit = (rb >> 1) * 32 + cb * 2 + (rb & 1);
  const int rpos = unit >> 1, rhalf = unit & 1;
  if (wv == 0) {
    wait_words((const unsigned*)(me + OFF_PF + rpos * 128) + (lane & 1), lane < 2, call, tmo, call, lane);
    __builtin_amdgcn_fence(__ATOMIC_ACQUIRE, "");
    WAIT_V(0);
  }
  __syncthreads();
  STAMP_AT(6);
  float y[16];
  {
    const char* pp = me + OFF_PS + rpos * 65536 + rhalf * 16384 + wv * 2048 + lane * 16;
    const u32x4 a0 = *(const u32x4*)(pp), a1 = *(const u32x4*)(pp + 1024);
    const u32x4 b0 = *(const u32x4*)(pp + 32768), b1 = *(const u32x4*)(pp + 32768 + 1024);
#pragma unroll
    for (int d = 0; d < 4; ++d) {
      y[2 * d] = bf_lo(a0[d]) + bf_lo(b0[d]);
      y[2 * d + 1] = bf_hi(a0[d]) + bf_hi(b0[d]);
      y[8 + 2 * d] = bf_lo(a1[d]) + bf_lo(b1[d]);
      y[8 + 2 * d + 1] = bf_hi(a1[d]) + bf_hi(b1[d]);
    }
  }
  unsigned am = 0;
#pragma unroll
  for (int t = 0; t < 16; ++t) am = max(am, __builtin_bit_cast(unsigned, y[t]) & 0x7fffffffu);
#pragma unroll
  for (int o = 32; o >= 1; o >>= 1) am = max(am, (unsigned)__shfl_xor((int)am, o));
  if (lane == 0) __hip_atomic_fetch_max(misc + 8, am, __ATOMIC_RELAXED, __HIP_MEMORY_SCOPE_WORKGROUP);
  __syncthreads();
  if (wv == 0) {
    unsigned t = 0;
    if (lane == 0) {
      __hip_atomic_fetch_max((u64*)(me + OFF_AMX), ((u64)call << 32) | misc[8], __ATOMIC_RELAXED, __HIP_MEMORY_SCOPE_AGENT);
      WAIT_V(0);
      t = __hip_atomic_fetch_add((unsigned*)(me + OFF_ACT), 1u, __ATOMIC_RELAXED, __HIP_MEMORY_SCOPE_AGENT);
    }
    t = __builtin_amdgcn_readfirstlane(t);
    if (t + 1u == call * (unsigned)NWG) {
      const u64 g = __hip_atomic_load((u64*)(me + OFF_AMX), __ATOMIC_RELAXED, __HIP_MEMORY_SCOPE_AGENT);
      __hip_atomic_store((u64*)(P.ws[lane & 7] + OFF_AG + (lane >> 3) * 128 + rank * 8), g, __ATOMIC_RELAXED,
                         __HIP_MEMORY_SCOPE_SYSTEM);
    }
    const u64* gp = lane == 63 ? (const u64*)tmo : (const u64*)(me + OFF_AG + xcd * 128) + (lane & 7);
    const bool act = lane < 8 || lane == 63;
    const u64 t0 = __builtin_amdgcn_s_memrealtime();
    unsigned gm, bad = 0;
    for (;;) {
      u64 g = 0;
      if (act) g = __hip_atomic_load(gp, __ATOMIC_RELAXED, __HIP_MEMORY_SCOPE_SYSTEM);
      gm = lane < 8 ? (unsigned)g : 0u;
      if (__any(lane == 63 && (unsigned)g == call)) { bad = 1; break; }
      if (__all(lane >= 8 || (int)((unsigned)(g >> 32) - call) >= 0)) break;
      if (__builtin_amdgcn_s_memrealtime() - t0 > DEADLINE) {
        if (lane == 0) __hip_atomic_store(tmo, call, __ATOMIC_RELAXED, __HIP_MEMORY_SCOPE_AGENT);
        bad = 1;
        break;
      }
      __builtin_amdgcn_s_sleep(1);
    }
#pragma unroll
    for (int o = 4; o >= 1; o >>= 1) gm = max(gm, (unsigned)__shfl_xor((int)gm, o));
    if (lane == 0) { misc[9] = gm; misc[10] = bad; }
  }
  __syncthreads();
  STAMP_AT(7);
  {
    const float amax = __builtin_bit_cast(float, misc[9]);
    const float inv = 448.0f / amax;
    u32x4 o;
#pragma unroll
    for (int d = 0; d < 4; ++d) {
      float a[4];
#pragma unroll
      for (int e = 0; e < 4; ++e) a[e] = y[4 * d + e] * inv;
      int r = __builtin_amdgcn_cvt_pk_fp8_f32(a[0], a[1], 0, false);
      r = __builtin_amdgcn_cvt_pk_fp8_f32(a[2], a[3], r, true);
      o[d] = (unsigned)r;
    }
    if (misc[10]) o = u32x4{0x7f7f7f7fu, 0x7f7f7f7fu, 0x7f7f7f7fu, 0x7f7f7f7fu};
    const int pi = rpos >> 6, prl = (rpos >> 5) & 1, pj = (rpos >> 4) & 1, pcb = rpos & 15;
    const int row = 256 * pi + 128 * prl + 16 * (rhalf * 4 + (wv >> 1)) + l15;
    const int col = 2048 * pj + 128 * pcb + 64 * (wv & 1) + 16 * q4;
    *(u32x4*)(P.out + (size_t)row * NN + col) = o;
  }
  STAMP_AT(8);
}

}

extern "C" void kernel_launch(int rank, int nranks, ncclComm_t comm, hipStream_t stream,
                              void* const* d_in, const int* in_sizes, int n_in, void* d_out, int out_size,
                              const int* in_dtypes, int out_dtype, void* const* d_out_all,
                              void* const* d_ws_all, size_t ws_bytes) {
  (void)comm; (void)in_dtypes; (void)out_dtype; (void)d_out_all;
  if (nranks != NR || (unsigned)rank >= (unsigned)NR || n_in < 2 || ws_bytes < WS_NEED ||
      in_sizes[0] != 4096 * KS || in_sizes[1] != KS * NN || out_size != 512 * NN) {
    fprintf(stderr, "unexpected problem shape\n");
    return;
  }
  Params p;
  p.x = (const float*)d_in[0];
  p.w = (const float*)d_in[1];
  p.out = (unsigned char*)d_out;
  for (int r = 0; r < NR; ++r) p.ws[r] = (char*)d_ws_all[r];
  p.rank = rank;
  gemm_rs_kernel<<<NWG, NTHR, 0, stream>>>(p);
  HIP_CHECK(hipGetLastError());
}
```

```cpp
#include <hip/hip_runtime.h>
#include <rccl/rccl.h>
#include <cstdio>

#define HIP_CHECK(expr) do { hipError_t _e = (expr); if (_e != hipSuccess) { \
    fprintf(stderr, "HIP error %s at %s:%d\n", hipGetErrorString(_e), __FILE__, __LINE__); \
    return; } } while (0)

namespace {

constexpr int NR = 8;
constexpr int NN = 4096;
constexpr int KS = 512;
constexpr int NWG = 256;
constexpr int NTHR = 512;

constexpr unsigned OFF_CALL = 0;
constexpr unsigned OFF_TMO = 32768;
constexpr unsigned OFF_PD = OFF_TMO + 2048;
constexpr unsigned OFF_XD = OFF_TMO + 3072;
constexpr unsigned OFF_AMX = OFF_TMO + 512;
constexpr unsigned OFF_ACT = OFF_TMO + 768;
constexpr unsigned OFF_AG = OFF_TMO + 1024;
constexpr unsigned OFF_XF = OFF_TMO + 4096;
constexpr unsigned OFF_WF = OFF_XF + 2048;
constexpr unsigned OFF_PF = OFF_TMO + 8192;
constexpr unsigned OFF_DBG = 65536;
constexpr unsigned OFF_XS = 1u << 20;
constexpr unsigned OFF_WS = OFF_XS + (8u << 20);
constexpr unsigned OFF_PS = OFF_WS + (8u << 20);
constexpr unsigned WS_NEED = OFF_PS + (8u << 20);

constexpr unsigned long long DEADLINE = 100000000ull;
#define TLOOK 20000
constexpr unsigned long long TMO_LOOK = TLOOK;
constexpr int AUX_SYS = 17;

constexpr int STAGE_B = 32768;
constexpr int NSTAGE = 4;
constexpr int LDS_MISC = NSTAGE * STAGE_B;
constexpr int LDS_B = LDS_MISC + 256;

typedef __bf16 bf16x8 __attribute__((ext_vector_type(8)));
typedef __bf16 bf16x2 __attribute__((ext_vector_type(2)));
typedef float f32x2 __attribute__((ext_vector_type(2)));
typedef float f32x4 __attribute__((ext_vector_type(4)));
typedef unsigned u32x4 __attribute__((ext_vector_type(4)));
typedef unsigned long long u64;

struct Params {
  const float* x;
  const float* w;
  unsigned char* out;
  char* ws[NR];
  int rank;
};

__device__ __forceinline__ unsigned pack2(float lo, float hi) {
  f32x2 v = {lo, hi};
  return __builtin_bit_cast(unsigned, __builtin_convertvector(v, bf16x2));
}
__device__ __forceinline__ float bf_lo(unsigned u) { return __builtin_bit_cast(float, u << 16); }
__device__ __forceinline__ float bf_hi(unsigned u) { return __builtin_bit_cast(float, u & 0xffff0000u); }

__device__ __forceinline__ __amdgpu_buffer_rsrc_t rsrc_of(const void* p, unsigned bytes) {
  return __builtin_amdgcn_make_buffer_rsrc(const_cast<void*>(p), 0, bytes, 0x00020000);
}
__device__ __forceinline__ char* uniform_ptr(char* p) {
  const u64 a = (u64)p;
  const unsigned lo = __builtin_amdgcn_readfirstlane((unsigned)a), hi = __builtin_amdgcn_readfirstlane((unsigned)(a >> 32));
  return (char*)(((u64)hi << 32) | lo);
}
#define WAIT_V_(n) asm volatile("s_waitcnt vmcnt(" #n ")" ::: "memory")
#define WAIT_V(n) WAIT_V_(n)
#define XLIN 0
#define XTH 16
#define P_BAR 4
#define PW 4
#define REL_U 3
#define SB __builtin_amdgcn_sched_barrier(0)
#define BAR do { __builtin_amdgcn_s_waitcnt(0xC07F);   __builtin_amdgcn_s_barrier(); SB; } while (0)

#define ACQ __builtin_amdgcn_fence(__ATOMIC_ACQUIRE, "")
#define STAMP_AT(k) do { } while (0)

#define SLP 2
__device__ __forceinline__ void wait_words(const unsigned* p, bool need, unsigned want, unsigned* tmo,
                                           unsigned call, int lane) {
  const u64 t0 = __builtin_amdgcn_s_memrealtime();
  for (;;) {
    unsigned v = want;
    if (need) v = __hip_atomic_load(p, __ATOMIC_RELAXED, __HIP_MEMORY_SCOPE_SYSTEM);
    if (__all((int)(v - want) >= 0)) break;
    const u64 dt = __builtin_amdgcn_s_memrealtime() - t0;
    if (dt > TMO_LOOK) {
      if (__hip_atomic_load(tmo, __ATOMIC_RELAXED, __HIP_MEMORY_SCOPE_AGENT) == call) break;
      if (dt > DEADLINE) {
        if (lane == 0) __hip_atomic_store(tmo, call, __ATOMIC_RELAXED, __HIP_MEMORY_SCOPE_AGENT);
        break;
      }
    }
    __builtin_amdgcn_s_sleep(SLP);
  }
}

__global__ void __launch_bounds__(NTHR, 2) gemm_rs_kernel(Params P) {
  __shared__ __attribute__((aligned(16))) char lds[LDS_B];
  const int tid = threadIdx.x;
  const int lane = tid & 63;
  const int wv = __builtin_amdgcn_readfirstlane(tid >> 6);
  const int l15 = lane & 15, q4 = lane >> 4;
  const int rank = P.rank;
  const int ri = rank >> 2, rj = (rank >> 1) & 1, rh = rank & 1;
  char* const me = P.ws[rank];
  unsigned* const tmo = (unsigned*)(me + OFF_TMO);
  unsigned* const misc = (unsigned*)(lds + LDS_MISC);

  const int b = __builtin_amdgcn_readfirstlane(blockIdx.x), xcd = b & 7, idx = b >> 3;
  STAMP_AT(0);
  const bool producer = idx < 4;
  int rb, cbi;
  if (producer) { rb = 2 * rank + (xcd & 1); cbi = ((xcd >> 1) & 1) * 4 + idx; }
  else { const int t = (xcd & 3) * 28 + idx - 4; rb = (2 * rank + 2 + (t >> 3)) & 15; cbi = t & 7; }
  const int cb = 2 * cbi + (xcd >> 2);
  const int owner = rb >> 1;

  const unsigned lx = (unsigned)((16 * wv + l15) * KS + 4 * q4) * 4u;
  const unsigned lw = (unsigned)((32 * (wv >> 1) + 4 * q4) * NN + 64 * (wv & 1) + 4 * l15) * 4u;
  auto load_item = [&](f32x4* v, int q, int dst, int hh) {
    const int blk = (q & 31) >> 1, kb = 256 * hh + 128 * (q & 1);
    const bool isx = !(q & 32);
    const auto rs = rsrc_of(uniform_ptr((char*)(isx ? P.x : P.w)), 4096u * KS * 4u);
    const unsigned so = isx ? (unsigned)((512 * (blk >> 1) + 256 * (dst >> 2) + 128 * (blk & 1)) * KS + kb) * 4u
                            : (unsigned)(kb * NN + 2048 * ((dst >> 1) & 1) + 128 * blk) * 4u;
    const unsigned sa = isx ? 64u : NN * 4u, sb = isx ? 256u : 16u * NN * 4u;
    const unsigned vo = isx ? lx : lw;
#pragma unroll
    for (int e = 0; e < 8; ++e)
      v[e] = __builtin_bit_cast(f32x4, __builtin_amdgcn_raw_buffer_load_b128(rs, vo, so + (e & 3) * sa + (e >> 2) * sb, 0));
  };
  auto convert_item = [&](u32x4* f, const f32x4* v, int q) {
    if (q < 32) {
#pragma unroll
      for (int u = 0; u < 4; ++u)
        f[u] = u32x4{pack2(v[2 * u][0], v[2 * u][1]), pack2(v[2 * u][2], v[2 * u][3]),
                     pack2(v[2 * u + 1][0], v[2 * u + 1][1]), pack2(v[2 * u + 1][2], v[2 * u + 1][3])};
    } else {
#pragma unroll
      for (int u = 0; u < 4; ++u)
        f[u] = u32x4{pack2(v[0][u], v[1][u]), pack2(v[2][u], v[3][u]), pack2(v[4][u], v[5][u]), pack2(v[6][u], v[7][u])};
    }
  };
  auto item_base = [&](int q, int chunk) -> unsigned {
    const unsigned pb = (q < 32 ? OFF_XS : OFF_WS) + (unsigned)(chunk * 16 + ((q & 31) >> 1)) * 65536u + (q & 1) * 32768u;
    return pb + (q < 32 && !XLIN ? wv * 1024u : wv * 4096u);
  };
  auto item_flag = [&](int q, int chunk) -> unsigned {
    return (q < 32 ? OFF_XF : OFF_WF) + ((q & 31) >> 1) * 128u + chunk * 8u + (q & 1) * 4u;
  };
  auto draw_call = [&]() {
    if (tid < 48 && tid != 16) misc[tid] = 0;
    if (tid == 0)
      misc[16] = __hip_atomic_fetch_add((unsigned*)(me + OFF_CALL + b * 128), 1u, __ATOMIC_RELAXED,
                                        __HIP_MEMORY_SCOPE_AGENT) + 1u;
  };
  unsigned call;
  if (b < 24) {
    const int dst = ((rank | 1) + 1 + b % 6) & 7, slot = b / 6 + 32;
    u32x4 g[8][4];
    f32x4 va[8], vb[8];
    load_item(va, slot | 32, dst, dst & 1);
    load_item(vb, (slot + 4) | 32, dst, dst & 1);
    draw_call();
#pragma unroll
    for (int n = 0; n < 8; n += 2) {
      convert_item(g[n], va, 32);
#pragma unroll
      for (int u = 0; u < 4; ++u) asm volatile("" : "+v"(g[n][u]));
      if (n < 6) load_item(va, (slot + 4 * (n + 2)) | 32, dst, dst & 1);
      convert_item(g[n + 1], vb, 32);
#pragma unroll
      for (int u = 0; u < 4; ++u) asm volatile("" : "+v"(g[n + 1][u]));
      if (n < 6) load_item(vb, (slot + 4 * (n + 3)) | 32, dst, dst & 1);
    }
    __syncthreads();
    call = misc[16];
    if (wv == 0) wait_words((const unsigned*)(me + OFF_XD + dst * 128), lane == 0, (call - 1u) * 32u + XTH, tmo, call, lane);
    __syncthreads();
    STAMP_AT(11);
    const auto rd = rsrc_of(uniform_ptr(P.ws[dst]), WS_NEED);
    auto publish = [&](int n) {
      unsigned tk = 0;
      if (lane == 0) tk = __hip_atomic_fetch_add(misc + n, 1u, __ATOMIC_RELAXED, __HIP_MEMORY_SCOPE_WORKGROUP);
      if (__builtin_amdgcn_readfirstlane(tk) == 7 && lane == 0)
        __builtin_amdgcn_raw_buffer_store_b32(call, rd, 0, item_flag(slot + 4 * n, rank), AUX_SYS);
    };
    auto release = [&]() {
      unsigned tk = 0;
      if (lane == 0) tk = __hip_atomic_fetch_add(misc + 17, 1u, __ATOMIC_RELAXED, __HIP_MEMORY_SCOPE_WORKGROUP);
      if (__builtin_amdgcn_readfirstlane(tk) == 7 && lane == 0)
        __hip_atomic_fetch_add((unsigned*)(me + OFF_PD + dst * 128), 1u, __ATOMIC_RELAXED, __HIP_MEMORY_SCOPE_AGENT);
    };
#pragma unroll
    for (int n = 0; n < 8; ++n) {
      const unsigned sb = item_base(slot + 4 * n, rank);
#pragma unroll
      for (int u = 0; u < 4; ++u) {
        if (u % P_BAR == 0) __builtin_amdgcn_s_barrier();
        __builtin_amdgcn_raw_buffer_store_b128(g[n][u], rd, lane * 16, sb + u * 1024u, AUX_SYS);
        WAIT_V(PW);
        if (n > 0 && u == PW - 1) publish(n - 1);
        if (n == 7 && u == REL_U) release();
      }
    }
    WAIT_V(0);
    publish(7);
  } else {
    draw_call();
#pragma unroll
    for (int pass = 0; pass < 2; ++pass) {
      const int t = pass == 0 ? b - 128 : b - 24;
      const bool on = pass == 0 ? t >= 0 : t < 192;
      const int q = pass == 0 ? t >> 1 : t / 6;
      const int dst = pass == 0 ? rank : ((rank | 1) + 1 + t % 6) & 7;
      const int hh = pass == 0 ? t & 1 : dst & 1;
      const int chunk = pass == 0 ? (rank & 6) + hh : rank;
      const auto rd = rsrc_of(uniform_ptr(P.ws[dst & 7]), WS_NEED);
      if (on) {
        f32x4 v[8];
        u32x4 f[4];
        load_item(v, q, dst, hh);
        convert_item(f, v, q);
        const unsigned sb = item_base(q, chunk), fs = q < 32 && !XLIN ? 8192u : 1024u;
#pragma unroll
        for (int u = 0; u < 4; ++u) {
          __builtin_amdgcn_raw_buffer_store_b128(f[u], rd, lane * 16, sb + u * fs, AUX_SYS);
          WAIT_V(0);
        }
        WAIT_V(0);
      }
      __syncthreads();
      call = misc[16];
      if (on && tid == 0) {
        __builtin_amdgcn_raw_buffer_store_b32(call, rd, 0, item_flag(q, chunk), AUX_SYS);
        if (pass) __hip_atomic_fetch_add((unsigned*)(me + OFF_XD + dst * 128), 1u, __ATOMIC_RELAXED, __HIP_MEMORY_SCOPE_AGENT);
      }
    }
  }
  STAMP_AT(1);

  f32x4 acc[4][4];
#pragma unroll
  for (int mi = 0; mi < 4; ++mi)
#pragma unroll
    for (int u = 0; u < 4; ++u) acc[mi][u] = f32x4{0.f, 0.f, 0.f, 0.f};
  const int ks = wv >> 2, wm = (wv >> 1) & 1, wn = wv & 1;
  {
    const char* gx = me + OFF_XS + rb * 65536 + wv * 2048 + lane * 16;
    const char* gw = me + OFF_WS + cb * 65536 + wv * 2048 + lane * 16;
    const char* lb = lds + lane * 16 + ks * 8192;
    auto stage1 = [&](int s, int part) {
      const unsigned gc = (unsigned)(((rank & 6) + (s >> 2)) & 7) * (16u * 65536u);
      const unsigned go = gc + (unsigned)(s & 3) * 16384u;
      char* l = lds + (s & 3) * STAGE_B + wv * 2048;
      if (XLIN && !(part & 2)) {
        __builtin_amdgcn_global_load_lds((const unsigned*)(gx + wv * 2048 + gc + (s & 2) * 16384 + (s & 1) * 2048 + (part & 1) * 1024),
                                         (unsigned*)(lds + (s & 3) * STAGE_B + wv * 1024 + (part & 1) * 8192), 16, 0, 0);
        return;
      }
      const char* g = (part & 2) ? gw : gx;
      __builtin_amdgcn_global_load_lds((const unsigned*)(g + go + (part & 1) * 1024),
                                       (unsigned*)(l + (part >> 1) * 16384 + (part & 1) * 1024), 16, 0, 0);
    };
    bf16x8 xa[4], wa[4], xb[4], wb[4];
    auto rd = [&](bf16x8* xf, bf16x8* wf, int s) {
      const char* l = lb + (s & 3) * STAGE_B;
#pragma unroll
      for (int i = 0; i < 4; ++i) {
        wf[i] = *(const bf16x8*)(l + 16384 + (4 * wn + i) * 1024);
        xf[i] = *(const bf16x8*)(l + (4 * wm + i) * 1024);
      }
    };
    auto mm = [&](bf16x8* xf, bf16x8* wf, int snext, bool ld) {
#pragma unroll
      for (int mi = 0; mi < 4; ++mi) {
#pragma unroll
        for (int u = 0; u < 4; ++u)
          acc[mi][u] = __builtin_amdgcn_mfma_f32_16x16x32_bf16(wf[u], xf[mi], acc[mi][u], 0, 0, 0);
        if (ld) stage1(snext, mi);
        SB;
      }
    };
#pragma unroll 1
    for (int run = 0; run < 2; ++run) {
      const int s0 = run * 8, s1 = 8 + run * 24;
      if (wv == 0) {
        const unsigned* fp = lane < 16 ? (const unsigned*)(me + OFF_XF + rb * 128) + lane
                                       : (const unsigned*)(me + OFF_WF + cb * 128) + l15;
        const bool own = (l15 >> 2) == (rank >> 1);
        wait_words(fp, lane < 32 && own == (run == 0), call, tmo, call, lane);
        ACQ;
        WAIT_V(0);
      }
      __syncthreads();
      STAMP_AT(run == 0 ? 9 : 2);
#pragma unroll
      for (int s = 0; s < 4; ++s)
#pragma unroll
        for (int part = 0; part < 4; ++part) stage1(s0 + s, part);
      WAIT_V(12);
      BAR;
      rd(xa, wa, s0);
#pragma unroll 1
      for (int s = s0; s < s1 - 4; s += 2) {
        WAIT_V(8); BAR; rd(xb, wb, s + 1); SB; mm(xa, wa, s + 4, true);
        WAIT_V(8); BAR; rd(xa, wa, s + 2); SB; mm(xb, wb, s + 5, true);
      }
      WAIT_V(8); BAR; rd(xb, wb, 1); SB; mm(xa, wa, 0, false);
      WAIT_V(4); BAR; rd(xa, wa, 2); SB; mm(xb, wb, 0, false);
      WAIT_V(0); BAR; rd(xb, wb, 3); SB; mm(xa, wa, 0, false);
      mm(xb, wb, 0, false);
      if (run == 0) STAMP_AT(10);
    }
  }
  f32x4 fin[2][4];
  {
    char* lo = lds + wv * 8192 + lane * 16;
    const char* li = lds + (wv ^ 4) * 8192 + lane * 16;
#pragma unroll
    for (int mi = 0; mi < 2; ++mi)
#pragma unroll
      for (int u = 0; u < 4; ++u) *(f32x4*)(lo + (mi * 4 + u) * 1024) = ks ? acc[mi][u] : acc[2 + mi][u];
    __syncthreads();
#pragma unroll
    for (int mi = 0; mi < 2; ++mi)
#pragma unroll
      for (int u = 0; u < 4; ++u) fin[mi][u] = (ks ? acc[2 + mi][u] : acc[mi][u]) + *(const f32x4*)(li + (mi * 4 + u) * 1024);
  }
  STAMP_AT(3);

  if ((owner | 1) != (rank | 1)) {
    if (wv == 0) wait_words((const unsigned*)(me + OFF_PD + owner * 128), lane == 0, call * 4u, tmo, call, lane);
    __syncthreads();
  }
  STAMP_AT(4);
  {
    const int pos = ((ri * 2 + (rb & 1)) * 2 + rj) * 16 + cb;
    char* const wo = P.ws[owner & 7];
    const auto rp = rsrc_of(wo + OFF_PS + (unsigned)(pos * 2 + rh) * 32768u, 32768u);
#pragma unroll
    for (int mi = 0; mi < 2; ++mi)
#pragma unroll
      for (int s = 0; s < 2; ++s) {
        const u32x4 o = {pack2(fin[mi][0][2 * s], fin[mi][1][2 * s]), pack2(fin[mi][2][2 * s], fin[mi][3][2 * s]),
                         pack2(fin[mi][0][2 * s + 1], fin[mi][1][2 * s + 1]),
                         pack2(fin[mi][2][2 * s + 1], fin[mi][3][2 * s + 1])};
        __builtin_amdgcn_raw_buffer_store_b128(o, rp, (unsigned)((((4 * wm + 2 * ks + mi) * 2 + wn) * 2 + s) * 1024 + lane * 16), 0, AUX_SYS);
        WAIT_V(0);
      }
    WAIT_V(0);
    __syncthreads();
    if (tid == 0)
      __hip_atomic_store((unsigned*)(wo + OFF_PF + pos * 128 + rh * 4), call, __ATOMIC_RELAXED, __HIP_MEMORY_SCOPE_SYSTEM);
  }
  STAMP_AT(5);

  const int unit = (rb >> 1) * 32 + cb * 2 + (rb & 1);
  const int rpos = unit >> 1, rhalf = unit & 1;
  if (wv == 0) {
    wait_words((const unsigned*)(me + OFF_PF + rpos * 128) + (lane & 1), lane < 2, call, tmo, call, lane);
    ACQ;
    WAIT_V(0);
  }
  __syncthreads();
  STAMP_AT(6);
  float y[16];
  {
    const char* pp = me + OFF_PS + rpos * 65536 + rhalf * 16384 + wv * 2048 + lane * 16;
    const u32x4 a0 = *(const u32x4*)(pp), a1 = *(const u32x4*)(pp + 1024);
    const u32x4 b0 = *(const u32x4*)(pp + 32768), b1 = *(const u32x4*)(pp + 32768 + 1024);
#pragma unroll
    for (int d = 0; d < 4; ++d) {
      y[2 * d] = bf_lo(a0[d]) + bf_lo(b0[d]);
      y[2 * d + 1] = bf_hi(a0[d]) + bf_hi(b0[d]);
      y[8 + 2 * d] = bf_lo(a1[d]) + bf_lo(b1[d]);
      y[8 + 2 * d + 1] = bf_hi(a1[d]) + bf_hi(b1[d]);
    }
  }
  unsigned am = 0;
#pragma unroll
  for (int t = 0; t < 16; ++t) am = max(am, __builtin_bit_cast(unsigned, y[t]) & 0x7fffffffu);
#pragma unroll
  for (int o = 32; o >= 1; o >>= 1) am = max(am, (unsigned)__shfl_xor((int)am, o));
  if (lane == 0) __hip_atomic_fetch_max(misc + 20, am, __ATOMIC_RELAXED, __HIP_MEMORY_SCOPE_WORKGROUP);
  __syncthreads();
  if (wv == 0) {
    unsigned t = 0;
    if (lane == 0) {
      __hip_atomic_fetch_max((u64*)(me + OFF_AMX), ((u64)call << 32) | misc[20], __ATOMIC_RELAXED, __HIP_MEMORY_SCOPE_AGENT);
      WAIT_V(0);
      t = __hip_atomic_fetch_add((unsigned*)(me + OFF_ACT), 1u, __ATOMIC_RELAXED, __HIP_MEMORY_SCOPE_AGENT);
    }
    t = __builtin_amdgcn_readfirstlane(t);
    if (t + 1u == call * (unsigned)NWG) {
      const u64 g = __hip_atomic_load((u64*)(me + OFF_AMX), __ATOMIC_RELAXED, __HIP_MEMORY_SCOPE_AGENT);
      __hip_atomic_store((u64*)(P.ws[lane & 7] + OFF_AG + (lane >> 3) * 128 + rank * 8), g, __ATOMIC_RELAXED,
                         __HIP_MEMORY_SCOPE_SYSTEM);
    }
    const u64* gp = lane == 63 ? (const u64*)tmo : (const u64*)(me + OFF_AG + xcd * 128) + (lane & 7);
    const bool act = lane < 8 || lane == 63;
    const u64 t0 = __builtin_amdgcn_s_memrealtime();
    unsigned gm, bad = 0;
    for (;;) {
      u64 g = 0;
      if (act) g = __hip_atomic_load(gp, __ATOMIC_RELAXED, __HIP_MEMORY_SCOPE_SYSTEM);
      gm = lane < 8 ? (unsigned)g : 0u;
      if (__any(lane == 63 && (unsigned)g == call)) { bad = 1; break; }
      if (__all(lane >= 8 || (int)((unsigned)(g >> 32) - call) >= 0)) break;
      if (__builtin_amdgcn_s_memrealtime() - t0 > DEADLINE) {
        if (lane == 0) __hip_atomic_store(tmo, call, __ATOMIC_RELAXED, __HIP_MEMORY_SCOPE_AGENT);
        bad = 1;
        break;
      }
      __builtin_amdgcn_s_sleep(1);
    }
#pragma unroll
    for (int o = 4; o >= 1; o >>= 1) gm = max(gm, (unsigned)__shfl_xor((int)gm, o));
    if (lane == 0) { misc[21] = gm; misc[22] = bad; }
  }
  __syncthreads();
  STAMP_AT(7);
  {
    const float amax = __builtin_bit_cast(float, misc[21]);
    const float inv = 448.0f / amax;
    u32x4 o;
#pragma unroll
    for (int d = 0; d < 4; ++d) {
      float a[4];
#pragma unroll
      for (int e = 0; e < 4; ++e) a[e] = y[4 * d + e] * inv;
      int r = __builtin_amdgcn_cvt_pk_fp8_f32(a[0], a[1], 0, false);
      r = __builtin_amdgcn_cvt_pk_fp8_f32(a[2], a[3], r, true);
      o[d] = (unsigned)r;
    }
    if (misc[22]) o = u32x4{0x7f7f7f7fu, 0x7f7f7f7fu, 0x7f7f7f7fu, 0x7f7f7f7fu};
    const int pi = rpos >> 6, prl = (rpos >> 5) & 1, pj = (rpos >> 4) & 1, pcb = rpos & 15;
    const int row = 256 * pi + 128 * prl + 16 * (rhalf * 4 + (wv >> 1)) + l15;
    const int col = 2048 * pj + 128 * pcb + 64 * (wv & 1) + 16 * q4;
    *(u32x4*)(P.out + (size_t)row * NN + col) = o;
  }
  STAMP_AT(8);
}

}

extern "C" void kernel_launch(int rank, int nranks, ncclComm_t comm, hipStream_t stream,
                              void* const* d_in, const int* in_sizes, int n_in, void* d_out, int out_size,
                              const int* in_dtypes, int out_dtype, void* const* d_out_all,
                              void* const* d_ws_all, size_t ws_bytes) {
  (void)comm; (void)in_dtypes; (void)out_dtype; (void)d_out_all;
  if (nranks != NR || (unsigned)rank >= (unsigned)NR || n_in < 2 || ws_bytes < WS_NEED ||
      in_sizes[0] != 4096 * KS || in_sizes[1] != KS * NN || out_size != 512 * NN) {
    fprintf(stderr, "unexpected problem shape\n");
    return;
  }
  Params p;
  p.x = (const float*)d_in[0];
  p.w = (const float*)d_in[1];
  p.out = (unsigned char*)d_out;
  for (int r = 0; r < NR; ++r) p.ws[r] = (char*)d_ws_all[r];
  p.rank = rank;
  gemm_rs_kernel<<<NWG, NTHR, 0, stream>>>(p);
  HIP_CHECK(hipGetLastError());
}
```

```cpp
#include <hip/hip_runtime.h>
#include <rccl/rccl.h>
#include <cstdio>

#define HIP_CHECK(expr) do { hipError_t _e = (expr); if (_e != hipSuccess) { \
    fprintf(stderr, "HIP error %s at %s:%d\n", hipGetErrorString(_e), __FILE__, __LINE__); \
    return; } } while (0)

namespace {

constexpr int NR = 8;
constexpr int NN = 4096;
constexpr int KS = 512;
constexpr int NWG = 256;
constexpr int NTHR = 512;

#define EC 8
#define WB 7
#define WP 7
#define PP 0
#define EW 1
#define LDELAY 150
#define PL 1
#define ORD 1
#define PRIO 1
#define SECOND 0
#define SIG 3
#define XPA 5
#define XPB 0
#define WF 1
#define EDTH 0
#define GAUX 17
#define ATH 8
#define SLP 2

constexpr unsigned OFF_CALL = 0;
constexpr unsigned OFF_TMO = 32768;
constexpr unsigned OFF_ACT = OFF_TMO + 1024;
constexpr unsigned OFF_LC = OFF_TMO + 2048;
constexpr unsigned OFF_LT = OFF_TMO + 3072;
constexpr unsigned OFF_LU = OFF_TMO + 128;
constexpr unsigned OFF_XF = OFF_TMO + 4096;
constexpr unsigned OFF_WF = OFF_XF + 2048;
constexpr unsigned OFF_AG = OFF_TMO + 8192;
constexpr unsigned OFF_DBG = 65536;
constexpr unsigned OFF_PF = 131072;
constexpr unsigned OFF_XS = 1u << 20;
constexpr unsigned OFF_WS = OFF_XS + (10u << 20);
constexpr unsigned OFF_PS = OFF_WS + (10u << 20);
constexpr unsigned WS_NEED = OFF_PS + 144u * 3u * 32768u;

constexpr unsigned long long DEADLINE = 100000000ull;
constexpr unsigned long long TMO_LOOK = 20000;
constexpr int AUX_SYS = 17;

constexpr int STAGE_B = 32768;
constexpr int LDS_MISC = 4 * STAGE_B;
constexpr int LDS_B = LDS_MISC + 256;

typedef __bf16 bf16x8 __attribute__((ext_vector_type(8)));
typedef __bf16 bf16x2 __attribute__((ext_vector_type(2)));
typedef float f32x2 __attribute__((ext_vector_type(2)));
typedef float f32x4 __attribute__((ext_vector_type(4)));
typedef unsigned u32x4 __attribute__((ext_vector_type(4)));
typedef unsigned long long u64;
typedef __attribute__((address_space(1))) unsigned long long gu64;

struct Params {
  const float* x;
  const float* w;
  unsigned char* out;
  char* ws[NR];
  int rank;
};

__device__ __forceinline__ unsigned pack2(float lo, float hi) {
  f32x2 v = {lo, hi};
  return __builtin_bit_cast(unsigned, __builtin_convertvector(v, bf16x2));
}
__device__ __forceinline__ float bf_lo(unsigned u) { return __builtin_bit_cast(float, u << 16); }
__device__ __forceinline__ float bf_hi(unsigned u) { return __builtin_bit_cast(float, u & 0xffff0000u); }

__device__ __forceinline__ __amdgpu_buffer_rsrc_t rsrc_of(const void* p, unsigned bytes) {
  return __builtin_amdgcn_make_buffer_rsrc(const_cast<void*>(p), 0, bytes, 0x00020000);
}
__device__ __forceinline__ void st128(u32x4 v, __amdgpu_buffer_rsrc_t r, unsigned vo, unsigned so) {
  __builtin_amdgcn_raw_buffer_store_b128(v, r, vo, so, AUX_SYS);
  asm volatile("s_nop 1" :: "v"(v) : "memory");
}
__device__ __forceinline__ unsigned wave_max(unsigned v) {
  v = max(v, (unsigned)__builtin_amdgcn_mov_dpp((int)v, 0xB1, 0xf, 0xf, true));
  v = max(v, (unsigned)__builtin_amdgcn_mov_dpp((int)v, 0x4E, 0xf, 0xf, true));
  v = max(v, (unsigned)__builtin_amdgcn_mov_dpp((int)v, 0x141, 0xf, 0xf, true));
  v = max(v, (unsigned)__builtin_amdgcn_mov_dpp((int)v, 0x140, 0xf, 0xf, true));
  const unsigned a = __builtin_amdgcn_readlane(v, 0), b = __builtin_amdgcn_readlane(v, 16);
  const unsigned c = __builtin_amdgcn_readlane(v, 32), d = __builtin_amdgcn_readlane(v, 48);
  return max(max(a, b), max(c, d));
}
__device__ __forceinline__ char* uniform_ptr(char* p) {
  const u64 a = (u64)p;
  const unsigned lo = __builtin_amdgcn_readfirstlane((unsigned)a), hi = __builtin_amdgcn_readfirstlane((unsigned)(a >> 32));
  return (char*)(((u64)hi << 32) | lo);
}
#define INL __attribute__((always_inline))
#define UNI(x) __builtin_amdgcn_readfirstlane(x)
#define WAIT_V_(n) asm volatile("s_waitcnt vmcnt(" #n ")" ::: "memory")
#define WAIT_V(n) WAIT_V_(n)
#define SB __builtin_amdgcn_sched_barrier(0)
#define BAR do { asm volatile("" ::: "memory"); __builtin_amdgcn_s_waitcnt(0xC07F);   \
                 __builtin_amdgcn_s_barrier(); asm volatile("" ::: "memory"); SB; } while (0)
#define ACQ __builtin_amdgcn_fence(__ATOMIC_ACQUIRE, "wavefront")

#define STAMP_AT(k) do { } while (0)

__device__ __forceinline__ bool expired(u64 t0, unsigned* tmo, unsigned call, int lane) {
  const u64 dt = __builtin_amdgcn_s_memrealtime() - t0;
  if (dt > TMO_LOOK) {
    if (UNI(__hip_atomic_load(tmo, __ATOMIC_RELAXED, __HIP_MEMORY_SCOPE_AGENT)) == call) return true;
    if (dt > DEADLINE) {
      if (lane == 0) __hip_atomic_store(tmo, call, __ATOMIC_RELAXED, __HIP_MEMORY_SCOPE_AGENT);
      return true;
    }
  }
  return false;
}
__device__ __forceinline__ void wait_words(const unsigned* p, bool need, unsigned want, unsigned* tmo,
                                           unsigned call, int lane) {
  const u64 t0 = __builtin_amdgcn_s_memrealtime();
  for (;;) {
    unsigned v = want;
    if (need) v = __hip_atomic_load(p, __ATOMIC_RELAXED, __HIP_MEMORY_SCOPE_SYSTEM);
    if (__all((int)(v - want) >= 0)) break;
    if (expired(t0, tmo, call, lane)) break;
    __builtin_amdgcn_s_sleep(SLP);
  }
}

__global__ void __launch_bounds__(NTHR, 2) gemm_rs_kernel(Params P) {
  __shared__ __attribute__((aligned(16))) char lds[LDS_B];
  const int tid = threadIdx.x;
  const int lane = tid & 63;
  const int wv = UNI(tid >> 6);
  const int l15 = lane & 15, q4 = lane >> 4;
  const int rank = P.rank;
  const int rh = rank & 1, pair0 = rank & 6;
  char* const me = P.ws[rank];
  unsigned* const tmo = (unsigned*)(me + OFF_TMO);
  unsigned* const misc = (unsigned*)(lds + LDS_MISC);

  const int b = UNI(blockIdx.x), xcd = b & 7, idx = b >> 3;
  STAMP_AT(0);
  const u64 tstart = __builtin_amdgcn_s_memrealtime();
  int orel, cq;
  const int p0 = (2 * xcd) % 6;
  if (idx < 4) { orel = rh; cq = idx; }
  else if (idx < 6) { orel = 2 + p0 + idx - 4; cq = 3; }
  else if (idx < 28) { const int ro = (idx - 6) % 7; orel = ro ? ro + 1 : rh ^ 1; cq = (idx - 6) / 7; }
  else { orel = 2 + (p0 + 2 + idx - 28) % 6; cq = 3; }
  int cb = 4 * cq + (xcd >> 1);
  const int rpos = orel * 16 + cb, rhalf = xcd & 1;
  auto giver = [](int rel, int c) { return ((c >> 2) ^ c ^ (rel >> 1)) & 1; };

  const unsigned lx = (unsigned)((16 * wv + (lane >> 3)) * KS + 4 * ((lane ^ (lane >> 3)) & 7)) * 4u;
  const unsigned lw = (unsigned)((64 * (wv >> 2) + 16 * ((wv >> 1) & 1) + 4 * q4) * NN + 64 * (wv & 1) + 4 * l15) * 4u;
  char* const px = uniform_ptr((char*)P.x);
  char* const pw = uniform_ptr((char*)P.w);
  struct Src { __amdgpu_buffer_rsrc_t rs; unsigned so, s0, s1, s2, vo; };
  auto src_of = [&](int q, int dst, int hh) INL {
    const int blk = (q & 31) >> 1, kb = 256 * hh + 128 * (q & 1);
    const bool isx = !(q & 32);
    Src r;
    r.rs = rsrc_of(uniform_ptr(pw + (isx ? px - pw : 0)), 4096u * KS * 4u);
    r.so = UNI(isx ? (unsigned)((512 * (blk >> 1) + 256 * (dst >> 2) + 128 * (blk & 1)) * KS + kb) * 4u
                   : (unsigned)(kb * NN + 2048 * ((dst >> 1) & 1) + 128 * blk) * 4u);
    r.s0 = UNI(isx ? 128u : NN * 4u);
    r.s1 = UNI(isx ? 8u * KS * 4u : 2u * NN * 4u);
    r.s2 = UNI(isx ? 256u : 32u * NN * 4u);
    r.vo = lw + (isx ? lx - lw : 0u);
    return r;
  };
  auto load1 = [&](const Src& r, int t) INL {
    return __builtin_bit_cast(f32x4, __builtin_amdgcn_raw_buffer_load_b128(r.rs, r.vo, r.so + (t & 1) * r.s0 + ((t >> 1) & 1) * r.s1 + (t >> 2) * r.s2, 0));
  };
  auto load_item = [&](f32x4* v, int q, int dst, int hh) INL {
    const Src r = src_of(q, dst, hh);
#pragma unroll
    for (int t = 0; t < 8; ++t) v[t] = load1(r, t);
  };
  auto convert_item = [&](u32x4* f, const f32x4* v, int q) INL {
    if (q < 32) {
#pragma unroll
      for (int u = 0; u < 4; ++u)
        f[u] = u32x4{pack2(v[2 * u][0], v[2 * u][1]), pack2(v[2 * u][2], v[2 * u][3]),
                     pack2(v[2 * u + 1][0], v[2 * u + 1][1]), pack2(v[2 * u + 1][2], v[2 * u + 1][3])};
    } else {
#pragma unroll
      for (int u = 0; u < 4; ++u)
        f[u] = u32x4{pack2(v[0][u], v[1][u]), pack2(v[2][u], v[3][u]), pack2(v[4][u], v[5][u]), pack2(v[6][u], v[7][u])};
    }
#pragma unroll
    for (int u = 0; u < 4; ++u) asm volatile("" : "+v"(f[u]));
  };
  auto item_base = [&](int q, int chunk) -> unsigned {
    const unsigned pb = (q < 32 ? OFF_XS : OFF_WS) + (unsigned)(chunk * 16 + ((q & 31) >> 1)) * 65536u + (q & 1) * 32768u;
    return pb + (q < 32 ? wv * 2048u : wv * 4096u);
  };
  auto frag_off = [](int q, int u) INL { return (u & 1) * 1024u + (u >> 1) * (q < 32 ? 16384u : 2048u); };
  auto item_flag = [&](int q, int chunk) -> unsigned {
    return (q < 32 ? OFF_XF : OFF_WF) + ((q & 31) >> 1) * 128u + chunk * 8u + (q & 1) * 4u;
  };
  auto draw_call = [&]() {
    if (tid < 32 && tid != 16) misc[tid] = 0;
    if (tid == 0)
      misc[16] = __hip_atomic_fetch_add((unsigned*)(me + OFF_CALL + b * 128), 1u, __ATOMIC_RELAXED,
                                        __HIP_MEMORY_SCOPE_AGENT) + 1u;
  };
  const auto rme = rsrc_of(uniform_ptr(me), WS_NEED);
  unsigned call;
  if (idx < 6) {
    const int dst = (pair0 + 2 + idx) & 7;
    int last = 0;
    auto unit_q = [&](int n) INL {
      int j = (n >> 1) * 4 + (xcd >> 1);
      if (n >= 7) return 32 + (12 + (last & 3)) * 2 + 1;
      if ((xcd ^ (n >> 1) ^ idx) & 1) return 32 + j * 2 + (n & 1);
      if (n == 6) return dst * 4 + (xcd >> 1);
      if (j < 2) return ((dst ^ 1) * 2 + j) * 2 + (n & 1);
      const int second_blk = j >= 7, jt = j - 2 - second_blk * 5;
      const int en = jt + (jt + 2 >= ((rank - (dst & 6)) & 7));
      return ((((dst & 6) + 2 + en) & 7) * 2 + second_blk) * 2 + (n & 1);
    };
    f32x4 va[8], vb[8];
    load_item(va, unit_q(0), dst, dst & 1);
    load_item(vb, unit_q(1), dst, dst & 1);
    draw_call();
    const auto rd = rsrc_of(uniform_ptr(P.ws[dst]), WS_NEED);
    auto publish = [&](int n) INL {
      unsigned tk = 0;
      const unsigned fo = UNI(item_flag(unit_q(n), rank));
      if (lane == 0) tk = __hip_atomic_fetch_add(misc + n, 1u, __ATOMIC_RELAXED, __HIP_MEMORY_SCOPE_WORKGROUP);
      if (UNI(tk) == 7 && lane == 0) __builtin_amdgcn_raw_buffer_store_b32(call, rd, 0, fo, AUX_SYS);
    };
    auto unit = [&](f32x4* v, f32x4* vn, int n, bool more) INL {
      const int q = unit_q(n);
      if (n == 6) {
        while ((last = UNI(__hip_atomic_load(misc + 17, __ATOMIC_RELAXED, __HIP_MEMORY_SCOPE_WORKGROUP))) == 0) __builtin_amdgcn_s_sleep(1);
        last &= 7;
      }
      u32x4 f[4];
      convert_item(f, v, q);
      const unsigned sb = item_base(q, rank);
      const Src rn = src_of(unit_q(n + 2), dst, dst & 1);
#pragma unroll
      for (int u = 0; u < 4; ++u) {
        if (u == 3 && n == (last < 4 ? 7 : 6) && tid == 0)
          __hip_atomic_fetch_add((unsigned*)(me + OFF_LC + (2 + idx) * 128), 1u, __ATOMIC_RELAXED, __HIP_MEMORY_SCOPE_AGENT);
        st128(f[u], rd, lane * 16, sb + frag_off(q, u));
        if (n == 5 && u == 3 && tid == 0)
          __hip_atomic_store(misc + 17, 8u + (__hip_atomic_fetch_add((unsigned*)(me + OFF_LU + idx * 128), 1u, __ATOMIC_RELAXED, __HIP_MEMORY_SCOPE_AGENT) & 7u),
                             __ATOMIC_RELAXED, __HIP_MEMORY_SCOPE_WORKGROUP);
        if (more) {
          vn[2 * u] = load1(rn, 2 * u);
          vn[2 * u + 1] = load1(rn, 2 * u + 1);
          WAIT_V(2);
        } else {
          WAIT_V(0);
        }
        if (n == 0 && u == 0) { BAR; call = UNI(misc[16]); }
      }
      publish(n);
    };
    unit(va, va, 0, true);
#pragma unroll 1
    for (int n = 1; n < 5; n += 2) {
      unit(vb, vb, n, true);
      unit(va, va, n + 1, true);
    }
    unit(vb, vb, 5, false);
    unit(va, vb, 6, true);
    asm volatile("" :: "v"(vb[0]), "v"(vb[1]), "v"(vb[2]), "v"(vb[3]), "v"(vb[4]), "v"(vb[5]), "v"(vb[6]), "v"(vb[7]));
    if (last < 4) unit(vb, vb, 7, false);
    WAIT_V(0);
  } else if (idx < 20 && (idx - 6) * 8 + xcd < 100) {
    f32x4 va[8], vb[8];
    u32x4 fa[4], fb[4];
    const int e = (idx - 6) * 8 + xcd;
    int q, c0, pd;
    if (e < 64) { q = e; c0 = pair0; pd = rank; }
    else if (e < 96) { q = e - 32; c0 = 8; pd = rank ^ 2; }
    else { q = 4 * rank + e - 96; c0 = 8; pd = rank ^ 4; }
    draw_call();
    while (__builtin_amdgcn_s_memrealtime() - tstart < LDELAY) __builtin_amdgcn_s_sleep(2);
    load_item(va, q, pd, 0);
    load_item(vb, q, pd, 1);
    convert_item(fa, va, q);
    convert_item(fb, vb, q);
    const unsigned sb = item_base(q, c0);
#pragma unroll
    for (int u = 0; u < 4; ++u) {
      st128(fa[u], rme, lane * 16, sb + frag_off(q, u));
      st128(fb[u], rme, lane * 16, sb + (1u << 20) + frag_off(q, u));
    }
    WAIT_V(0);
    BAR;
    call = UNI(misc[16]);
    if (tid < 2) __builtin_amdgcn_raw_buffer_store_b32(call, rme, tid * 8, item_flag(q, c0), AUX_SYS);
  } else {
    draw_call();
    BAR;
    call = UNI(misc[16]);
  }
  STAMP_AT(1);

  f32x4 acc[4][4];
  const int ks = wv >> 2, wm = (wv >> 1) & 1, wn = wv & 1;
  const bool has_brick = idx >= 20;
  const int e = (idx - 20) * 8 + xcd;
  const bool second = orel >= 2 && orel < 6 && !((xcd ^ orel) & 1) && cb < 8 && giver(orel, cb) == rh;
  const char* lbw = lds + 16384 + ks * 8192 + wn * 4096 + lane * 16;
  const char* lbx = lds + (64 * wm + l15) * 128 + (((4 * ks + q4) ^ (lane & 7)) * 16);
  u64 gaddr = (u64)(P.ws[lane & 7] + OFF_AG) + (rank * NWG + b) * 8;
  asm volatile("" : "+v"(gaddr));
  const int job1 = !SECOND ? 2 : idx == 4 || idx == 5 ? 0 : second ? 3 : 2;
  int seg = 0;
#pragma unroll 1
  for (int job = has_brick ? 0 : 1; job < job1;) {
    const bool brick = job == 0;
    const int owner = (pair0 + orel) & 7, rb = 2 * owner + ((xcd ^ orel) & 1);
    const bool early_tile = orel >= 2 && !(rb & 1) && cb < EC;
    const bool hand_over = early_tile && giver(orel, cb) == rh;
    const int quad = 1 + (e >> 5);
    const int trb = brick ? 2 * rank + (e & 1) : rb, tcb = brick ? (e >> 1) & 15 : cb;
    const int cx0 = brick && (quad & 2) ? 8 : pair0, cw0 = brick && (quad & 1) ? 8 : pair0;
    const int skip = orel >= 2 ? orel : 8;
    const int hs = orel >= 2 ? 10 : 12;
    if (seg == 0) {
#pragma unroll
      for (int mi = 0; mi < 4; ++mi)
#pragma unroll
        for (int u = 0; u < 4; ++u) acc[mi][u] = f32x4{0.f, 0.f, 0.f, 0.f};
    }
    if (wv == 0) {
      const int c = (lane & 31) >> 1;
      const unsigned* fp = (const unsigned*)(me + (lane < 32 ? OFF_XF + trb * 128 : OFF_WF + tcb * 128)) + (lane & 31);
      const int crel = (c - pair0) & 7;
      const int lseg = (c & ~1) == (lane < 32 ? cx0 : cw0) ? 0 : c < 8 && crel >= 2 && crel != skip ? 1 + (lane & 1) : 3;
      const u64 t0 = __builtin_amdgcn_s_memrealtime();
      int upto;
      for (;;) {
        unsigned v = call;
        if (lseg < 3 && lseg >= seg) v = __hip_atomic_load(fp, __ATOMIC_RELAXED, __HIP_MEMORY_SCOPE_SYSTEM);
        const bool miss = (int)(v - call) < 0;
        upto = __any(miss && lseg == 0) ? 0 : __any(miss && lseg == 1) ? 1 : __any(miss) ? 2 : 3;
        if (upto > seg) break;
        if (expired(t0, tmo, call, lane)) { upto = 3; break; }
        __builtin_amdgcn_s_sleep(SLP);
      }
      if (lane == 0) misc[18] = brick ? 1 : upto;
      ACQ;
      WAIT_V(0);
    }
    BAR;
    const int upto = UNI(misc[18]);
    const int s0 = seg == 0 ? 0 : 8 + (seg - 1) * hs, ns = 8 + (upto - 1) * hs - s0;
    STAMP_AT(brick ? 14 : seg == 0 ? 9 : upto == 3 ? 2 : 11);
    const char* gx = me + OFF_XS + trb * 65536 + wv * 2048 + lane * 16;
    const char* gw = me + OFF_WS + tcb * 65536 + wv * 2048 + lane * 16;
    unsigned tab;
    {
      const int s = s0 + (lane & 31), kh = s >= 8 + hs, t = s - 8 - kh * hs;
      int c = s < 8 ? s >> 2 : 2 + (t >> 1);
      const int kst = s < 8 ? s & 3 : 2 * kh + (t & 1);
      c += c >= skip;
      const int base = lane < 32 ? cx0 : cw0;
      tab = (unsigned)(base == 8 ? 8 + (c & 1) : (base + c) & 7) * (16u * 65536u) + (unsigned)kst * 16384u;
    }
    auto stage1 = [&](int i, int part) INL {
      const unsigned go = __builtin_amdgcn_readlane(tab, i + (part & 2) * 16);
      char* l = lds + (i & 3) * STAGE_B + wv * 2048;
      const char* g = (part & 2) ? gw : gx;
      __builtin_amdgcn_global_load_lds((const unsigned*)(g + go + (part & 1) * 1024),
                                       (unsigned*)(l + (part >> 1) * 16384 + (part & 1) * 1024), 16, 0, GAUX);
    };
    bf16x8 xa[4], wa[4], xb[4], wb[4];
    auto step = [&](bf16x8* xc, bf16x8* wc, bf16x8* xn, bf16x8* wx, int s, bool ld, bool rd) INL {
      const int bo = ((s + 1) & 3) * STAGE_B;
#pragma unroll
      for (int mi = 0; mi < 4; ++mi) {
        acc[mi][0] = __builtin_amdgcn_mfma_f32_16x16x32_bf16(wc[0], xc[mi], acc[mi][0], 0, 0, 0);
        if (rd) {
          wx[mi] = *(const bf16x8*)(lbw + bo + mi * 1024);
          xn[mi] = *(const bf16x8*)(lbx + bo + mi * 2048);
        }
        SB;
#pragma unroll
        for (int u = 1; u < 4; ++u)
          acc[mi][u] = __builtin_amdgcn_mfma_f32_16x16x32_bf16(wc[u], xc[mi], acc[mi][u], 0, 0, 0);
        if (ld) stage1(s + 4, mi);
        SB;
      }
    };
#pragma unroll
    for (int s = 0; s < 4; ++s)
#pragma unroll
      for (int part = 0; part < 4; ++part) stage1(s, part);
    WAIT_V(12);
    BAR;
#pragma unroll
    for (int i = 0; i < 4; ++i) {
      wa[i] = *(const bf16x8*)(lbw + i * 1024);
      xa[i] = *(const bf16x8*)(lbx + i * 2048);
    }
#pragma unroll 1
    for (int s = 0; s < ns - 4; s += 2) {
      WAIT_V(8); BAR; step(xa, wa, xb, wb, s, true, true);
      WAIT_V(8); BAR; step(xb, wb, xa, wa, s + 1, true, true);
    }
    WAIT_V(8); BAR; step(xa, wa, xb, wb, ns - 4, false, true);
    WAIT_V(4); BAR; step(xb, wb, xa, wa, ns - 3, false, true);
    WAIT_V(0); BAR; step(xa, wa, xb, wb, ns - 2, false, true);
    step(xb, wb, xa, wa, 0, false, false);
    if (!brick && upto < 3) { if (seg == 0) STAMP_AT(10); seg = upto; continue; }
    const int run = brick ? 0 : 2;

    f32x4 fin[2][4];
    {
      char* xb0 = lds + (ns & 2) * STAGE_B + lane * 16;
      char* lo = xb0 + wv * 8192;
      const char* li = xb0 + (wv ^ 4) * 8192;
#pragma unroll
      for (int mi = 0; mi < 2; ++mi)
#pragma unroll
        for (int u = 0; u < 4; ++u) *(f32x4*)(lo + (mi * 4 + u) * 1024) = ks ? acc[mi][u] : acc[2 + mi][u];
      BAR;
#pragma unroll
      for (int mi = 0; mi < 2; ++mi)
#pragma unroll
        for (int u = 0; u < 4; ++u) fin[mi][u] = (ks ? acc[2 + mi][u] : acc[mi][u]) + *(const f32x4*)(li + (mi * 4 + u) * 1024);
    }
    if (run == 2) STAMP_AT(3);
    const unsigned wo = (unsigned)(((4 * wm + 2 * ks) * 2 + wn) * 2048);
    const int hpos = 128 + (orel - 2) / 3 * 8 + cb, hslot = (orel - 2) % 3;
    if (run == 2 && early_tile && !hand_over) {
      if (wv == 0) wait_words((const unsigned*)(me + OFF_PF + hpos * 128 + hslot * 4), lane == 0, call, tmo, call, lane);
      BAR;
      const unsigned sh = OFF_PS + (unsigned)(hpos * 3 + hslot) * 32768u + wo;
#pragma unroll
      for (int mi = 0; mi < 2; ++mi)
#pragma unroll
        for (int s = 0; s < 2; ++s) {
          const u32x4 t = __builtin_amdgcn_raw_buffer_load_b128(rme, lane * 16, sh + mi * 4096 + s * 1024, AUX_SYS);
          fin[mi][0][2 * s] += bf_lo(t[0]); fin[mi][1][2 * s] += bf_hi(t[0]);
          fin[mi][2][2 * s] += bf_lo(t[1]); fin[mi][3][2 * s] += bf_hi(t[1]);
          fin[mi][0][2 * s + 1] += bf_lo(t[2]); fin[mi][1][2 * s + 1] += bf_hi(t[2]);
          fin[mi][2][2 * s + 1] += bf_lo(t[3]); fin[mi][3][2 * s + 1] += bf_hi(t[3]);
        }
    }
    const int blk = run == 0 ? (rank >> 1) ^ quad : rank >> 1;
    const int tdst = run == 0 ? rank : hand_over ? rank ^ 1 : owner;
    const int pos = run == 2 && hand_over ? hpos : (((blk >> 1) * 2 + (trb & 1)) * 2 + (blk & 1)) * 16 + tcb;
    const int slot = run == 0 ? 2 : hand_over ? hslot : rh;
    const int trel = (tdst - pair0) & 7;
    if (tdst != rank) {
      if (wv == 0) {
        unsigned tk = 0;
        unsigned* lt = (unsigned*)(me + OFF_LT + trel * 128);
        if (PRIO && trel < 2) {
          if (hand_over) { if (lane == 0) __hip_atomic_fetch_add(lt + 1, 1u, __ATOMIC_RELAXED, __HIP_MEMORY_SCOPE_AGENT); }
          else if (cb >= 4) wait_words(lt + 1, lane == 0, call * (3u * EC), tmo, call, lane);
        }
        if (lane == 0) tk = __hip_atomic_fetch_add(lt, 1u, __ATOMIC_RELAXED, __HIP_MEMORY_SCOPE_AGENT);
        tk = UNI(tk);
        const unsigned c0 = (call - 1u) * (40u - EC / 2);
        wait_words((const unsigned*)(me + OFF_LC + trel * 128), lane == 0,
                   trel >= 2 ? max(call * 8u + tk + 1u - WB, c0 + 1u) : tk + 1u - WP, tmo, call, lane);
      }
      BAR;
    }
    if (run == 2) STAMP_AT(4);
    const auto rp = rsrc_of(uniform_ptr(P.ws[tdst & 7]), WS_NEED);
    const unsigned sp = OFF_PS + (unsigned)(pos * 3 + slot) * 32768u + wo;
#pragma unroll
    for (int mi = 0; mi < 2; ++mi)
#pragma unroll
      for (int s = 0; s < 2; ++s) {
        const u32x4 o = {pack2(fin[mi][0][2 * s], fin[mi][1][2 * s]), pack2(fin[mi][2][2 * s], fin[mi][3][2 * s]),
                         pack2(fin[mi][0][2 * s + 1], fin[mi][1][2 * s + 1]),
                         pack2(fin[mi][2][2 * s + 1], fin[mi][3][2 * s + 1])};
        if (mi * 2 + s == SIG && tid == 0) __hip_atomic_fetch_add((unsigned*)(me + OFF_LC + trel * 128), 1u, __ATOMIC_RELAXED, __HIP_MEMORY_SCOPE_AGENT);
        st128(o, rp, lane * 16, sp + mi * 4096 + s * 1024);
        if (mi + s < 2 && tdst != rank) { if (cb >= 12) WAIT_V(PL); else WAIT_V(PP); }
      }
    WAIT_V(0);
    BAR;
    if (tid == 0) __builtin_amdgcn_raw_buffer_store_b32(call, rp, 0, OFF_PF + pos * 128 + slot * 4, AUX_SYS);
    seg = 0;
    if (++job == 2) { STAMP_AT(12); orel = 2 + p0 + ((orel - 2) >> 1 & 1); cb = 12 + (xcd >> 1); }
  }
  STAMP_AT(5);

  float y[16];
  {
    const int sblk = (rpos >> 6) * 2 + ((rpos >> 4) & 1);
    const bool foreign = sblk != (rank >> 1);
    unsigned mask = foreign ? 7u : 3u;
    if (foreign && !((rpos >> 5) & 1) && (rpos & 15) < EC) mask = 4u | (2u >> giver((rank - 2 * sblk) & 7, rpos & 15));
    if (wv == 0) wait_words((const unsigned*)(me + OFF_PF + rpos * 128) + (lane & 3), lane < 3 && (mask >> lane & 1), call, tmo, call, lane);
    BAR;
    STAMP_AT(6);
    const unsigned so = OFF_PS + rpos * 3 * 32768 + rhalf * 16384 + wv * 2048;
    u32x4 p[3][2];
#pragma unroll
    for (int t = 0; t < 3; ++t) {
      const unsigned vo = (mask >> t & 1) ? lane * 16 : 0x7ffffff0u;
      p[t][0] = __builtin_amdgcn_raw_buffer_load_b128(rme, vo, so + t * 32768, AUX_SYS);
      p[t][1] = __builtin_amdgcn_raw_buffer_load_b128(rme, vo, so + t * 32768 + 1024, AUX_SYS);
    }
#pragma unroll
    for (int s = 0; s < 2; ++s)
#pragma unroll
      for (int d = 0; d < 4; ++d) {
        y[8 * s + 2 * d] = bf_lo(p[0][s][d]) + bf_lo(p[1][s][d]) + bf_lo(p[2][s][d]);
        y[8 * s + 2 * d + 1] = bf_hi(p[0][s][d]) + bf_hi(p[1][s][d]) + bf_hi(p[2][s][d]);
      }
    unsigned am = 0;
#pragma unroll
    for (int t = 0; t < 16; ++t) am = max(am, __builtin_bit_cast(unsigned, y[t]) & 0x7fffffffu);
    am = wave_max(am);
    unsigned tk = 0;
    if (lane == 0) {
      __hip_atomic_fetch_max(misc + 20, am, __ATOMIC_RELAXED, __HIP_MEMORY_SCOPE_WORKGROUP);
      tk = __hip_atomic_fetch_add(misc + 19, 1u, __ATOMIC_RELAXED, __HIP_MEMORY_SCOPE_WORKGROUP);
    }
    if (UNI(tk) == 7) {
      const unsigned wmax = __hip_atomic_load(misc + 20, __ATOMIC_RELAXED, __HIP_MEMORY_SCOPE_WORKGROUP);
      if (lane < 8) {
        __hip_atomic_store((gu64*)gaddr, ((u64)call << 32) | wmax, __ATOMIC_RELAXED, __HIP_MEMORY_SCOPE_SYSTEM);
        __hip_atomic_fetch_add((unsigned*)(me + OFF_ACT + lane * 128), 1u, __ATOMIC_RELAXED, __HIP_MEMORY_SCOPE_AGENT);
      }
    }
  }
  if (wv == 0) wait_words((const unsigned*)(me + OFF_ACT + xcd * 128), lane == 0, call * NWG - ATH, tmo, call, lane);
  BAR;
  {
    const u64 t0 = __builtin_amdgcn_s_memrealtime();
    unsigned gm;
    for (;;) {
      asm volatile("" ::: "memory");
      const u32x4 g0 = __builtin_amdgcn_raw_buffer_load_b128(rme, lane * 16, OFF_AG + wv * 2048, AUX_SYS);
      const u32x4 g1 = __builtin_amdgcn_raw_buffer_load_b128(rme, lane * 16, OFF_AG + wv * 2048 + 1024, AUX_SYS);
      gm = max(max(g0[0], g0[2]), max(g1[0], g1[2]));
      if (__all(g0[1] == call && g0[3] == call && g1[1] == call && g1[3] == call)) break;
      if (expired(t0, tmo, call, lane)) break;
      __builtin_amdgcn_s_sleep(1);
    }
    gm = wave_max(gm);
    if (lane == 0) __hip_atomic_fetch_max(misc + 21, gm, __ATOMIC_RELAXED, __HIP_MEMORY_SCOPE_WORKGROUP);
    if (__builtin_amdgcn_s_memrealtime() - tstart > TMO_LOOK &&
        __hip_atomic_load(tmo, __ATOMIC_RELAXED, __HIP_MEMORY_SCOPE_AGENT) == call && lane == 0)
      misc[22] = 1;
  }
  BAR;
  STAMP_AT(7);
  {
    const float amax = __builtin_bit_cast(float, misc[21]);
    const float inv = 448.0f / amax;
    u32x4 o;
#pragma unroll
    for (int d = 0; d < 4; ++d) {
      int r = __builtin_amdgcn_cvt_pk_fp8_f32(y[4 * d] * inv, y[4 * d + 1] * inv, 0, false);
      r = __builtin_amdgcn_cvt_pk_fp8_f32(y[4 * d + 2] * inv, y[4 * d + 3] * inv, r, true);
      o[d] = (unsigned)r;
    }
    if (misc[22]) o = u32x4{0x7f7f7f7fu, 0x7f7f7f7fu, 0x7f7f7f7fu, 0x7f7f7f7fu};
    const int row = 128 * (rpos >> 5) + 16 * (rhalf * 4 + (wv >> 1)) + l15;
    const int col = 128 * (rpos & 31) + 64 * (wv & 1) + 16 * q4;
    st128(o, rsrc_of(P.out, 512u * NN), (unsigned)(row * NN + col), 0);
  }
  STAMP_AT(8);
}

}

extern "C" void kernel_launch(int rank, int nranks, ncclComm_t comm, hipStream_t stream,
                              void* const* d_in, const int* in_sizes, int n_in, void* d_out, int out_size,
                              const int* in_dtypes, int out_dtype, void* const* d_out_all,
                              void* const* d_ws_all, size_t ws_bytes) {
  (void)comm; (void)in_dtypes; (void)out_dtype; (void)d_out_all;
  if (nranks != NR || (unsigned)rank >= (unsigned)NR || n_in < 2 || ws_bytes < WS_NEED ||
      in_sizes[0] != 4096 * KS || in_sizes[1] != KS * NN || out_size != 512 * NN) {
    fprintf(stderr, "unexpected problem shape\n");
    return;
  }
  Params p;
  p.x = (const float*)d_in[0];
  p.w = (const float*)d_in[1];
  p.out = (unsigned char*)d_out;
  for (int r = 0; r < NR; ++r) p.ws[r] = (char*)d_ws_all[r];
  p.rank = rank;
  gemm_rs_kernel<<<NWG, NTHR, 0, stream>>>(p);
  HIP_CHECK(hipGetLastError());
}
```

```cpp
#include <hip/hip_runtime.h>
#include <rccl/rccl.h>
#include <cstdio>

#define HIP_CHECK(expr) do { hipError_t _e = (expr); if (_e != hipSuccess) { \
    fprintf(stderr, "HIP error %s at %s:%d\n", hipGetErrorString(_e), __FILE__, __LINE__); \
    return; } } while (0)

namespace {

constexpr int NR = 8;
constexpr int NN = 4096;
constexpr int KS = 512;
constexpr int NWG = 256;
constexpr int NTHR = 512;

#define EC 6
#define LDELAY 150
#define DRAW_U 3
#define FPOS 16
#define SLPE 2
#define MINA 2
#define OWNLAST 1
#define LASTW 0
#define PRO 2
#define E3 0
#define G8 1
#define LSC "sc0 sc1"
#define FAKELD 0
#define NOGEMM 0
#define DW 1
#define PW0 4
#define BDELAY 0
#define ODELAY 0
#define PTH 16
#define PWV 0
#define OWNPOS 56
#define LDELAY2 LDELAY
#define LOOK 1
#define FADAPT 1
#define FP0 0
#define FP1 8
#define ATH 8
#define SLP 2

constexpr unsigned OFF_CALL = 0;
constexpr unsigned OFF_TMO = 32768;
constexpr unsigned OFF_LU = OFF_TMO + 128;
constexpr unsigned OFF_ACT = OFF_TMO + 1024;
constexpr unsigned OFF_XF = OFF_TMO + 4096;
constexpr unsigned OFF_WF = OFF_XF + 2048;
constexpr unsigned OFF_AG = OFF_TMO + 8192;
constexpr unsigned OFF_DBG = 65536;
constexpr unsigned OFF_AF = OFF_TMO + 2048;
constexpr unsigned OFF_TF = 131072;
constexpr unsigned OFF_XS = 1u << 20;
constexpr unsigned OFF_WS = OFF_XS + (12u << 20);
constexpr unsigned OFF_TS = OFF_WS + (12u << 20);
constexpr int T_HAND = 384;
constexpr int T_PARK = T_HAND + 48;
constexpr unsigned WS_NEED = OFF_TS + (T_PARK + 256) * 32768u;

constexpr unsigned long long DEADLINE = 100000000ull;
constexpr unsigned long long TMO_LOOK = 20000;
constexpr int AUX_SYS = 17;

constexpr int STAGE_B = 32768;
constexpr int LDS_MISC = 4 * STAGE_B;
constexpr int LDS_B = LDS_MISC + 2048;

typedef __bf16 bf16x8 __attribute__((ext_vector_type(8)));
typedef __bf16 bf16x2 __attribute__((ext_vector_type(2)));
typedef float f32x2 __attribute__((ext_vector_type(2)));
typedef float f32x4 __attribute__((ext_vector_type(4)));
typedef unsigned u32x4 __attribute__((ext_vector_type(4)));
typedef unsigned long long u64;
typedef __attribute__((address_space(1))) unsigned long long gu64;
typedef __attribute__((address_space(1))) unsigned gu32;

struct Params {
  const float* x;
  const float* w;
  unsigned char* out;
  char* ws[NR];
  int rank;
};

__device__ __forceinline__ unsigned pack2(float lo, float hi) {
  f32x2 v = {lo, hi};
  return __builtin_bit_cast(unsigned, __builtin_convertvector(v, bf16x2));
}
__device__ __forceinline__ float bf_lo(unsigned u) { return __builtin_bit_cast(float, u << 16); }
__device__ __forceinline__ float bf_hi(unsigned u) { return __builtin_bit_cast(float, u & 0xffff0000u); }

__device__ __forceinline__ __amdgpu_buffer_rsrc_t rsrc_of(const void* p, unsigned bytes) {
  return __builtin_amdgcn_make_buffer_rsrc(const_cast<void*>(p), 0, bytes, 0x00020000);
}
__device__ __forceinline__ void st128(u32x4 v, __amdgpu_buffer_rsrc_t r, unsigned vo, unsigned so) {
  __builtin_amdgcn_raw_buffer_store_b128(v, r, vo, so, AUX_SYS);
  asm volatile("s_nop 1" :: "v"(v) : "memory");
}
__device__ __forceinline__ unsigned lds_addr(const void* p) { return (unsigned)(size_t)(const __attribute__((address_space(3))) char*)p; }
__device__ __forceinline__ unsigned lds_get(const unsigned* p) {
  unsigned r;
  asm volatile("ds_read_b32 %0, %1\n\ts_waitcnt lgkmcnt(0)" : "=&v"(r) : "v"(lds_addr(p)) : "memory");
  return r;
}
__device__ __forceinline__ unsigned lds_inc(unsigned* p) {
  unsigned r;
  asm volatile("ds_add_rtn_u32 %0, %1, %2\n\ts_waitcnt lgkmcnt(0)" : "=&v"(r) : "v"(lds_addr(p)), "v"(1u) : "memory");
  return r;
}
__device__ __forceinline__ void lds_max(unsigned* p, unsigned v) {
  asm volatile("ds_max_u32 %0, %1\n\ts_waitcnt lgkmcnt(0)" :: "v"(lds_addr(p)), "v"(v) : "memory");
}
__device__ __forceinline__ unsigned wave_max(unsigned v) {
  v = max(v, (unsigned)__builtin_amdgcn_mov_dpp((int)v, 0xB1, 0xf, 0xf, true));
  v = max(v, (unsigned)__builtin_amdgcn_mov_dpp((int)v, 0x4E, 0xf, 0xf, true));
  v = max(v, (unsigned)__builtin_amdgcn_mov_dpp((int)v, 0x141, 0xf, 0xf, true));
  v = max(v, (unsigned)__builtin_amdgcn_mov_dpp((int)v, 0x140, 0xf, 0xf, true));
  const unsigned a = __builtin_amdgcn_readlane(v, 0), b = __builtin_amdgcn_readlane(v, 16);
  const unsigned c = __builtin_amdgcn_readlane(v, 32), d = __builtin_amdgcn_readlane(v, 48);
  return max(max(a, b), max(c, d));
}
__device__ __forceinline__ char* uniform_ptr(char* p) {
  const u64 a = (u64)p;
  const unsigned lo = __builtin_amdgcn_readfirstlane((unsigned)a), hi = __builtin_amdgcn_readfirstlane((unsigned)(a >> 32));
  return (char*)(((u64)hi << 32) | lo);
}
#define INL __attribute__((always_inline))
#define UNI(x) __builtin_amdgcn_readfirstlane(x)
#define WAIT_V_(n) asm volatile("s_waitcnt vmcnt(" #n ")" ::: "memory")
#define WAIT_V(n) WAIT_V_(n)
#define SB __builtin_amdgcn_sched_barrier(0)
#define BAR do { asm volatile("" ::: "memory"); __builtin_amdgcn_s_waitcnt(0xC07F);   \
                 __builtin_amdgcn_s_barrier(); asm volatile("" ::: "memory"); SB; } while (0)
#define ACQ __builtin_amdgcn_fence(__ATOMIC_ACQUIRE, "wavefront")

#define STAMP_AT(k) do { } while (0)

__device__ __forceinline__ bool expired(u64 t0, unsigned* tmo, unsigned call, int lane) {
  const u64 dt = __builtin_amdgcn_s_memrealtime() - t0;
  if (dt > TMO_LOOK) {
    if (UNI(__hip_atomic_load(tmo, __ATOMIC_RELAXED, __HIP_MEMORY_SCOPE_AGENT)) == call) return true;
    if (dt > DEADLINE) {
      if (lane == 0) __hip_atomic_store(tmo, call, __ATOMIC_RELAXED, __HIP_MEMORY_SCOPE_AGENT);
      return true;
    }
  }
  return false;
}
__device__ __forceinline__ void wait_words(const unsigned* p, bool need, unsigned want, unsigned* tmo,
                                           unsigned call, int lane) {
  const u64 t0 = __builtin_amdgcn_s_memrealtime();
  for (;;) {
    unsigned v = want;
    if (need) v = __hip_atomic_load(p, __ATOMIC_RELAXED, __HIP_MEMORY_SCOPE_SYSTEM);
    if (__all((int)(v - want) >= 0)) break;
    if (expired(t0, tmo, call, lane)) break;
    __builtin_amdgcn_s_sleep(SLP);
  }
}

__global__ void __launch_bounds__(NTHR, 2) gemm_rs_kernel(Params P) {
  __shared__ __attribute__((aligned(16))) char lds[LDS_B];
  const int tid = threadIdx.x;
  const int lane = tid & 63;
  const int wv = UNI(tid >> 6);
  const int l15 = lane & 15, q4 = lane >> 4;
  const int rank = P.rank;
  const int rh = rank & 1, pair0 = rank & 6;
  char* const me = P.ws[rank];
  unsigned* const tmo = (unsigned*)(me + OFF_TMO);
  unsigned* const misc = (unsigned*)(lds + LDS_MISC);

  const int b = UNI(blockIdx.x), xcd = b & 7, idx = b >> 3;
  STAMP_AT(0);
  const u64 tstart = __builtin_amdgcn_s_memrealtime();
  const int t = idx - 7;
  int orel, cq;
  if (t < 22) { const int ro = t % 7; orel = ro ? ro + 1 : rh ^ 1; cq = t / 7; }
  else { orel = 2 + 2 * (t - 22) + (~xcd & 1); cq = 3; }
  int cb = 4 * cq + (xcd >> 1);
  int rpos, rhalf;
  {
    const int u = b - 56, far = idx < 6 ? (pair0 + 2 + idx) & 7 : idx == 6 ? rank ^ 1 : rank;
    const bool last = idx < 7 || u >= 192;
    const int blk = last ? far >> 1 : (u >> 2) & 3, ra = last ? xcd & 1 : (u >> 1) & 1, rc = last ? 12 + (xcd >> 1) : u >> 4;
    rpos = (((blk >> 1) * 2 + ra) * 2 + (blk & 1)) * 16 + rc;
    rhalf = (last ? far : u) & 1;
  }
  auto giver = [](int rel, int c) { return ((c >> 2) ^ c ^ (rel >> 1)) & 1; };

  const unsigned lx = (unsigned)((16 * wv + (lane >> 3)) * KS + 4 * ((lane ^ (lane >> 3)) & 7)) * 4u;
  const unsigned lw = (unsigned)((64 * (wv >> 2) + 16 * ((wv >> 1) & 1) + 4 * q4) * NN + 64 * (wv & 1) + 4 * l15) * 4u;
  char* const px = uniform_ptr((char*)P.x);
  char* const pw = uniform_ptr((char*)P.w);
  struct Src { __amdgpu_buffer_rsrc_t rs; unsigned so, s0, s1, s2, vo; };
  auto src_of = [&](int q, int dst, int hh) INL {
    const int blk = (q & 31) >> 1, kb = 256 * hh + 128 * (q & 1);
    const bool isx = !(q & 32);
    Src r;
    r.rs = rsrc_of(uniform_ptr(pw + (isx ? px - pw : 0)), 4096u * KS * 4u);
    r.so = UNI(isx ? (unsigned)((512 * (blk >> 1) + 256 * (dst >> 2) + 128 * (blk & 1)) * KS + kb) * 4u
                   : (unsigned)(kb * NN + 2048 * ((dst >> 1) & 1) + 128 * blk) * 4u);
    r.s0 = UNI(isx ? 128u : NN * 4u);
    r.s1 = UNI(isx ? 8u * KS * 4u : 2u * NN * 4u);
    r.s2 = UNI(isx ? 256u : 32u * NN * 4u);
    r.vo = lw + (isx ? lx - lw : 0u);
    return r;
  };
  auto load1 = [&](const Src& r, int t) INL {
    return __builtin_bit_cast(f32x4, __builtin_amdgcn_raw_buffer_load_b128(r.rs, r.vo, r.so + (t & 1) * r.s0 + ((t >> 1) & 1) * r.s1 + (t >> 2) * r.s2, 0));
  };
  auto load_item = [&](f32x4* v, int q, int dst, int hh) INL {
    const Src r = src_of(q, dst, hh);
#pragma unroll
    for (int t = 0; t < 8; ++t) v[t] = load1(r, t);
  };
  auto convert_item = [&](u32x4* f, const f32x4* v, int q) INL {
    if (q < 32) {
#pragma unroll
      for (int u = 0; u < 4; ++u)
        f[u] = u32x4{pack2(v[2 * u][0], v[2 * u][1]), pack2(v[2 * u][2], v[2 * u][3]),
                     pack2(v[2 * u + 1][0], v[2 * u + 1][1]), pack2(v[2 * u + 1][2], v[2 * u + 1][3])};
    } else {
#pragma unroll
      for (int u = 0; u < 4; ++u)
        f[u] = u32x4{pack2(v[0][u], v[1][u]), pack2(v[2][u], v[3][u]), pack2(v[4][u], v[5][u]), pack2(v[6][u], v[7][u])};
    }
#pragma unroll
    for (int u = 0; u < 4; ++u) asm volatile("" : "+v"(f[u]));
  };
  auto item_base = [&](int q, int chunk) -> unsigned {
    const unsigned pb = (q < 32 ? OFF_XS : OFF_WS) + (unsigned)(chunk * 16 + ((q & 31) >> 1)) * 65536u + (q & 1) * 32768u;
    return pb + (q < 32 ? wv * 2048u : wv * 4096u);
  };
  auto frag_off = [](int q, int u) INL { return (u & 1) * 1024u + (u >> 1) * (q < 32 ? 16384u : 2048u); };
  auto item_flag = [&](int q, int chunk) -> unsigned {
    return (q < 32 ? OFF_XF : OFF_WF) + ((q & 31) >> 1) * 128u + chunk * 8u + (q & 1) * 4u;
  };
  auto draw_call = [&]() {
    if (tid != 16) misc[tid] = 0;
    if (tid == NTHR - 64)
      misc[16] = __hip_atomic_fetch_add((unsigned*)(me + OFF_CALL + b * 128), 1u, __ATOMIC_RELAXED,
                                        __HIP_MEMORY_SCOPE_AGENT) + 1u;
  };
  const auto rme = rsrc_of(uniform_ptr(me), WS_NEED);
  unsigned call;
  if (idx < 7) {
    const bool busy = idx < 6;
    const int lrel = busy ? 2 + idx : rh ^ 1, far = (pair0 + lrel) & 7;
    const int ni = busy ? 60 : 0, nu = busy ? 84 - EC / 2 : 24 + 3 * EC;
    const int nj = 2 * nu + 8;
    int qtab, ttab;
    {
      int k = lane < OWNPOS ? lane : lane < OWNPOS + 4 ? lane - OWNPOS + 56 : lane - 4;
      const int dst = k < 8 ? far : rank, src = k < 8 ? rank : far;
      const int j = k < 32 ? (k >> 3) * 2 + ((k >> 1) & 1) : k < 40 ? 8 + (k & 3) : (k >> 3) * 4 - 12 + (k & 3);
      const int kh = (k >> 2) & 1;
      const bool isw = k < 32 ? k & 1 : k >= 40;
      const int second_blk = j >= 7, jt = second_blk ? 11 - j : j - 2;
      const int en = jt + (jt + 2 >= ((src - (dst & 6)) & 7));
      const int row = j < 2 ? (dst ^ 1) * 2 + j : (((dst & 6) + 2 + en) & 7) * 2 + second_blk;
      qtab = k >= 56 ? dst * 4 + (k & 3) : isw ? 32 + j * 2 + kh : row * 2 + kh;
      k = lane;
      const int mrel = (rank - (far & 6)) & 7;
      int to = mrel, ta, tc, sum = 0;
      if (busy) {
        int i = k - (8 - EC);
        const int fpos = FADAPT ? (mrel < 4 ? FP0 : mrel < 6 ? FP1 : 16) : FPOS;
        const bool one = i >= fpos && i < fpos + EC / 2;
        if (i >= fpos + EC / 2) i -= EC / 2;
        if (i < 0) { ta = 0; tc = 8 + i; }
        else if (one) { ta = 0; tc = 2 * (i - fpos); tc += giver(to, tc) == (far & 1); }
        else if (i < 8) { ta = 1; tc = i; }
        else { ta = i & 1; tc = (4 + (i >> 1)) & 15; }
      } else if (k >= 8 && k < 8 + 3 * EC) {
        if (FADAPT) { to = 2 + (k - 8) / (EC + !EC) * 2 + (k & 1); tc = (k - 8) % (EC + !EC) & ~1; }
        else { to = 2 + (k - 8) % 6; tc = 2 * ((k - 8) / 6); }
        ta = 0; tc += giver(to, tc) == rh; sum = to;
      } else {
        const int i = k < 8 ? k : k - 3 * EC;
        ta = i & 1; tc = (i >> 1) & 15;
      }
      const int there = T_PARK + to * 32 + ta * 16 + tc;
      ttab = there | (sum ? there : ((((far >> 2) * 2 + ta) * 2 + ((far >> 1) & 1)) * 16 + tc) * 3 + (far & 1)) << 10 | sum << 20;
    }
    char* const pfar = uniform_ptr(P.ws[far]);
    unsigned* const jobs = misc + 128;
    if (busy) {
      const auto rd = rsrc_of(pfar, WS_NEED);
      const int q = __builtin_amdgcn_readlane(qtab, xcd);
      f32x4 v[8];
      u32x4 f[4];
      load_item(v, q, far, far & 1);
      draw_call();
      convert_item(f, v, q);
      const unsigned sb = item_base(q, rank);
#pragma unroll
      for (int u = 0; u < 4; ++u) {
        st128(f[u], rd, lane * 16, sb + frag_off(q, u));
        if (u == 0) BAR;
        if (u == 3 || (wv >= PW0 && wv < 7)) WAIT_V(0); else if (wv < PW0) WAIT_V(1);
      }
      call = UNI(misc[16]);
      unsigned tk = 0;
      if (lane == 0) tk = __hip_atomic_fetch_add(misc + 23, 1u, __ATOMIC_RELAXED, __HIP_MEMORY_SCOPE_WORKGROUP);
      if (UNI(tk) == 7 && lane == 0) __builtin_amdgcn_raw_buffer_store_b32(2u * call, rd, item_flag(q, rank), 0, AUX_SYS);
    } else {
      draw_call();
      BAR;
      call = UNI(misc[16]);
    }
    if (wv < 4) {
      auto job_of = [&](int n) INL { return (int)UNI(lds_get(jobs + n)) - 1; };
      char* const ring = lds + wv * 8192;
      unsigned so = 0, dn = 0, dc = 0;
      int sn = 0, sc = 0;
      unsigned an = 1, ac = 1, ap = 1;
      u64 fn = 0, fc = 0, fp = 0;
      auto decode = [&](int kk) INL {
        const int k = kk < 2 * nu ? kk >> 1 : kk - nu;
        const unsigned wo = ((kk < 2 * nu ? kk : ~far) & 1) * 16384u + wv * 4096u;
        an = kk < 2 * nu ? 1 : 2;
        const int q = __builtin_amdgcn_readlane(qtab, min(k, 59));
        const unsigned tn = __builtin_amdgcn_readlane(ttab, max(k - ni, 0));
        if (k < ni) {
          const unsigned o = (q < 32 ? OFF_XS : OFF_WS) + (q & 31) * 32768u + wo;
          so = o + (unsigned)(8 + (q < 32 ? rank >> 2 : (rank >> 1) & 1) * 2 + rh) * (1u << 20);
          dn = o + (unsigned)far * (1u << 20);
          sn = 0;
          fn = (u64)me + item_flag(q, far);
        } else {
          so = OFF_TS + (tn & 1023u) * 32768u + wo;
          dn = OFF_TS + ((tn >> 10) & 1023u) * 32768u + wo;
          sn = tn >> 20;
          fn = sn ? (u64)P.ws[(pair0 + sn) & 7] + OFF_TF + 4 + (((rank >> 2) * 4 + ((rank >> 1) & 1)) * 16 + (tn & 15u)) * 96u + rh * 32u
                  : (u64)me + OFF_TF + ((tn >> 10) & 1023u) * 32u;
        }
      };
      auto read = [&](int u) INL {
        __builtin_amdgcn_global_load_lds((const unsigned*)(pfar + so + u * 1024 + lane * 16), (unsigned*)(ring + u * 1024), 16, 0, AUX_SYS);
        if (!busy) __builtin_amdgcn_global_load_lds((const unsigned*)(me + so + u * 1024 + lane * 16), (unsigned*)(ring + 4096 + u * 1024), 16, 0, AUX_SYS);
      };
      auto write = [&](int u) INL {
        u32x4 x = *(const u32x4*)(ring + u * 1024 + lane * 16);
        if (sc) {
          const u32x4 y = *(const u32x4*)(ring + 4096 + u * 1024 + lane * 16);
#pragma unroll
          for (int d = 0; d < 4; ++d) x[d] = pack2(bf_lo(x[d]) + bf_lo(y[d]), bf_hi(x[d]) + bf_hi(y[d]));
        }
        st128(x, rme, lane * 16, dc + u * 1024);
      };
      auto publish = [&](int n) INL {
        if (fp) {
          unsigned tk = 0;
          if (lane == 0) tk = lds_inc(misc + 320 + n);
          if (UNI(tk) == 3 && lane == 0) __hip_atomic_fetch_add((gu32*)fp, ap, __ATOMIC_RELAXED, __HIP_MEMORY_SCOPE_SYSTEM);
        }
        fp = 0;
      };
#define LANDED(nb, np) do { if (busy) WAIT_V(nb); else WAIT_V(np); } while (0)
      bool flow = false;
#pragma unroll 1
      for (int n = 0;; ++n) {
        if (!flow) {
          int kn;
          while ((kn = job_of(n)) < 0) __builtin_amdgcn_s_sleep(1);
          if (kn >= nj) { lds_max(misc + 26, 1u); break; }
          decode(kn);
          read(0);
          read(1);
        }
        if (LOOK != 1 && lane == 0) lds_max(misc + 17, (unsigned)n + (LOOK ? 2u : 1u));
        dc = dn; sc = sn; fc = fn; ac = an;
        LANDED(1, 2);
        write(0); read(2);
        publish(n - 1);
        LANDED(2, 3);
        write(1); read(3);
        const int kn = job_of(n + 1);
        flow = kn >= 0 && kn < nj;
        if (flow) decode(kn);
        if (LOOK == 1 && lane == 0) lds_max(misc + 17, (unsigned)n + 2u);
        LANDED(2, 3);
        write(2);
        if (flow) read(0);
        if (flow) LANDED(2, 3); else WAIT_V(1);
        if (kn >= nj) lds_max(misc + 26, 1u);
        write(3);
        if (flow) read(1);
        fp = fc; ap = ac;
        if (!flow) { WAIT_V(0); publish(n); }
      }
    } else if (wv == 7) {
#pragma unroll 1
      for (int n = 0;; ++n) {
        unsigned kk = (busy ? 16 : 0) + n * 8 + xcd;
        if (n >= 2) {
          while ((int)UNI(__hip_atomic_load(misc + 17, __ATOMIC_RELAXED, __HIP_MEMORY_SCOPE_WORKGROUP)) < n) __builtin_amdgcn_s_sleep(1);
          if (lane == 0) kk = __hip_atomic_fetch_add((unsigned*)(me + OFF_LU + idx * 128), 1u, __ATOMIC_RELAXED, __HIP_MEMORY_SCOPE_AGENT);
          kk = UNI(kk) + (busy ? 32u : 16u) - (call - 1u) * (nj - (busy ? 24 : 8));
        }
        if ((int)kk < nj) {
          const int k = (int)kk < 2 * nu ? kk >> 1 : kk - nu;
          const unsigned tn = __builtin_amdgcn_readlane(ttab, max(k - ni, 0)), sum = k < ni ? 0u : tn >> 20;
          const unsigned fo = k < ni ? OFF_AF + (far * 64 + __builtin_amdgcn_readlane(qtab, min(k, 59))) * 4u
                            : !sum ? OFF_TF + 4 + ((tn >> 10) & 1023u) * 32u
                            : lane ? OFF_TF + (tn & 1023u) * 32u : OFF_TF + 4 + (T_HAND + (sum - 2) * 8 + (tn & 15u)) * 32u;
          wait_words((const unsigned*)(me + fo), lane < (sum ? 2 : 1), 2u * call, tmo, call, lane);
        }
        if (lane == 0) __hip_atomic_store(jobs + n, kk + 1u, __ATOMIC_RELAXED, __HIP_MEMORY_SCOPE_WORKGROUP);
        if ((int)kk >= nj) break;
      }
    }
  } else if (t < 16) {
    f32x4 va[8], vb[8];
    u32x4 fa[4], fb[4];
    const int e = t * 8 + xcd, q = e >> 1, hf = e & 1;
    const int fd = q < 32 ? hf * 4 + (lane & 3) : hf * 2 + (lane & 2) * 2 + (lane & 1);
    u64 fa4 = (u64)P.ws[fd] + OFF_AF + (rank * 64 + q) * 4;
    draw_call();
    const bool soon = q < 32 ? !(q & 2) : q < 44;
    while (__builtin_amdgcn_s_memrealtime() - tstart < (soon ? LDELAY : LDELAY2)) __builtin_amdgcn_s_sleep(2);
    load_item(va, q, hf * 6, 0);
    load_item(vb, q, hf * 6, 1);
    convert_item(fa, va, q);
    convert_item(fb, vb, q);
    const unsigned sb = item_base(q, 8 + hf * 2);
#pragma unroll
    for (int u = 0; u < 4; ++u) {
      st128(fa[u], rme, lane * 16, sb + frag_off(q, u));
      st128(fb[u], rme, lane * 16, sb + (1u << 20) + frag_off(q, u));
    }
    WAIT_V(0);
    BAR;
    call = UNI(misc[16]);
    if (tid < 4) __hip_atomic_store((gu32*)fa4, 2u * call, __ATOMIC_RELAXED, __HIP_MEMORY_SCOPE_SYSTEM);
    if (tid < 2) __builtin_amdgcn_raw_buffer_store_b32(2u * call, rme, tid * 8, item_flag(q, 8 + hf * 2), AUX_SYS);
  } else {
    draw_call();
    BAR;
    call = UNI(misc[16]);
  }
  STAMP_AT(1);

  f32x4 acc[4][4];
  const int ks = wv >> 2, wm = (wv >> 1) & 1, wn = wv & 1;
  const bool has_brick = t >= 14 || t == 12 + (~xcd & 1);
  const int e = (t >= 14 ? t - 14 : 11) * 8 + xcd;
  const bool second = t == 0 || (t < 14 && orel >= 2 && !((xcd ^ orel) & 1));
  const char* lbw = lds + 16384 + ks * 8192 + wn * 4096 + lane * 16;
  const char* lbx = lds + (64 * wm + l15) * 128 + (((4 * ks + q4) ^ (lane & 7)) * 16);
  const int job1 = idx < 7 ? 0 : second ? 3 : 2;
  if (t >= 0) {
    const unsigned hold = has_brick ? BDELAY : !second && t != 7 ? ODELAY : 0;
    while (__builtin_amdgcn_s_memrealtime() - tstart < hold) __builtin_amdgcn_s_sleep(8);
  }
#pragma unroll 1
  for (int job = has_brick ? 0 : 1; job < job1;) {
    const bool brick = job == 0;
    const int owner = (pair0 + orel) & 7, rb = 2 * owner + ((xcd ^ orel) & 1);
    const int quad = 1 + (e >> 5);
    const int trb = brick ? 2 * rank + (e & 1) : rb, tcb = brick ? (e >> 1) & 15 : cb;
    const int blk = brick ? (rank >> 1) ^ quad : rank >> 1;
    unsigned todo = brick ? 0xfu : orel >= 2 ? 0xffffu & ~(3u << 2 * orel) : 0xffffu;
    if (!brick) STAMP_AT(9);
#pragma unroll
    for (int mi = 0; mi < 4; ++mi)
#pragma unroll
      for (int u = 0; u < 4; ++u) acc[mi][u] = f32x4{0.f, 0.f, 0.f, 0.f};
    const unsigned sw = l15 < 4 ? 16 + (lane < 32 ? blk & 2 : (blk & 1) * 2) * 2 + l15 : (2 * pair0 + lane) & 15;
    const char* fp = me + (lane < 32 ? OFF_XF + trb * 128 : OFF_WF + tcb * 128) + sw * 4;
    const unsigned tab = (sw >> 1) * (16u * 65536u) + (sw & 1u) * 32768u;
    const char* gx = me + OFF_XS + trb * 65536 + wv * 2048 + lane * 16;
    const char* gw = me + OFF_WS + tcb * 65536 + wv * 2048 + lane * 16;
    unsigned ox = 0, ow = 0;
    auto stage1 = [&](int buf, int part) INL {
      char* l = lds + buf * STAGE_B + wv * 2048;
      const char* g = (part & 2) ? gw + ow : gx + ox;
      __builtin_amdgcn_global_load_lds((const unsigned*)(g + (buf & 1) * 16384 + (part & 1) * 1024),
                                       (unsigned*)(l + (part >> 1) * 16384 + (part & 1) * 1024), 16, 0, AUX_SYS);
    };
    bf16x8 xa[4] = {}, wa[4] = {}, xb[4] = {}, wb[4] = {};
    auto step = [&](bf16x8* xc, bf16x8* wc, bf16x8* xn, bf16x8* wx, int buf, bool full, bool mf, bool rd, bool ld) INL {
      const int bo = ((buf + 1) & 3) * STAGE_B;
      if (full) {
#pragma unroll
        for (int mi = 0; mi < 4; ++mi) {
          acc[mi][0] = __builtin_amdgcn_mfma_f32_16x16x32_bf16(wc[0], xc[mi], acc[mi][0], 0, 0, 0);
          wx[mi] = *(const bf16x8*)(lbw + bo + mi * 1024);
          xn[mi] = *(const bf16x8*)(lbx + bo + mi * 2048);
          SB;
#pragma unroll
          for (int u = 1; u < 4; ++u)
            acc[mi][u] = __builtin_amdgcn_mfma_f32_16x16x32_bf16(wc[u], xc[mi], acc[mi][u], 0, 0, 0);
          stage1(buf, mi);
          SB;
        }
      } else {
        if (ld) {
#pragma unroll
          for (int part = 0; part < 4; ++part) stage1(buf, part);
        }
        if (rd) {
#pragma unroll
          for (int mi = 0; mi < 4; ++mi) {
            wx[mi] = *(const bf16x8*)(lbw + bo + mi * 1024);
            xn[mi] = *(const bf16x8*)(lbx + bo + mi * 2048);
          }
        }
        SB;
        if (mf) {
#pragma unroll
          for (int mi = 0; mi < 4; ++mi)
#pragma unroll
            for (int u = 0; u < 4; ++u)
              acc[mi][u] = __builtin_amdgcn_mfma_f32_16x16x32_bf16(wc[u], xc[mi], acc[mi][u], 0, 0, 0);
        }
      }
    };
    __builtin_amdgcn_s_waitcnt(0x0F70);
    if (wv == PWV) __builtin_amdgcn_global_load_lds((const unsigned*)fp, misc + 384 + 64, 4, 0, AUX_SYS);
    const u64 t0 = __builtin_amdgcn_s_memrealtime();
    unsigned ready = 0;
    bool h0 = false, h1 = false;
    int pb = 0;
    auto round = [&](bool full) INL {
      const unsigned avail = ready & todo;
      const bool h2 = full || avail != 0;
      if (h2) {
        const int sl = __builtin_ctz(avail);
        todo &= ~(1u << sl);
        ox = __builtin_amdgcn_readlane(tab, sl);
        ow = __builtin_amdgcn_readlane(tab, sl + 32);
      } else if (!h0 && !h1 && wv == PWV) {
        if (expired(t0, tmo, call, lane) && lane == 0) misc[24] = 0xffffu;
        __builtin_amdgcn_s_sleep(SLP);
      }
      if (full || h1) WAIT_V(8); else WAIT_V(0);
      BAR;
      const bool poll = wv == PWV && (!full || (__popc(ready & todo) <= PTH && (todo & ~ready)));
      if (poll) __builtin_amdgcn_global_load_lds((const unsigned*)fp, misc + 384 + pb * 32, 4, 0, AUX_SYS);
      const unsigned fl = misc[384 + (pb ^ 2) * 32 + lane], force = full ? 0u : misc[24];
      step(xa, wa, xb, wb, pb, full, h0, h0, h2);
      if (!h2) WAIT_V(4); else if (poll) WAIT_V(9); else WAIT_V(8);
      BAR;
      const u64 ok = __ballot((int)(fl - 2u * call) >= 0);
      ready |= ((unsigned)ok & (unsigned)(ok >> 32)) | UNI(force);
      step(xb, wb, xa, wa, pb + 1, full, h0, h1, h2);
      h0 = h1; h1 = h2; pb ^= 2;
    };
#pragma unroll 1
    do {
      if (h0 && h1) {
#pragma unroll 1
        while (ready & todo) round(true);
      }
      round(false);
    } while (todo | h0 | h1);

    f32x4 fin[2][4];
    {
      char* xb0 = lds + lane * 16;
      char* lo = xb0 + wv * 8192;
      const char* li = xb0 + (wv ^ 4) * 8192;
#pragma unroll
      for (int mi = 0; mi < 2; ++mi)
#pragma unroll
        for (int u = 0; u < 4; ++u) *(f32x4*)(lo + (mi * 4 + u) * 1024) = ks ? acc[mi][u] : acc[2 + mi][u];
      BAR;
#pragma unroll
      for (int mi = 0; mi < 2; ++mi)
#pragma unroll
        for (int u = 0; u < 4; ++u) fin[mi][u] = (ks ? acc[2 + mi][u] : acc[mi][u]) + *(const f32x4*)(li + (mi * 4 + u) * 1024);
    }
    if (!brick) STAMP_AT(3);
    const int pos = (((blk >> 1) * 2 + (trb & 1)) * 2 + (blk & 1)) * 16 + tcb;
    const int ti = brick ? pos * 3 + 2 : orel == rh ? pos * 3 + rh : T_PARK + orel * 32 + (rb & 1) * 16 + cb;
    const unsigned sp = OFF_TS + (unsigned)ti * 32768u + (unsigned)(((4 * wm + 2 * ks) * 2 + wn) * 2048);
#pragma unroll
    for (int mi = 0; mi < 2; ++mi)
#pragma unroll
      for (int s = 0; s < 2; ++s) {
        const u32x4 o = {pack2(fin[mi][0][2 * s], fin[mi][1][2 * s]), pack2(fin[mi][2][2 * s], fin[mi][3][2 * s]),
                         pack2(fin[mi][0][2 * s + 1], fin[mi][1][2 * s + 1]),
                         pack2(fin[mi][2][2 * s + 1], fin[mi][3][2 * s + 1])};
        st128(o, rme, lane * 16, sp + mi * 4096 + s * 1024);
      }
    WAIT_V(0);
    BAR;
    {
      const bool early = !brick && orel >= 2 && !(rb & 1) && cb < EC, give = early && giver(orel, cb) == rh;
      const bool here = brick || orel == rh || (early && !give);
      const auto rf = rsrc_of(uniform_ptr(P.ws[here ? rank : give ? rank ^ 1 : owner]), WS_NEED);
      const unsigned fo = here ? ti * 32 : 4 + (give ? T_HAND + (orel - 2) * 8 + cb : pos * 3 + rh) * 32;
      if (tid == 0) __builtin_amdgcn_raw_buffer_store_b32(2u * call, rf, fo, OFF_TF, AUX_SYS);
    }
    if (!brick) STAMP_AT(4);
    if (++job == 2) {
      STAMP_AT(12);
      if (t >= 1 && t < 7) cb = 12 + (xcd >> 1);
      else { cb = 4 * (t == 0 ? 3 : (orel - 2) >> 1) + (xcd >> 1); orel = rh; }
    }
  }
  STAMP_AT(5);

  float y[16];
  {
    const int sblk = (rpos >> 6) * 2 + ((rpos >> 4) & 1);
    const bool foreign = sblk != (rank >> 1);
    unsigned mask = foreign ? 7u : 3u;
    if (foreign && !((rpos >> 5) & 1) && (rpos & 15) < EC) mask = 4u | (2u >> giver((rank - 2 * sblk) & 7, rpos & 15));
    const unsigned* tf = (const unsigned*)(me + OFF_TF + rpos * 96 + (lane & 3) * 32);
    if (wv == 6 && lane < 8) *(u64*)(misc + 32 + 2 * lane) = (u64)(P.ws[lane] + OFF_AG) + (rank * 64 + min(b, 56)) * 8;
    u32x4 p[3][2] = {};
    if (idx < 7) {
      mask &= ~(1u << rhalf);
      if (wv == 6) {
        wait_words(tf + 1, lane == rhalf, 2u * call, tmo, call, lane);
        lds_max(misc + 25, 1u);
        do {
          unsigned v = 2u * call;
          if (lane < 3 && (mask >> lane & 1)) v = __hip_atomic_load(tf, __ATOMIC_RELAXED, __HIP_MEMORY_SCOPE_SYSTEM);
          if (__all((int)(v - 2u * call) >= 0)) { lds_max(misc + 27, 1u); break; }
        } while (!UNI(lds_get(misc + 26)));
      }
      while (!(UNI(lds_get(misc + 25)) && UNI(lds_get(misc + 26)))) __builtin_amdgcn_s_sleep(1);
      const unsigned sr = OFF_TS + (T_PARK + ((rank - 2 * sblk) & 7) * 32 + ((rpos >> 5) & 1) * 16 + (rpos & 15)) * 32768u + rhalf * 16384 + wv * 2048;
      const auto rr = rsrc_of(uniform_ptr(P.ws[2 * sblk + rhalf]), WS_NEED);
      p[0][0] = __builtin_amdgcn_raw_buffer_load_b128(rr, lane * 16, sr, AUX_SYS);
      if (DW == 1) WAIT_V(0);
      p[0][1] = __builtin_amdgcn_raw_buffer_load_b128(rr, lane * 16, sr + 1024, AUX_SYS);
    }
    if (wv == 6 && !UNI(lds_get(misc + 27))) {
      wait_words(tf, lane < 3 && (mask >> lane & 1), 2u * call, tmo, call, lane);
      lds_max(misc + 27, 1u);
    }
    if (idx < 7) {
      while (!UNI(lds_get(misc + 27))) __builtin_amdgcn_s_sleep(1);
    } else {
      BAR;
    }
    STAMP_AT(6);
    const unsigned so = OFF_TS + rpos * 3 * 32768 + rhalf * 16384 + wv * 2048;
    if (idx >= 7) {
      const unsigned vo = (mask & 1) ? lane * 16 : 0x7ffffff0u;
      p[0][0] = __builtin_amdgcn_raw_buffer_load_b128(rme, vo, so, AUX_SYS);
      p[0][1] = __builtin_amdgcn_raw_buffer_load_b128(rme, vo, so + 1024, AUX_SYS);
    }
#pragma unroll
    for (int t = 1; t < 3; ++t) {
      const int sl = t == 1 && idx < 7 ? rhalf ^ 1 : t;
      const unsigned vo = (mask >> sl & 1) ? lane * 16 : 0x7ffffff0u;
      p[t][0] = __builtin_amdgcn_raw_buffer_load_b128(rme, vo, so + sl * 32768, AUX_SYS);
      p[t][1] = __builtin_amdgcn_raw_buffer_load_b128(rme, vo, so + sl * 32768 + 1024, AUX_SYS);
    }
    STAMP_AT(10);
#pragma unroll
    for (int s = 0; s < 2; ++s)
#pragma unroll
      for (int d = 0; d < 4; ++d) {
        y[8 * s + 2 * d] = bf_lo(p[0][s][d]) + bf_lo(p[1][s][d]) + bf_lo(p[2][s][d]);
        y[8 * s + 2 * d + 1] = bf_hi(p[0][s][d]) + bf_hi(p[1][s][d]) + bf_hi(p[2][s][d]);
      }
    unsigned am = 0;
#pragma unroll
    for (int t = 0; t < 16; ++t) am = max(am, __builtin_bit_cast(unsigned, y[t]) & 0x7fffffffu);
    am = wave_max(am);
    unsigned tk = 0;
    if (lane == 0) {
      __hip_atomic_fetch_max(misc + 20, am, __ATOMIC_RELAXED, __HIP_MEMORY_SCOPE_WORKGROUP);
      tk = __hip_atomic_fetch_add(misc + 19, 1u, __ATOMIC_RELAXED, __HIP_MEMORY_SCOPE_WORKGROUP);
    }
    if (UNI(tk) == 7) {
      u64 g = ((u64)call << 32) | __hip_atomic_load(misc + 20, __ATOMIC_RELAXED, __HIP_MEMORY_SCOPE_WORKGROUP);
      bool out = idx < 7;
      if (!out) {
        unsigned n = 0;
        if (lane == 0) {
          __hip_atomic_fetch_max((u64*)(me + OFF_ACT), g, __ATOMIC_RELAXED, __HIP_MEMORY_SCOPE_AGENT);
          WAIT_V(0);
          n = __hip_atomic_fetch_add((unsigned*)(me + OFF_ACT + 128), 1u, __ATOMIC_RELAXED, __HIP_MEMORY_SCOPE_AGENT);
        }
        out = UNI(n) + 1u == call * (NWG - 56);
        if (out) g = __hip_atomic_load((u64*)(me + OFF_ACT), __ATOMIC_RELAXED, __HIP_MEMORY_SCOPE_AGENT);
      }
      if (out && lane < 8) __hip_atomic_store((gu64*)*(const u64*)(misc + 32 + 2 * lane), g, __ATOMIC_RELAXED, __HIP_MEMORY_SCOPE_SYSTEM);
    }
  }
  if (idx >= 7) {
    if (wv == 0) wait_words((const unsigned*)(me + OFF_AG + (rank * 64 + 56) * 8 + 4), lane == 0, call, tmo, call, lane);
    BAR;
  }
  {
    const u64 t0 = __builtin_amdgcn_s_memrealtime();
    unsigned gm;
    for (;;) {
      const u64 g = __hip_atomic_load((const u64*)(me + OFF_AG) + wv * 64 + min(lane, 56), __ATOMIC_RELAXED, __HIP_MEMORY_SCOPE_SYSTEM);
      gm = (unsigned)g;
      if (__all((unsigned)(g >> 32) == call)) break;
      if (expired(t0, tmo, call, lane)) break;
      __builtin_amdgcn_s_sleep(1);
    }
    gm = wave_max(gm);
    if (lane == 0) __hip_atomic_fetch_max(misc + 21, gm, __ATOMIC_RELAXED, __HIP_MEMORY_SCOPE_WORKGROUP);
    if (__builtin_amdgcn_s_memrealtime() - tstart > TMO_LOOK &&
        __hip_atomic_load(tmo, __ATOMIC_RELAXED, __HIP_MEMORY_SCOPE_AGENT) == call && lane == 0)
      misc[22] = 1;
  }
  BAR;
  STAMP_AT(7);
  {
    const float amax = __builtin_bit_cast(float, misc[21]);
    const float inv = 448.0f / amax;
    u32x4 o;
#pragma unroll
    for (int d = 0; d < 4; ++d) {
      int r = __builtin_amdgcn_cvt_pk_fp8_f32(y[4 * d] * inv, y[4 * d + 1] * inv, 0, false);
      r = __builtin_amdgcn_cvt_pk_fp8_f32(y[4 * d + 2] * inv, y[4 * d + 3] * inv, r, true);
      o[d] = (unsigned)r;
    }
    if (misc[22]) o = u32x4{0x7f7f7f7fu, 0x7f7f7f7fu, 0x7f7f7f7fu, 0x7f7f7f7fu};
    const int row = 128 * (rpos >> 5) + 16 * (rhalf * 4 + (wv >> 1)) + l15;
    const int col = 128 * (rpos & 31) + 64 * (wv & 1) + 16 * q4;
    st128(o, rsrc_of(P.out, 512u * NN), (unsigned)(row * NN + col), 0);
  }
  STAMP_AT(8);
}

}

extern "C" void kernel_launch(int rank, int nranks, ncclComm_t comm, hipStream_t stream,
                              void* const* d_in, const int* in_sizes, int n_in, void* d_out, int out_size,
                              const int* in_dtypes, int out_dtype, void* const* d_out_all,
                              void* const* d_ws_all, size_t ws_bytes) {
  (void)comm; (void)in_dtypes; (void)out_dtype; (void)d_out_all;
  if (nranks != NR || (unsigned)rank >= (unsigned)NR || n_in < 2 || ws_bytes < WS_NEED ||
      in_sizes[0] != 4096 * KS || in_sizes[1] != KS * NN || out_size != 512 * NN) {
    fprintf(stderr, "unexpected problem shape\n");
    return;
  }
  Params p;
  p.x = (const float*)d_in[0];
  p.w = (const float*)d_in[1];
  p.out = (unsigned char*)d_out;
  for (int r = 0; r < NR; ++r) p.ws[r] = (char*)d_ws_all[r];
  p.rank = rank;
  gemm_rs_kernel<<<NWG, NTHR, 0, stream>>>(p);
  HIP_CHECK(hipGetLastError());
}
```

```cpp
#include <hip/hip_runtime.h>
#include <rccl/rccl.h>
#include <cstdio>

#define HIP_CHECK(expr) do { hipError_t _e = (expr); if (_e != hipSuccess) { \
    fprintf(stderr, "HIP error %s at %s:%d\n", hipGetErrorString(_e), __FILE__, __LINE__); \
    return; } } while (0)

namespace {

constexpr int NR = 8;
constexpr int NN = 4096;
constexpr int KS = 512;
constexpr int NWG = 256;
constexpr int NTHR = 512;

#define EC 4
#define LDELAY 80
#define DRAW_U 3
#define FPOS 16
#define SLPE 2
#define MINA 2
#define OWNLAST 1
#define LASTW 0
#define PRO 2
#define E3 0
#define G8 1
#define LSC "sc0 sc1"
#define FAKELD 0
#define NOGEMM 0
#define DW 1
#define PW0 4
#define BDELAY 0
#define ODELAY 0
#define PTH 16
#define PWV 0
#define OWNPOS 56
#define LDELAY2 250
#define LOOK 1
#define FADAPT 1
#define FP0 0
#define FP1 8
#define NP 1
#define LATEF 4
#define MW 0
#define MWN 1
#define SLACK 0
#define ILV 0
#define US (ILV ? 4096 : 1024)
#define NDIR 8
#define WARM 1
#define CODE_LINES 150
#define IAUX 2
#define OAUX 17
#define RAUX 17
#define SAUX 17
#define ATH 8
#define SLP 2

constexpr unsigned OFF_CALL = 0;
constexpr unsigned OFF_TMO = 32768;
constexpr unsigned OFF_LU = OFF_TMO + 128;
constexpr unsigned OFF_ACT = OFF_TMO + 1024;
constexpr unsigned OFF_XF = OFF_TMO + 4096;
constexpr unsigned OFF_WF = OFF_XF + 2048;
constexpr unsigned OFF_AG = OFF_TMO + 8192;
constexpr unsigned OFF_DBG = 65536;
constexpr unsigned OFF_AF = OFF_TMO + 2048;
constexpr unsigned OFF_TF = 131072;
constexpr unsigned OFF_XS = 1u << 20;
constexpr unsigned OFF_WS = OFF_XS + (12u << 20);
constexpr unsigned OFF_TS = OFF_WS + (12u << 20);
constexpr int T_HAND = 384;
constexpr int T_PARK = T_HAND + 48;
constexpr unsigned WS_NEED = OFF_TS + (T_PARK + 256) * 32768u;

constexpr unsigned long long DEADLINE = 100000000ull;
constexpr unsigned long long TMO_LOOK = 20000;
constexpr int AUX_SYS = 17;

constexpr int STAGE_B = 32768;
constexpr int LDS_MISC = 4 * STAGE_B;
constexpr int LDS_B = LDS_MISC + 2048;

typedef __bf16 bf16x8 __attribute__((ext_vector_type(8)));
typedef __bf16 bf16x2 __attribute__((ext_vector_type(2)));
typedef float f32x2 __attribute__((ext_vector_type(2)));
typedef float f32x4 __attribute__((ext_vector_type(4)));
typedef unsigned u32x4 __attribute__((ext_vector_type(4)));
typedef unsigned long long u64;
typedef __attribute__((address_space(1))) unsigned long long gu64;
typedef __attribute__((address_space(1))) unsigned gu32;

struct Params {
  const float* x;
  const float* w;
  char* me;
  long rank;
  unsigned char* out;
  char* ws[NR];
};

__device__ __forceinline__ unsigned pack2(float lo, float hi) {
  f32x2 v = {lo, hi};
  return __builtin_bit_cast(unsigned, __builtin_convertvector(v, bf16x2));
}
__device__ __forceinline__ float bf_lo(unsigned u) { return __builtin_bit_cast(float, u << 16); }
__device__ __forceinline__ float bf_hi(unsigned u) { return __builtin_bit_cast(float, u & 0xffff0000u); }

__device__ __forceinline__ __amdgpu_buffer_rsrc_t rsrc_of(const void* p, unsigned bytes) {
  return __builtin_amdgcn_make_buffer_rsrc(const_cast<void*>(p), 0, bytes, 0x00020000);
}
__device__ __forceinline__ void st128(u32x4 v, __amdgpu_buffer_rsrc_t r, unsigned vo, unsigned so) {
  __builtin_amdgcn_raw_buffer_store_b128(v, r, vo, so, SAUX);
  asm volatile("s_nop 1" :: "v"(v) : "memory");
}
__device__ __forceinline__ unsigned lds_addr(const void* p) { return (unsigned)(size_t)(const __attribute__((address_space(3))) char*)p; }
__device__ __forceinline__ unsigned lds_get(const unsigned* p) {
  unsigned r;
  asm volatile("ds_read_b32 %0, %1\n\ts_waitcnt lgkmcnt(0)" : "=&v"(r) : "v"(lds_addr(p)) : "memory");
  return r;
}
__device__ __forceinline__ unsigned lds_inc(unsigned* p) {
  unsigned r;
  asm volatile("ds_add_rtn_u32 %0, %1, %2\n\ts_waitcnt lgkmcnt(0)" : "=&v"(r) : "v"(lds_addr(p)), "v"(1u) : "memory");
  return r;
}
__device__ __forceinline__ void lds_max(unsigned* p, unsigned v) {
  asm volatile("ds_max_u32 %0, %1\n\ts_waitcnt lgkmcnt(0)" :: "v"(lds_addr(p)), "v"(v) : "memory");
}
__device__ __forceinline__ unsigned wave_max(unsigned v) {
  v = max(v, (unsigned)__builtin_amdgcn_mov_dpp((int)v, 0xB1, 0xf, 0xf, true));
  v = max(v, (unsigned)__builtin_amdgcn_mov_dpp((int)v, 0x4E, 0xf, 0xf, true));
  v = max(v, (unsigned)__builtin_amdgcn_mov_dpp((int)v, 0x141, 0xf, 0xf, true));
  v = max(v, (unsigned)__builtin_amdgcn_mov_dpp((int)v, 0x140, 0xf, 0xf, true));
  const unsigned a = __builtin_amdgcn_readlane(v, 0), b = __builtin_amdgcn_readlane(v, 16);
  const unsigned c = __builtin_amdgcn_readlane(v, 32), d = __builtin_amdgcn_readlane(v, 48);
  return max(max(a, b), max(c, d));
}
__device__ __forceinline__ char* uniform_ptr(char* p) {
  const u64 a = (u64)p;
  const unsigned lo = __builtin_amdgcn_readfirstlane((unsigned)a), hi = __builtin_amdgcn_readfirstlane((unsigned)(a >> 32));
  return (char*)(((u64)hi << 32) | lo);
}
#define INL __attribute__((always_inline))
#define UNI(x) __builtin_amdgcn_readfirstlane(x)
#define WAIT_V_(n) asm volatile("s_waitcnt vmcnt(" #n ")" ::: "memory")
#define WAIT_V(n) WAIT_V_(n)
#define SB __builtin_amdgcn_sched_barrier(0)
#define BAR do { asm volatile("" ::: "memory"); __builtin_amdgcn_s_waitcnt(0xC07F);   \
                 __builtin_amdgcn_s_barrier(); asm volatile("" ::: "memory"); SB; } while (0)
#define ACQ __builtin_amdgcn_fence(__ATOMIC_ACQUIRE, "wavefront")

#define STAMP_AT(k) do { } while (0)

__device__ __forceinline__ bool expired(u64 t0, unsigned* tmo, unsigned call, int lane) {
  const u64 dt = __builtin_amdgcn_s_memrealtime() - t0;
  if (dt > TMO_LOOK) {
    if (UNI(__hip_atomic_load(tmo, __ATOMIC_RELAXED, __HIP_MEMORY_SCOPE_AGENT)) == call) return true;
    if (dt > DEADLINE) {
      if (lane == 0) __hip_atomic_store(tmo, call, __ATOMIC_RELAXED, __HIP_MEMORY_SCOPE_AGENT);
      return true;
    }
  }
  return false;
}
__device__ __forceinline__ void wait_words(const unsigned* p, bool need, unsigned want, unsigned* tmo,
                                           unsigned call, int lane) {
  const u64 t0 = __builtin_amdgcn_s_memrealtime();
  for (;;) {
    unsigned v = want;
    if (need) v = __hip_atomic_load(p, __ATOMIC_RELAXED, __HIP_MEMORY_SCOPE_SYSTEM);
    if (__all((int)(v - want) >= 0)) break;
    if (expired(t0, tmo, call, lane)) break;
    __builtin_amdgcn_s_sleep(SLP);
  }
}

__global__ void __launch_bounds__(NTHR, 2) gemm_rs_kernel(Params P) {
  __shared__ __attribute__((aligned(16))) char lds[LDS_B];
  const int tid = threadIdx.x;
  const int lane = tid & 63;
  const int wv = UNI(tid >> 6);
  const int l15 = lane & 15, q4 = lane >> 4;
  u64 pc0;
  asm volatile("s_getpc_b64 %0" : "=s"(pc0));
  char* const px = (char*)P.x;
  char* const pw = (char*)P.w;
  char* const me = P.me;
  const int rank = (int)P.rank;
  asm volatile("" :: "s"(px), "s"(pw), "s"(me), "s"(rank));
  const int rh = rank & 1, pair0 = rank & 6;
  unsigned* const tmo = (unsigned*)(me + OFF_TMO);
  unsigned* const misc = (unsigned*)(lds + LDS_MISC);

  const int b = UNI(blockIdx.x), xcd = b & 7, idx = b >> 3;
  STAMP_AT(0);
  const u64 tstart = __builtin_amdgcn_s_memrealtime();
  const int t = idx - 7;
  int orel, cq;
  if (t < 22) { const int ro = t % 7; orel = ro ? ro + 1 : rh ^ 1; cq = t / 7; }
  else { orel = 2 + 2 * (t - 22) + (~xcd & 1); cq = 3; }
  int cb = 4 * cq + (xcd >> 1);
  int rpos, rhalf;
  {
    const int u = b - 56, far = idx < 6 ? (pair0 + 2 + idx) & 7 : idx == 6 ? rank ^ 1 : rank;
    const bool last = idx < 7 || u >= 192;
    const int blk = last ? far >> 1 : (u >> 2) & 3, ra = last ? xcd & 1 : (u >> 1) & 1, rc = last ? 12 + (xcd >> 1) : u >> 4;
    rpos = (((blk >> 1) * 2 + ra) * 2 + (blk & 1)) * 16 + rc;
    rhalf = (last ? far : u) & 1;
  }
  auto giver = [](int rel, int c) { return ((c >> 2) ^ c ^ (rel >> 1)) & 1; };

  const unsigned lx = (unsigned)((16 * wv + (lane >> 3)) * KS + 4 * ((lane ^ (lane >> 3)) & 7)) * 4u;
  const unsigned lw = (unsigned)((64 * (wv >> 2) + 16 * ((wv >> 1) & 1) + 4 * q4) * NN + 64 * (wv & 1) + 4 * l15) * 4u;
  struct Src { __amdgpu_buffer_rsrc_t rs; unsigned so, s0, s1, s2, vo; };
  auto src_of = [&](int q, int dst, int hh) INL {
    const int blk = (q & 31) >> 1, kb = 256 * hh + 128 * (q & 1);
    const bool isx = !(q & 32);
    Src r;
    r.rs = rsrc_of(uniform_ptr(pw + (isx ? px - pw : 0)), 4096u * KS * 4u);
    r.so = UNI(isx ? (unsigned)((512 * (blk >> 1) + 256 * (dst >> 2) + 128 * (blk & 1)) * KS + kb) * 4u
                   : (unsigned)(kb * NN + 2048 * ((dst >> 1) & 1) + 128 * blk) * 4u);
    r.s0 = UNI(isx ? 128u : NN * 4u);
    r.s1 = UNI(isx ? 8u * KS * 4u : 2u * NN * 4u);
    r.s2 = UNI(isx ? 256u : 32u * NN * 4u);
    r.vo = lw + (isx ? lx - lw : 0u);
    return r;
  };
  auto load1 = [&](const Src& r, int t) INL {
    return __builtin_bit_cast(f32x4, __builtin_amdgcn_raw_buffer_load_b128(r.rs, r.vo, r.so + (t & 1) * r.s0 + ((t >> 1) & 1) * r.s1 + (t >> 2) * r.s2, IAUX));
  };
  auto load_item = [&](f32x4* v, int q, int dst, int hh) INL {
    const Src r = src_of(q, dst, hh);
#pragma unroll
    for (int t = 0; t < 8; ++t) v[t] = load1(r, t);
  };
  auto convert_item = [&](u32x4* f, const f32x4* v, int q) INL {
    if (q < 32) {
#pragma unroll
      for (int u = 0; u < 4; ++u)
        f[u] = u32x4{pack2(v[2 * u][0], v[2 * u][1]), pack2(v[2 * u][2], v[2 * u][3]),
                     pack2(v[2 * u + 1][0], v[2 * u + 1][1]), pack2(v[2 * u + 1][2], v[2 * u + 1][3])};
    } else {
#pragma unroll
      for (int u = 0; u < 4; ++u)
        f[u] = u32x4{pack2(v[0][u], v[1][u]), pack2(v[2][u], v[3][u]), pack2(v[4][u], v[5][u]), pack2(v[6][u], v[7][u])};
    }
#pragma unroll
    for (int u = 0; u < 4; ++u) asm volatile("" : "+v"(f[u]));
  };
  auto item_base = [&](int q, int chunk) -> unsigned {
    const unsigned pb = (q < 32 ? OFF_XS : OFF_WS) + (unsigned)(chunk * 16 + ((q & 31) >> 1)) * 65536u + (q & 1) * 32768u;
    return pb + (q < 32 ? wv * 2048u : wv * 4096u);
  };
  auto frag_off = [](int q, int u) INL { return (u & 1) * 1024u + (u >> 1) * (q < 32 ? 16384u : 2048u); };
  auto item_flag = [&](int q, int chunk) -> unsigned {
    return (q < 32 ? OFF_XF : OFF_WF) + ((q & 31) >> 1) * 128u + chunk * 8u + (q & 1) * 4u;
  };
  auto draw_call = [&]() {
    if (tid != 16) misc[tid] = 0;
    if (tid == NTHR - 64)
      misc[16] = __hip_atomic_fetch_add((unsigned*)(me + OFF_CALL + b * 128), 1u, __ATOMIC_RELAXED,
                                        __HIP_MEMORY_SCOPE_AGENT) + 1u;
  };
  const auto rme = rsrc_of(uniform_ptr(me), WS_NEED);
  unsigned call;
  if (idx < 7) {
    const bool busy = idx < 6;
    const int lrel = busy ? 2 + idx : rh ^ 1, far = (pair0 + lrel) & 7;
    const int ni = busy ? 60 : 0, nu = busy ? 84 - EC / 2 : 24 + 3 * EC;
    const int nj = 2 * nu + 8;
    char* const pfar = uniform_ptr(P.ws[far]);
    unsigned* const jobs = misc + 128;
    const auto rd = rsrc_of(pfar, WS_NEED);
    const int q0 = (xcd & 1 ? 32 : (far ^ 1) * 4) + (xcd & 2) + (xcd >> 2);
    f32x4 v[8];
    load_item(v, q0, far, far & 1);
    asm volatile("" ::: "memory");
    SB;
    int qtab, ttab;
    {
      int k = lane < OWNPOS ? lane : lane < OWNPOS + 4 ? lane - OWNPOS + 56 : lane - 4;
      const int dst = k < 8 * NP ? far : rank, src = k < 8 * NP ? rank : far;
      const int j = k < 32 ? (k >> 3) * 2 + ((k >> 1) & 1) : k < 40 ? 8 + (k & 3) : (k >> 3) * 4 - 12 + (k & 3);
      const int kh = (k >> 2) & 1;
      const bool isw = k < 32 ? k & 1 : k >= 40;
      const int second_blk = j >= 7, jt = second_blk ? 11 - j : j - 2;
      const int en = jt + (jt + 2 >= ((src - (dst & 6)) & 7));
      const int row = j < 2 ? (dst ^ 1) * 2 + j : (((dst & 6) + 2 + en) & 7) * 2 + second_blk;
      qtab = k >= 56 ? dst * 4 + (k & 3) : isw ? 32 + j * 2 + kh : row * 2 + kh;
      k = lane;
      const int mrel = (rank - (far & 6)) & 7;
      int to = mrel, ta, tc, sum = 0;
      if (busy) {
        int i = k - (8 - EC);
        const int fpos = FADAPT ? (mrel < 4 ? FP0 : mrel < 6 ? FP1 : 16) : FPOS;
        const bool one = i >= fpos && i < fpos + EC / 2;
        if (i >= fpos + EC / 2) i -= EC / 2;
        if (i < 0) { ta = 0; tc = 8 + i; }
        else if (one) { ta = 0; tc = 2 * (i - fpos); tc += giver(to, tc) == (far & 1); }
        else if (i < 8) { ta = 1; tc = i; }
        else { ta = i & 1; tc = (4 + (i >> 1)) & 15; }
      } else if (k >= 8 && k < 8 + 3 * EC) {
        if (FADAPT) { to = 2 + (k - 8) / (EC + !EC) * 2 + (k & 1); tc = (k - 8) % (EC + !EC) & ~1; }
        else { to = 2 + (k - 8) % 6; tc = 2 * ((k - 8) / 6); }
        ta = 0; tc += giver(to, tc) == rh; sum = to;
      } else {
        const int i = k < 8 ? k : k - 3 * EC;
        ta = i & 1; tc = (i >> 1) & 15;
      }
      const int there = T_PARK + to * 32 + ta * 16 + tc;
      ttab = there | (sum ? there : ((((far >> 2) * 2 + ta) * 2 + ((far >> 1) & 1)) * 16 + tc) * 3 + (far & 1)) << 10 | sum << 20;
    }
    if (busy) {
      u32x4 f[4];
      draw_call();
#pragma unroll
      for (int r = 0; r < NP; ++r) {
        const int q = r ? __builtin_amdgcn_readlane(qtab, 8 * r + xcd) : q0;
        convert_item(f, v, q);
        const unsigned sb = item_base(q, rank);
#pragma unroll
        for (int u = 0; u < 4; ++u) {
          st128(f[u], rd, lane * 16, sb + frag_off(q, u));
          if (u == 0 && r + 1 < NP) { load_item(v, __builtin_amdgcn_readlane(qtab, 8 * r + 8 + xcd), far, far & 1); WAIT_V(8); }
          if (u == 0 && r == 0) { BAR; call = UNI(misc[16]);
          }
          if (u == 0 && r + 1 < NP) continue;
          if (u == 3 || r + 1 < NP || (wv >= PW0 && wv < 7)) WAIT_V(0); else if (wv < PW0) WAIT_V(1);
        }
        unsigned tk = 0;
        if (lane == 0) tk = __hip_atomic_fetch_add(misc + 28 + r, 1u, __ATOMIC_RELAXED, __HIP_MEMORY_SCOPE_WORKGROUP);
        if (UNI(tk) == 7 && lane == 0) __builtin_amdgcn_raw_buffer_store_b32(2u * call, rd, item_flag(q, rank), 0, AUX_SYS);
      }
    } else {
      draw_call();
      BAR;
      call = UNI(misc[16]);
    }
    if (wv < 4) {
      auto job_of = [&](int n) INL { return (int)UNI(lds_get(jobs + n)) - 1; };
      char* const ring = lds + wv * 8192;
      unsigned so = 0, dn = 0, dc = 0;
      int sn = 0, sc = 0;
      unsigned an = 1, ac = 1, ap = 1;
      u64 fn = 0, fc = 0, fp = 0;
      auto decode = [&](int kk) INL {
        const int k = kk < 2 * nu ? kk >> 1 : kk - nu;
        const unsigned wo = ((kk < 2 * nu ? kk : ~far) & 1) * 16384u + wv * (ILV ? 1024u : 4096u);
        an = kk < 2 * nu ? 1 : 2;
        const int q = __builtin_amdgcn_readlane(qtab, min(k, 59));
        const unsigned tn = __builtin_amdgcn_readlane(ttab, max(k - ni, 0));
        if (k < ni) {
          const unsigned o = (q < 32 ? OFF_XS : OFF_WS) + (q & 31) * 32768u + wo;
          so = o + (unsigned)(8 + (q < 32 ? rank >> 2 : (rank >> 1) & 1) * 2 + rh) * (1u << 20);
          dn = o + (unsigned)far * (1u << 20);
          sn = 0;
          fn = (u64)me + item_flag(q, far);
        } else {
          so = OFF_TS + (tn & 1023u) * 32768u + wo;
          dn = OFF_TS + ((tn >> 10) & 1023u) * 32768u + wo;
          sn = tn >> 20;
          fn = sn ? (u64)P.ws[(pair0 + sn) & 7] + OFF_TF + 4 + (((rank >> 2) * 4 + ((rank >> 1) & 1)) * 16 + (tn & 15u)) * 96u + rh * 32u
                  : (u64)me + OFF_TF + ((tn >> 10) & 1023u) * 32u;
        }
      };
      auto read = [&](int u) INL {
        __builtin_amdgcn_global_load_lds((const unsigned*)(pfar + so + u * US + lane * 16), (unsigned*)(ring + u * 1024), 16, 0, RAUX);
        if (!busy) __builtin_amdgcn_global_load_lds((const unsigned*)(me + so + u * US + lane * 16), (unsigned*)(ring + 4096 + u * 1024), 16, 0, AUX_SYS);
      };
      auto write = [&](int u) INL {
        u32x4 x = *(const u32x4*)(ring + u * 1024 + lane * 16);
        if (sc) {
          const u32x4 y = *(const u32x4*)(ring + 4096 + u * 1024 + lane * 16);
#pragma unroll
          for (int d = 0; d < 4; ++d) x[d] = pack2(bf_lo(x[d]) + bf_lo(y[d]), bf_hi(x[d]) + bf_hi(y[d]));
        }
        st128(x, rme, lane * 16, dc + u * US);
      };
      auto publish = [&](int n) INL {
        if (fp) {
          unsigned tk = 0;
          if (lane == 0) tk = lds_inc(misc + 320 + n);
          if (UNI(tk) == 3 && lane == 0) __hip_atomic_fetch_add((gu32*)fp, ap, __ATOMIC_RELAXED, __HIP_MEMORY_SCOPE_SYSTEM);
        }
        fp = 0;
      };
#define LANDED(nb, np) do { if (busy) WAIT_V(nb); else WAIT_V(np); } while (0)
      bool flow = false;
#pragma unroll 1
      for (int n = 0;; ++n) {
        if (!flow) {
          int kn;
          while ((kn = job_of(n)) < 0) __builtin_amdgcn_s_sleep(1);
          if (kn >= nj) { lds_max(misc + 26, 1u + (kn - nj >= LATEF)); break; }
          decode(kn);
          read(0);
          read(1);
        }
        if (LOOK == 2 && lane == 0) lds_inc(misc + 17);
        dc = dn; sc = sn; fc = fn; ac = an;
        LANDED(1, 2);
        write(0); read(2);
        publish(n - 1);
        LANDED(2, 3);
        write(1); read(3);
        const int kn = job_of(n + 1);
        flow = kn >= 0 && kn < nj;
        if (flow) decode(kn);
        if (LOOK == 1 && lane == 0) lds_inc(misc + 17);
        LANDED(2, 3);
        write(2);
        const bool thin = (MW >> (MWN ? (n + wv) & 3 : wv)) & 1;
        if (flow && !thin) read(0);
        if (flow && !thin) LANDED(2, 3); else WAIT_V(1);
        if (flow && thin) read(0);
        if (kn >= nj) lds_max(misc + 26, 1u + (kn - nj >= LATEF));
        write(3);
        if (flow) read(1);
        fp = fc; ap = ac;
        if (!flow) { WAIT_V(0); publish(n); }
      }
    } else if (wv == 7) {
      const unsigned sum = (unsigned)ttab >> 20;
      const unsigned* fin = (const unsigned*)(me + OFF_AF) + far * 64 + qtab;
      const unsigned* ft0 = (const unsigned*)(me + OFF_TF + 4 + (sum ? T_HAND + (sum - 2) * 8 + (ttab & 15) : (ttab >> 10) & 1023) * 32);
      const unsigned* ft1 = (const unsigned*)(me + OFF_TF + (ttab & 1023) * 32);
      u64 seen_in = 0, seen_t = 0;
#pragma unroll 1
      for (int n = 0;; ++n) {
        unsigned kk = (busy ? 16 * NP : 0) + n * 8 + xcd;
        if (n >= 2) {
          while ((int)UNI(__hip_atomic_load(misc + 17, __ATOMIC_RELAXED, __HIP_MEMORY_SCOPE_WORKGROUP)) < 4 * (n - 1) - SLACK) __builtin_amdgcn_s_sleep(1);
          if (lane == 0) kk = __hip_atomic_fetch_add((unsigned*)(me + OFF_LU + idx * 128), 1u, __ATOMIC_RELAXED, __HIP_MEMORY_SCOPE_AGENT);
          kk = UNI(kk) + (busy ? 16u * NP + 16u : 16u) - (call - 1u) * (nj - (busy ? 16 * NP + 8 : 8));
        }
        if ((int)kk < nj) {
          const int k = (int)kk < 2 * nu ? kk >> 1 : kk - nu;
          const u64 t0 = __builtin_amdgcn_s_memrealtime();
          while (!((k < ni ? seen_in >> k : seen_t >> (k - ni)) & 1)) {
            const unsigned a = __hip_atomic_load(k < ni ? fin : ft0, __ATOMIC_RELAXED, __HIP_MEMORY_SCOPE_SYSTEM);
            unsigned c = a;
            if (!busy && sum) c = __hip_atomic_load(ft1, __ATOMIC_RELAXED, __HIP_MEMORY_SCOPE_SYSTEM);
            const u64 ok = __ballot((int)(a - 2u * call) >= 0 && (int)(c - 2u * call) >= 0);
            if (k < ni) seen_in = ok; else seen_t = ok;
            if ((ok >> (k < ni ? k : k - ni)) & 1) break;
            if (expired(t0, tmo, call, lane)) break;
            __builtin_amdgcn_s_sleep(SLP);
          }
        }
        if (lane == 0) __hip_atomic_store(jobs + n, kk + 1u, __ATOMIC_RELAXED, __HIP_MEMORY_SCOPE_WORKGROUP);
        if ((int)kk >= nj) break;
      }
    }
  } else if (t < 16) {
    f32x4 va[8], vb[8];
    u32x4 fa[4], fb[4];
    const int e = t * 8 + xcd, q = e >> 1, hf = e & 1;
    const int fd = q < 32 ? hf * 4 + (lane & 3) : hf * 2 + (lane & 2) * 2 + (lane & 1);
    u64 fa4 = (u64)P.ws[fd] + OFF_AF + (rank * 64 + q) * 4;
    draw_call();
    const bool soon = q < 32 ? !(q & 2) : q < 44;
    while (__builtin_amdgcn_s_memrealtime() - tstart < (soon ? LDELAY : LDELAY2)) __builtin_amdgcn_s_sleep(2);
    load_item(va, q, hf * 6, 0);
    load_item(vb, q, hf * 6, 1);
    convert_item(fa, va, q);
    convert_item(fb, vb, q);
    const unsigned sb = item_base(q, 8 + hf * 2);
#pragma unroll
    for (int u = 0; u < 4; ++u) {
      st128(fa[u], rme, lane * 16, sb + frag_off(q, u));
      st128(fb[u], rme, lane * 16, sb + (1u << 20) + frag_off(q, u));
    }
    WAIT_V(0);
    BAR;
    call = UNI(misc[16]);
    if (tid < 4) __hip_atomic_store((gu32*)fa4, 2u * call, __ATOMIC_RELAXED, __HIP_MEMORY_SCOPE_SYSTEM);
    if (tid < 2) __builtin_amdgcn_raw_buffer_store_b32(2u * call, rme, tid * 8, item_flag(q, 8 + hf * 2), AUX_SYS);
  } else {
    if (WARM & 1 && t == 16) {
      const char* c = (const char*)pc0 + tid * 128;
      if (tid < CODE_LINES) __builtin_amdgcn_global_load_lds((const unsigned*)c, (unsigned*)(lds + wv * 256), 4, 0, 0);
    }
    draw_call();
    BAR;
    call = UNI(misc[16]);
  }
  STAMP_AT(1);

  f32x4 acc[4][4];
  const int ks = wv >> 2, wm = (wv >> 1) & 1, wn = wv & 1;
  const bool has_brick = t >= 14 || t == 12 + (~xcd & 1);
  const int e = (t >= 14 ? t - 14 : 11) * 8 + xcd;
  const bool second = t == 0 || (t < 14 && orel >= 2 && !((xcd ^ orel) & 1));
  const char* lbw = lds + 16384 + ks * 8192 + wn * 4096 + lane * 16;
  const char* lbx = lds + (64 * wm + l15) * 128 + (((4 * ks + q4) ^ (lane & 7)) * 16);
  const int job1 = idx < 7 ? 0 : second ? 3 : 2;
  if (t >= 0) {
    const unsigned hold = has_brick ? BDELAY : !second && t != 7 ? ODELAY : 0;
    while (__builtin_amdgcn_s_memrealtime() - tstart < hold) __builtin_amdgcn_s_sleep(8);
  }
#pragma unroll 1
  for (int job = has_brick ? 0 : 1; job < job1;) {
    const bool brick = job == 0;
    const int owner = (pair0 + orel) & 7, rb = 2 * owner + ((xcd ^ orel) & 1);
    const int quad = 1 + (e >> 5);
    const int trb = brick ? 2 * rank + (e & 1) : rb, tcb = brick ? (e >> 1) & 15 : cb;
    const int blk = brick ? (rank >> 1) ^ quad : rank >> 1;
    unsigned todo = brick ? 0xfu : orel >= 2 ? 0xffffu & ~(3u << 2 * orel) : 0xffffu;
    if (!brick) STAMP_AT(9);
#pragma unroll
    for (int mi = 0; mi < 4; ++mi)
#pragma unroll
      for (int u = 0; u < 4; ++u) acc[mi][u] = f32x4{0.f, 0.f, 0.f, 0.f};
    const unsigned sw = l15 < 4 ? 16 + (lane < 32 ? blk & 2 : (blk & 1) * 2) * 2 + l15 : (2 * pair0 + lane) & 15;
    const char* fp = me + (lane < 32 ? OFF_XF + trb * 128 : OFF_WF + tcb * 128) + sw * 4;
    const unsigned tab = (sw >> 1) * (16u * 65536u) + (sw & 1u) * 32768u;
    const char* gx = me + OFF_XS + trb * 65536 + wv * 2048 + lane * 16;
    const char* gw = me + OFF_WS + tcb * 65536 + wv * 2048 + lane * 16;
    unsigned ox = 0, ow = 0;
    auto stage1 = [&](int buf, int part) INL {
      char* l = lds + buf * STAGE_B + wv * 2048;
      const char* g = (part & 2) ? gw + ow : gx + ox;
      __builtin_amdgcn_global_load_lds((const unsigned*)(g + (buf & 1) * 16384 + (part & 1) * 1024),
                                       (unsigned*)(l + (part >> 1) * 16384 + (part & 1) * 1024), 16, 0, AUX_SYS);
    };
    bf16x8 xa[4] = {}, wa[4] = {}, xb[4] = {}, wb[4] = {};
    auto step = [&](bf16x8* xc, bf16x8* wc, bf16x8* xn, bf16x8* wx, int buf, bool full, bool mf, bool rd, bool ld) INL {
      const int bo = ((buf + 1) & 3) * STAGE_B;
      if (full) {
#pragma unroll
        for (int mi = 0; mi < 4; ++mi) {
          acc[mi][0] = __builtin_amdgcn_mfma_f32_16x16x32_bf16(wc[0], xc[mi], acc[mi][0], 0, 0, 0);
          wx[mi] = *(const bf16x8*)(lbw + bo + mi * 1024);
          xn[mi] = *(const bf16x8*)(lbx + bo + mi * 2048);
          SB;
#pragma unroll
          for (int u = 1; u < 4; ++u)
            acc[mi][u] = __builtin_amdgcn_mfma_f32_16x16x32_bf16(wc[u], xc[mi], acc[mi][u], 0, 0, 0);
          stage1(buf, mi);
          SB;
        }
      } else {
        if (ld) {
#pragma unroll
          for (int part = 0; part < 4; ++part) stage1(buf, part);
        }
        if (rd) {
#pragma unroll
          for (int mi = 0; mi < 4; ++mi) {
            wx[mi] = *(const bf16x8*)(lbw + bo + mi * 1024);
            xn[mi] = *(const bf16x8*)(lbx + bo + mi * 2048);
          }
        }
        SB;
        if (mf) {
#pragma unroll
          for (int mi = 0; mi < 4; ++mi)
#pragma unroll
            for (int u = 0; u < 4; ++u)
              acc[mi][u] = __builtin_amdgcn_mfma_f32_16x16x32_bf16(wc[u], xc[mi], acc[mi][u], 0, 0, 0);
        }
      }
    };
    __builtin_amdgcn_s_waitcnt(0x0F70);
    if (wv == PWV) __builtin_amdgcn_global_load_lds((const unsigned*)fp, misc + 384 + 64, 4, 0, AUX_SYS);
    const u64 t0 = __builtin_amdgcn_s_memrealtime();
    unsigned ready = 0;
    bool h0 = false, h1 = false;
    int pb = 0;
    auto round = [&](bool full) INL {
      const unsigned avail = ready & todo;
      const bool h2 = full || avail != 0;
      if (h2) {
        const int sl = __builtin_ctz(avail);
        todo &= ~(1u << sl);
        ox = __builtin_amdgcn_readlane(tab, sl);
        ow = __builtin_amdgcn_readlane(tab, sl + 32);
      } else if (!h0 && !h1 && wv == PWV) {
        if (expired(t0, tmo, call, lane) && lane == 0) misc[24] = 0xffffu;
        __builtin_amdgcn_s_sleep(SLP);
      }
      if (full || h1) WAIT_V(8); else WAIT_V(0);
      BAR;
      const bool poll = wv == PWV && (!full || (__popc(ready & todo) <= PTH && (todo & ~ready)));
      if (poll) __builtin_amdgcn_global_load_lds((const unsigned*)fp, misc + 384 + pb * 32, 4, 0, AUX_SYS);
      const unsigned fl = misc[384 + (pb ^ 2) * 32 + lane], force = full ? 0u : misc[24];
      step(xa, wa, xb, wb, pb, full, h0, h0, h2);
      if (!h2) WAIT_V(4); else if (poll) WAIT_V(9); else WAIT_V(8);
      BAR;
      const u64 ok = __ballot((int)(fl - 2u * call) >= 0);
      ready |= ((unsigned)ok & (unsigned)(ok >> 32)) | UNI(force);
      step(xb, wb, xa, wa, pb + 1, full, h0, h1, h2);
      h0 = h1; h1 = h2; pb ^= 2;
    };
#pragma unroll 1
    do {
      if (h0 && h1) {
#pragma unroll 1
        while (ready & todo) round(true);
      }
      round(false);
    } while (todo | h0 | h1);

    f32x4 fin[2][4];
    {
      char* xb0 = lds + lane * 16;
      char* lo = xb0 + wv * 8192;
      const char* li = xb0 + (wv ^ 4) * 8192;
#pragma unroll
      for (int mi = 0; mi < 2; ++mi)
#pragma unroll
        for (int u = 0; u < 4; ++u) *(f32x4*)(lo + (mi * 4 + u) * 1024) = ks ? acc[mi][u] : acc[2 + mi][u];
      BAR;
#pragma unroll
      for (int mi = 0; mi < 2; ++mi)
#pragma unroll
        for (int u = 0; u < 4; ++u) fin[mi][u] = (ks ? acc[2 + mi][u] : acc[mi][u]) + *(const f32x4*)(li + (mi * 4 + u) * 1024);
    }
    if (!brick) STAMP_AT(3);
    const int pos = (((blk >> 1) * 2 + (trb & 1)) * 2 + (blk & 1)) * 16 + tcb;
    const int ti = brick ? pos * 3 + 2 : orel == rh ? pos * 3 + rh : T_PARK + orel * 32 + (rb & 1) * 16 + cb;
    const unsigned sp = OFF_TS + (unsigned)ti * 32768u + (unsigned)(((4 * wm + 2 * ks) * 2 + wn) * 2048);
#pragma unroll
    for (int mi = 0; mi < 2; ++mi)
#pragma unroll
      for (int s = 0; s < 2; ++s) {
        const u32x4 o = {pack2(fin[mi][0][2 * s], fin[mi][1][2 * s]), pack2(fin[mi][2][2 * s], fin[mi][3][2 * s]),
                         pack2(fin[mi][0][2 * s + 1], fin[mi][1][2 * s + 1]),
                         pack2(fin[mi][2][2 * s + 1], fin[mi][3][2 * s + 1])};
        st128(o, rme, lane * 16, sp + mi * 4096 + s * 1024);
      }
    WAIT_V(0);
    BAR;
    {
      const bool early = !brick && orel >= 2 && !(rb & 1) && cb < EC, give = early && giver(orel, cb) == rh;
      const bool here = brick || orel == rh || (early && !give);
      const auto rf = rsrc_of(uniform_ptr(P.ws[here ? rank : give ? rank ^ 1 : owner]), WS_NEED);
      const unsigned fo = here ? ti * 32 : 4 + (give ? T_HAND + (orel - 2) * 8 + cb : pos * 3 + rh) * 32;
      if (tid == 0) __builtin_amdgcn_raw_buffer_store_b32(2u * call, rf, fo, OFF_TF, AUX_SYS);
    }
    if (!brick) STAMP_AT(4);
    if (++job == 2) {
      STAMP_AT(12);
      if (t >= 1 && t < 7) cb = 12 + (xcd >> 1);
      else { cb = 4 * (t == 0 ? 3 : (orel - 2) >> 1) + (xcd >> 1); orel = rh; }
    }
  }
  STAMP_AT(5);

  float y[16];
  {
    const int sblk = (rpos >> 6) * 2 + ((rpos >> 4) & 1);
    const bool foreign = sblk != (rank >> 1);
    unsigned mask = foreign ? 7u : 3u;
    if (foreign && !((rpos >> 5) & 1) && (rpos & 15) < EC) mask = 4u | (2u >> giver((rank - 2 * sblk) & 7, rpos & 15));
    const unsigned* tf = (const unsigned*)(me + OFF_TF + rpos * 96 + (lane & 3) * 32);
    if (wv == 6 && lane < 8) *(u64*)(misc + 32 + 2 * lane) = (u64)(P.ws[lane] + OFF_AG) + (rank * 128 + (b < 56 ? b : b >= NWG - NDIR ? b - (NWG - NDIR) + 57 : 56)) * 8;
    u32x4 p[3][2] = {};
    if (idx < 7) {
      mask &= ~(1u << rhalf);
      if (wv == 6) {
        wait_words(tf + 1, lane == rhalf, 2u * call, tmo, call, lane);
        lds_max(misc + 25, 1u);
        do {
          unsigned v = 2u * call;
          if (lane < 3 && (mask >> lane & 1)) v = __hip_atomic_load(tf, __ATOMIC_RELAXED, __HIP_MEMORY_SCOPE_SYSTEM);
          if (__all((int)(v - 2u * call) >= 0)) { lds_max(misc + 27, 1u); break; }
        } while (!UNI(lds_get(misc + 26)));
      }
      while (!(UNI(lds_get(misc + 25)) && UNI(lds_get(misc + 26)))) __builtin_amdgcn_s_sleep(1);
      const unsigned sr = OFF_TS + (T_PARK + ((rank - 2 * sblk) & 7) * 32 + ((rpos >> 5) & 1) * 16 + (rpos & 15)) * 32768u + rhalf * 16384 + wv * 2048;
      const auto rr = rsrc_of(uniform_ptr(P.ws[2 * sblk + rhalf]), WS_NEED);
      p[0][0] = __builtin_amdgcn_raw_buffer_load_b128(rr, lane * 16, sr, AUX_SYS);
      if (UNI(lds_get(misc + 26)) == 1) WAIT_V(0);
      p[0][1] = __builtin_amdgcn_raw_buffer_load_b128(rr, lane * 16, sr + 1024, AUX_SYS);
    }
    if (wv == 6 && !UNI(lds_get(misc + 27))) {
      wait_words(tf, lane < 3 && (mask >> lane & 1), 2u * call, tmo, call, lane);
      lds_max(misc + 27, 1u);
    }
    if (idx < 7) {
      while (!UNI(lds_get(misc + 27))) __builtin_amdgcn_s_sleep(1);
    } else {
      BAR;
    }
    STAMP_AT(6);
    const unsigned so = OFF_TS + rpos * 3 * 32768 + rhalf * 16384 + wv * 2048;
    if (idx >= 7) {
      const unsigned vo = (mask & 1) ? lane * 16 : 0x7ffffff0u;
      p[0][0] = __builtin_amdgcn_raw_buffer_load_b128(rme, vo, so, AUX_SYS);
      p[0][1] = __builtin_amdgcn_raw_buffer_load_b128(rme, vo, so + 1024, AUX_SYS);
    }
#pragma unroll
    for (int t = 1; t < 3; ++t) {
      const int sl = t == 1 && idx < 7 ? rhalf ^ 1 : t;
      const unsigned vo = (mask >> sl & 1) ? lane * 16 : 0x7ffffff0u;
      p[t][0] = __builtin_amdgcn_raw_buffer_load_b128(rme, vo, so + sl * 32768, AUX_SYS);
      p[t][1] = __builtin_amdgcn_raw_buffer_load_b128(rme, vo, so + sl * 32768 + 1024, AUX_SYS);
    }
    STAMP_AT(10);
#pragma unroll
    for (int s = 0; s < 2; ++s)
#pragma unroll
      for (int d = 0; d < 4; ++d) {
        y[8 * s + 2 * d] = bf_lo(p[0][s][d]) + bf_lo(p[1][s][d]) + bf_lo(p[2][s][d]);
        y[8 * s + 2 * d + 1] = bf_hi(p[0][s][d]) + bf_hi(p[1][s][d]) + bf_hi(p[2][s][d]);
      }
    unsigned am = 0;
#pragma unroll
    for (int t = 0; t < 16; ++t) am = max(am, __builtin_bit_cast(unsigned, y[t]) & 0x7fffffffu);
    am = wave_max(am);
    unsigned tk = 0;
    if (lane == 0) {
      __hip_atomic_fetch_max(misc + 20, am, __ATOMIC_RELAXED, __HIP_MEMORY_SCOPE_WORKGROUP);
      tk = __hip_atomic_fetch_add(misc + 19, 1u, __ATOMIC_RELAXED, __HIP_MEMORY_SCOPE_WORKGROUP);
    }
    if (UNI(tk) == 7) {
      u64 g = ((u64)call << 32) | __hip_atomic_load(misc + 20, __ATOMIC_RELAXED, __HIP_MEMORY_SCOPE_WORKGROUP);
      bool out = idx < 7 || b >= NWG - NDIR;
      if (!out) {
        unsigned n = 0;
        if (lane == 0) {
          __hip_atomic_fetch_max((u64*)(me + OFF_ACT), g, __ATOMIC_RELAXED, __HIP_MEMORY_SCOPE_AGENT);
          WAIT_V(0);
          n = __hip_atomic_fetch_add((unsigned*)(me + OFF_ACT + 128), 1u, __ATOMIC_RELAXED, __HIP_MEMORY_SCOPE_AGENT);
        }
        out = UNI(n) + 1u == call * (NWG - 56 - NDIR);
        if (out) g = __hip_atomic_load((u64*)(me + OFF_ACT), __ATOMIC_RELAXED, __HIP_MEMORY_SCOPE_AGENT);
      }
      if (out && lane < 8) __hip_atomic_store((gu64*)*(const u64*)(misc + 32 + 2 * lane), g, __ATOMIC_RELAXED, __HIP_MEMORY_SCOPE_SYSTEM);
    }
  }
  if (idx >= 7) {
    if (wv == 0) wait_words((const unsigned*)(me + OFF_AG + (rank * 128 + 56) * 8 + 4), lane == 0, call, tmo, call, lane);
    BAR;
  }
  {
    const u64 t0 = __builtin_amdgcn_s_memrealtime();
    unsigned gm;
    for (;;) {
      const u64 g = __hip_atomic_load((const u64*)(me + OFF_AG) + wv * 128 + min(lane, 56 + NDIR), __ATOMIC_RELAXED, __HIP_MEMORY_SCOPE_SYSTEM);
      const u64 h = __hip_atomic_load((const u64*)(me + OFF_AG) + wv * 128 + min(64 + lane, 56 + NDIR), __ATOMIC_RELAXED, __HIP_MEMORY_SCOPE_SYSTEM);
      gm = max((unsigned)g, (unsigned)h);
      if (__all((unsigned)(g >> 32) == call && (unsigned)(h >> 32) == call)) break;
      if (expired(t0, tmo, call, lane)) break;
      __builtin_amdgcn_s_sleep(1);
    }
    gm = wave_max(gm);
    if (lane == 0) __hip_atomic_fetch_max(misc + 21, gm, __ATOMIC_RELAXED, __HIP_MEMORY_SCOPE_WORKGROUP);
    if (__builtin_amdgcn_s_memrealtime() - tstart > TMO_LOOK &&
        __hip_atomic_load(tmo, __ATOMIC_RELAXED, __HIP_MEMORY_SCOPE_AGENT) == call && lane == 0)
      misc[22] = 1;
  }
  BAR;
  STAMP_AT(7);
  {
    const float amax = __builtin_bit_cast(float, misc[21]);
    const float inv = 448.0f / amax;
    u32x4 o;
#pragma unroll
    for (int d = 0; d < 4; ++d) {
      int r = __builtin_amdgcn_cvt_pk_fp8_f32(y[4 * d] * inv, y[4 * d + 1] * inv, 0, false);
      r = __builtin_amdgcn_cvt_pk_fp8_f32(y[4 * d + 2] * inv, y[4 * d + 3] * inv, r, true);
      o[d] = (unsigned)r;
    }
    if (misc[22]) o = u32x4{0x7f7f7f7fu, 0x7f7f7f7fu, 0x7f7f7f7fu, 0x7f7f7f7fu};
    const int row = 128 * (rpos >> 5) + 16 * (rhalf * 4 + (wv >> 1)) + l15;
    const int col = 128 * (rpos & 31) + 64 * (wv & 1) + 16 * q4;
    __builtin_amdgcn_raw_buffer_store_b128(o, rsrc_of(P.out, 512u * NN), (unsigned)(row * NN + col), 0, OAUX);
  }
  STAMP_AT(8);
}

}

extern "C" void kernel_launch(int rank, int nranks, ncclComm_t comm, hipStream_t stream,
                              void* const* d_in, const int* in_sizes, int n_in, void* d_out, int out_size,
                              const int* in_dtypes, int out_dtype, void* const* d_out_all,
                              void* const* d_ws_all, size_t ws_bytes) {
  (void)comm; (void)in_dtypes; (void)out_dtype; (void)d_out_all;
  if (nranks != NR || (unsigned)rank >= (unsigned)NR || n_in < 2 || ws_bytes < WS_NEED ||
      in_sizes[0] != 4096 * KS || in_sizes[1] != KS * NN || out_size != 512 * NN) {
    fprintf(stderr, "unexpected problem shape\n");
    return;
  }
  Params p;
  p.x = (const float*)d_in[0];
  p.w = (const float*)d_in[1];
  p.out = (unsigned char*)d_out;
  for (int r = 0; r < NR; ++r) p.ws[r] = (char*)d_ws_all[r];
  p.rank = rank;
  p.me = p.ws[rank];
  gemm_rs_kernel<<<NWG, NTHR, 0, stream>>>(p);
  HIP_CHECK(hipGetLastError());
}
```

```cpp
#include <hip/hip_runtime.h>
#include <rccl/rccl.h>
#include <cstdio>

#define HIP_CHECK(expr) do { hipError_t _e = (expr); if (_e != hipSuccess) { \
    fprintf(stderr, "HIP error %s at %s:%d\n", hipGetErrorString(_e), __FILE__, __LINE__); \
    return; } } while (0)

namespace {

constexpr int NR = 8;
constexpr int NN = 4096;
constexpr int KS = 512;
constexpr int NWG = 256;
constexpr int NTHR = 512;

#define EC 4
#define LDELAY 80
#define DRAW_U 3
#define FPOS 16
#define SLPE 2
#define MINA 2
#define OWNLAST 1
#define LASTW 0
#define PRO 2
#define E3 0
#define G8 1
#define LSC "sc0 sc1"
#define FAKELD 0
#define NOGEMM 0
#define DW 1
#define PW0 4
#define BDELAY 0
#define ODELAY 0
#define PTH 16
#define PWV 0
#define OWNPOS 56
#define LDELAY2 250
#define LOOK 2
#define FADAPT 1
#define FP0 0
#define FP1 8
#define NP 1
#define LATEF 4
#define MW 0
#define MWN 1
#define SLACK 0
#define ILV 0
#define US (ILV ? 4096 : 1024)
#define NDIR 40
#define WARM 1
#define CODE_LINES 150
#define IAUX 2
#define OAUX 17
#define RAUX 17
#define SAUX 17
#define SPEC 4800
#define HALF2 0
#define PT0 9
#define ATH 32
#define SLP 2

constexpr unsigned OFF_CALL = 0;
constexpr unsigned OFF_TMO = 32768;
constexpr unsigned OFF_LU = OFF_TMO + 128;
constexpr unsigned OFF_ACT = OFF_TMO + 1024;
constexpr unsigned OFF_XF = OFF_TMO + 4096;
constexpr unsigned OFF_WF = OFF_XF + 2048;
constexpr unsigned OFF_AG = OFF_TMO + 8192;
constexpr unsigned OFF_HC = OFF_TMO + 12288;
constexpr unsigned OFF_DBG = 65536;
constexpr unsigned OFF_AF = OFF_TMO + 2048;
constexpr unsigned OFF_TF = 131072;
constexpr unsigned OFF_XS = 1u << 20;
constexpr unsigned OFF_WS = OFF_XS + (12u << 20);
constexpr unsigned OFF_TS = OFF_WS + (12u << 20);
constexpr int T_HAND = 384;
constexpr int T_PARK = T_HAND + 48;
constexpr unsigned WS_NEED = OFF_TS + (T_PARK + 256) * 32768u;

constexpr unsigned long long DEADLINE = 100000000ull;
constexpr unsigned long long TMO_LOOK = 20000;
constexpr int AUX_SYS = 17;

constexpr int STAGE_B = 32768;
constexpr int LDS_MISC = 4 * STAGE_B;
constexpr int LDS_B = LDS_MISC + 2048;

typedef __bf16 bf16x8 __attribute__((ext_vector_type(8)));
typedef __bf16 bf16x2 __attribute__((ext_vector_type(2)));
typedef float f32x2 __attribute__((ext_vector_type(2)));
typedef float f32x4 __attribute__((ext_vector_type(4)));
typedef unsigned u32x4 __attribute__((ext_vector_type(4)));
typedef unsigned long long u64;
typedef __attribute__((address_space(1))) unsigned long long gu64;
typedef __attribute__((address_space(1))) unsigned gu32;

struct Params {
  const float* x;
  const float* w;
  char* me;
  long rank;
  unsigned char* out;
  char* ws[NR];
};

__device__ __forceinline__ unsigned pack2(float lo, float hi) {
  f32x2 v = {lo, hi};
  return __builtin_bit_cast(unsigned, __builtin_convertvector(v, bf16x2));
}
__device__ __forceinline__ float bf_lo(unsigned u) { return __builtin_bit_cast(float, u << 16); }
__device__ __forceinline__ float bf_hi(unsigned u) { return __builtin_bit_cast(float, u & 0xffff0000u); }

__device__ __forceinline__ __amdgpu_buffer_rsrc_t rsrc_of(const void* p, unsigned bytes) {
  return __builtin_amdgcn_make_buffer_rsrc(const_cast<void*>(p), 0, bytes, 0x00020000);
}
__device__ __forceinline__ void st128(u32x4 v, __amdgpu_buffer_rsrc_t r, unsigned vo, unsigned so) {
  __builtin_amdgcn_raw_buffer_store_b128(v, r, vo, so, SAUX);
  asm volatile("s_nop 1" :: "v"(v) : "memory");
}
__device__ __forceinline__ unsigned lds_addr(const void* p) { return (unsigned)(size_t)(const __attribute__((address_space(3))) char*)p; }
__device__ __forceinline__ unsigned lds_get(const unsigned* p) {
  unsigned r;
  asm volatile("ds_read_b32 %0, %1\n\ts_waitcnt lgkmcnt(0)" : "=&v"(r) : "v"(lds_addr(p)) : "memory");
  return r;
}
__device__ __forceinline__ unsigned lds_inc(unsigned* p) {
  unsigned r;
  asm volatile("ds_add_rtn_u32 %0, %1, %2\n\ts_waitcnt lgkmcnt(0)" : "=&v"(r) : "v"(lds_addr(p)), "v"(1u) : "memory");
  return r;
}
__device__ __forceinline__ void lds_max(unsigned* p, unsigned v) {
  asm volatile("ds_max_u32 %0, %1\n\ts_waitcnt lgkmcnt(0)" :: "v"(lds_addr(p)), "v"(v) : "memory");
}
__device__ __forceinline__ unsigned wave_max(unsigned v) {
  v = max(v, (unsigned)__builtin_amdgcn_mov_dpp((int)v, 0xB1, 0xf, 0xf, true));
  v = max(v, (unsigned)__builtin_amdgcn_mov_dpp((int)v, 0x4E, 0xf, 0xf, true));
  v = max(v, (unsigned)__builtin_amdgcn_mov_dpp((int)v, 0x141, 0xf, 0xf, true));
  v = max(v, (unsigned)__builtin_amdgcn_mov_dpp((int)v, 0x140, 0xf, 0xf, true));
  const unsigned a = __builtin_amdgcn_readlane(v, 0), b = __builtin_amdgcn_readlane(v, 16);
  const unsigned c = __builtin_amdgcn_readlane(v, 32), d = __builtin_amdgcn_readlane(v, 48);
  return max(max(a, b), max(c, d));
}
__device__ __forceinline__ char* uniform_ptr(char* p) {
  const u64 a = (u64)p;
  const unsigned lo = __builtin_amdgcn_readfirstlane((unsigned)a), hi = __builtin_amdgcn_readfirstlane((unsigned)(a >> 32));
  return (char*)(((u64)hi << 32) | lo);
}
#define INL __attribute__((always_inline))
#define UNI(x) __builtin_amdgcn_readfirstlane(x)
#define WAIT_V_(n) asm volatile("s_waitcnt vmcnt(" #n ")" ::: "memory")
#define WAIT_V(n) WAIT_V_(n)
#define SB __builtin_amdgcn_sched_barrier(0)
#define BAR do { asm volatile("" ::: "memory"); __builtin_amdgcn_s_waitcnt(0xC07F);   \
                 __builtin_amdgcn_s_barrier(); asm volatile("" ::: "memory"); SB; } while (0)
#define ACQ __builtin_amdgcn_fence(__ATOMIC_ACQUIRE, "wavefront")

#define STAMP_AT(k) do { } while (0)

__device__ __forceinline__ bool expired(u64 t0, unsigned* tmo, unsigned call, int lane) {
  const u64 dt = __builtin_amdgcn_s_memrealtime() - t0;
  if (dt > TMO_LOOK) {
    if (UNI(__hip_atomic_load(tmo, __ATOMIC_RELAXED, __HIP_MEMORY_SCOPE_AGENT)) == call) return true;
    if (dt > DEADLINE) {
      if (lane == 0) __hip_atomic_store(tmo, call, __ATOMIC_RELAXED, __HIP_MEMORY_SCOPE_AGENT);
      return true;
    }
  }
  return false;
}
__device__ __forceinline__ void wait_words(const unsigned* p, bool need, unsigned want, unsigned* tmo,
                                           unsigned call, int lane) {
  const u64 t0 = __builtin_amdgcn_s_memrealtime();
  for (;;) {
    unsigned v = want;
    if (need) v = __hip_atomic_load(p, __ATOMIC_RELAXED, __HIP_MEMORY_SCOPE_SYSTEM);
    if (__all((int)(v - want) >= 0)) break;
    if (expired(t0, tmo, call, lane)) break;
    __builtin_amdgcn_s_sleep(SLP);
  }
}

__global__ void __launch_bounds__(NTHR, 2) gemm_rs_kernel(Params P) {
  __shared__ __attribute__((aligned(16))) char lds[LDS_B];
  const int tid = threadIdx.x;
  const int lane = tid & 63;
  const int wv = UNI(tid >> 6);
  const int l15 = lane & 15, q4 = lane >> 4;
  u64 pc0;
  asm volatile("s_getpc_b64 %0" : "=s"(pc0));
  char* const px = (char*)P.x;
  char* const pw = (char*)P.w;
  char* const me = P.me;
  const int rank = (int)P.rank;
  asm volatile("" :: "s"(px), "s"(pw), "s"(me), "s"(rank));
  const int rh = rank & 1, pair0 = rank & 6;
  unsigned* const tmo = (unsigned*)(me + OFF_TMO);
  unsigned* const misc = (unsigned*)(lds + LDS_MISC);

  const int b = UNI(blockIdx.x), xcd = b & 7, idx = b >> 3;
  STAMP_AT(0);
  const u64 tstart = __builtin_amdgcn_s_memrealtime();
  const int t = idx - 7;
  int orel, cq;
  if (t < 22) { const int ro = t % 7; orel = ro ? ro + 1 : rh ^ 1; cq = t / 7; }
  else { orel = 2 + 2 * (t - 22) + (~xcd & 1); cq = 3; }
  int cb = 4 * cq + (xcd >> 1);
  int rpos, rhalf;
  {
    const int u = b - 56, far = idx < 6 ? (pair0 + 2 + idx) & 7 : idx == 6 ? rank ^ 1 : rank;
    const bool last = idx < 7 || u >= 192;
    const int blk = last ? far >> 1 : (u >> 2) & 3, ra = last ? xcd & 1 : (u >> 1) & 1, rc = last ? 12 + (xcd >> 1) : u >> 4;
    rpos = (((blk >> 1) * 2 + ra) * 2 + (blk & 1)) * 16 + rc;
    rhalf = (last ? far : u) & 1;
  }
  auto giver = [](int rel, int c) { return ((c >> 2) ^ c ^ (rel >> 1)) & 1; };

  const unsigned lx = (unsigned)((16 * wv + (lane >> 3)) * KS + 4 * ((lane ^ (lane >> 3)) & 7)) * 4u;
  const unsigned lw = (unsigned)((64 * (wv >> 2) + 16 * ((wv >> 1) & 1) + 4 * q4) * NN + 64 * (wv & 1) + 4 * l15) * 4u;
  struct Src { __amdgpu_buffer_rsrc_t rs; unsigned so, s0, s1, s2, vo; };
  auto src_of = [&](int q, int dst, int hh) INL {
    const int blk = (q & 31) >> 1, kb = 256 * hh + 128 * (q & 1);
    const bool isx = !(q & 32);
    Src r;
    r.rs = rsrc_of(uniform_ptr(pw + (isx ? px - pw : 0)), 4096u * KS * 4u);
    r.so = UNI(isx ? (unsigned)((512 * (blk >> 1) + 256 * (dst >> 2) + 128 * (blk & 1)) * KS + kb) * 4u
                   : (unsigned)(kb * NN + 2048 * ((dst >> 1) & 1) + 128 * blk) * 4u);
    r.s0 = UNI(isx ? 128u : NN * 4u);
    r.s1 = UNI(isx ? 8u * KS * 4u : 2u * NN * 4u);
    r.s2 = UNI(isx ? 256u : 32u * NN * 4u);
    r.vo = lw + (isx ? lx - lw : 0u);
    return r;
  };
  auto load1 = [&](const Src& r, int t) INL {
    return __builtin_bit_cast(f32x4, __builtin_amdgcn_raw_buffer_load_b128(r.rs, r.vo, r.so + (t & 1) * r.s0 + ((t >> 1) & 1) * r.s1 + (t >> 2) * r.s2, IAUX));
  };
  auto load_item = [&](f32x4* v, int q, int dst, int hh) INL {
    const Src r = src_of(q, dst, hh);
#pragma unroll
    for (int t = 0; t < 8; ++t) v[t] = load1(r, t);
  };
  auto convert_item = [&](u32x4* f, const f32x4* v, int q) INL {
    if (q < 32) {
#pragma unroll
      for (int u = 0; u < 4; ++u)
        f[u] = u32x4{pack2(v[2 * u][0], v[2 * u][1]), pack2(v[2 * u][2], v[2 * u][3]),
                     pack2(v[2 * u + 1][0], v[2 * u + 1][1]), pack2(v[2 * u + 1][2], v[2 * u + 1][3])};
    } else {
#pragma unroll
      for (int u = 0; u < 4; ++u)
        f[u] = u32x4{pack2(v[0][u], v[1][u]), pack2(v[2][u], v[3][u]), pack2(v[4][u], v[5][u]), pack2(v[6][u], v[7][u])};
    }
#pragma unroll
    for (int u = 0; u < 4; ++u) asm volatile("" : "+v"(f[u]));
  };
  auto item_base = [&](int q, int chunk) -> unsigned {
    const unsigned pb = (q < 32 ? OFF_XS : OFF_WS) + (unsigned)(chunk * 16 + ((q & 31) >> 1)) * 65536u + (q & 1) * 32768u;
    return pb + (q < 32 ? wv * 2048u : wv * 4096u);
  };
  auto frag_off = [](int q, int u) INL { return (u & 1) * 1024u + (u >> 1) * (q < 32 ? 16384u : 2048u); };
  auto item_flag = [&](int q, int chunk) -> unsigned {
    return (q < 32 ? OFF_XF : OFF_WF) + ((q & 31) >> 1) * 128u + chunk * 8u + (q & 1) * 4u;
  };
  auto draw_call = [&]() {
    if (tid != 16) misc[tid] = 0;
    if (tid == NTHR - 64)
      misc[16] = __hip_atomic_fetch_add((unsigned*)(me + OFF_CALL + b * 128), 1u, __ATOMIC_RELAXED,
                                        __HIP_MEMORY_SCOPE_AGENT) + 1u;
  };
  const auto rme = rsrc_of(uniform_ptr(me), WS_NEED);
  unsigned call;
  if (idx < 7) {
    const bool busy = idx < 6;
    const int lrel = busy ? 2 + idx : rh ^ 1, far = (pair0 + lrel) & 7;
    const int ni = busy ? 60 : 0, nu = busy ? 84 - EC / 2 : 24 + 3 * EC;
    const int nj = 2 * nu + 8;
    char* const pfar = uniform_ptr(P.ws[far]);
    unsigned* const jobs = misc + 128;
    const auto rd = rsrc_of(pfar, WS_NEED);
    const int q0 = (xcd & 1 ? 32 : (far ^ 1) * 4) + (xcd & 2) + (xcd >> 2);
    f32x4 v[8];
    load_item(v, q0, far, far & 1);
    asm volatile("" ::: "memory");
    SB;
    int qtab, ttab;
    {
      int k = lane < OWNPOS ? lane : lane < OWNPOS + 4 ? lane - OWNPOS + 56 : lane - 4;
      const int dst = k < 8 * NP ? far : rank, src = k < 8 * NP ? rank : far;
      const int j = k < 32 ? (k >> 3) * 2 + ((k >> 1) & 1) : k < 40 ? 8 + (k & 3) : (k >> 3) * 4 - 12 + (k & 3);
      const int kh = (k >> 2) & 1;
      const bool isw = k < 32 ? k & 1 : k >= 40;
      const int second_blk = j >= 7, jt = second_blk ? 11 - j : j - 2;
      const int en = jt + (jt + 2 >= ((src - (dst & 6)) & 7));
      const int row = j < 2 ? (dst ^ 1) * 2 + j : (((dst & 6) + 2 + en) & 7) * 2 + second_blk;
      qtab = k >= 56 ? dst * 4 + (k & 3) : isw ? 32 + j * 2 + kh : row * 2 + kh;
      k = lane;
      const int mrel = (rank - (far & 6)) & 7;
      int to = mrel, ta, tc, sum = 0;
      if (busy) {
        int i = k - (8 - EC);
        const int fpos = FADAPT ? (mrel < 4 ? FP0 : mrel < 6 ? FP1 : 16) : FPOS;
        const bool one = i >= fpos && i < fpos + EC / 2;
        if (i >= fpos + EC / 2) i -= EC / 2;
        if (i < 0) { ta = 0; tc = 8 + i; }
        else if (one) { ta = 0; tc = 2 * (i - fpos); tc += giver(to, tc) == (far & 1); }
        else if (i < 8) { ta = 1; tc = i; }
        else { ta = i & 1; tc = (4 + (i >> 1)) & 15; }
      } else if (k >= 8 && k < 8 + 3 * EC) {
        if (FADAPT) { to = 2 + (k - 8) / (EC + !EC) * 2 + (k & 1); tc = (k - 8) % (EC + !EC) & ~1; }
        else { to = 2 + (k - 8) % 6; tc = 2 * ((k - 8) / 6); }
        ta = 0; tc += giver(to, tc) == rh; sum = to;
      } else {
        const int i = k < 8 ? k : k - 3 * EC;
        ta = i & 1; tc = (i >> 1) & 15;
      }
      const int there = T_PARK + to * 32 + ta * 16 + tc;
      ttab = there | (sum ? there : ((((far >> 2) * 2 + ta) * 2 + ((far >> 1) & 1)) * 16 + tc) * 3 + (far & 1)) << 10 | sum << 20;
    }
    if (busy) {
      u32x4 f[4];
      draw_call();
#pragma unroll
      for (int r = 0; r < NP; ++r) {
        const int q = r ? __builtin_amdgcn_readlane(qtab, 8 * r + xcd) : q0;
        convert_item(f, v, q);
        const unsigned sb = item_base(q, rank);
#pragma unroll
        for (int u = 0; u < 4; ++u) {
          st128(f[u], rd, lane * 16, sb + frag_off(q, u));
          if (u == 0 && r + 1 < NP) { load_item(v, __builtin_amdgcn_readlane(qtab, 8 * r + 8 + xcd), far, far & 1); WAIT_V(8); }
          if (u == 0 && r == 0) { BAR; call = UNI(misc[16]);
          }
          if (u == 0 && r + 1 < NP) continue;
          if (u == 3 || r + 1 < NP || (wv >= PW0 && wv < 7)) WAIT_V(0); else if (wv < PW0) WAIT_V(1);
        }
        unsigned tk = 0;
        if (lane == 0) tk = __hip_atomic_fetch_add(misc + 28 + r, 1u, __ATOMIC_RELAXED, __HIP_MEMORY_SCOPE_WORKGROUP);
        if (UNI(tk) == 7 && lane == 0) __builtin_amdgcn_raw_buffer_store_b32(2u * call, rd, item_flag(q, rank), 0, AUX_SYS);
      }
    } else {
      draw_call();
      BAR;
      call = UNI(misc[16]);
    }
    auto job_of = [&](int n) INL { return (int)UNI(lds_get(jobs + n)) - 1; };
    if (wv < 4) {
      char* const ring = lds + wv * 8192;
      unsigned so = 0, dn = 0, dc = 0;
      bool sn = false, sc = false;
      auto decode = [&](int kk) INL {
        const int k = kk < 2 * nu ? kk >> 1 : kk - nu;
        const unsigned wo = ((kk < 2 * nu ? kk : ~far) & 1) * 16384u + wv * 4096u;
        const int q = __builtin_amdgcn_readlane(qtab, min(k, 59));
        const unsigned tn = __builtin_amdgcn_readlane(ttab, max(k - ni, 0));
        const unsigned o = (q < 32 ? OFF_XS : OFF_WS) + (q & 31) * 32768u + wo;
        so = k < ni ? o + (unsigned)(8 + (q < 32 ? rank >> 2 : (rank >> 1) & 1) * 2 + rh) * (1u << 20) : OFF_TS + (tn & 1023u) * 32768u + wo;
        dn = k < ni ? o + (unsigned)far * (1u << 20) : OFF_TS + ((tn >> 10) & 1023u) * 32768u + wo;
        sn = tn >> 20;
      };
      auto read = [&](int u) INL {
        __builtin_amdgcn_global_load_lds((const unsigned*)(pfar + so + u * 1024 + lane * 16), (unsigned*)(ring + u * 1024), 16, 0, AUX_SYS);
        if (!busy) __builtin_amdgcn_global_load_lds((const unsigned*)(me + so + u * 1024 + lane * 16), (unsigned*)(ring + 4096 + u * 1024), 16, 0, AUX_SYS);
      };
      auto write = [&](int u) INL {
        u32x4 x = *(const u32x4*)(ring + u * 1024 + lane * 16);
        if (sc) {
          const u32x4 y = *(const u32x4*)(ring + 4096 + u * 1024 + lane * 16);
#pragma unroll
          for (int d = 0; d < 4; ++d) x[d] = pack2(bf_lo(x[d]) + bf_lo(y[d]), bf_hi(x[d]) + bf_hi(y[d]));
        }
        st128(x, rme, lane * 16, dc + u * 1024);
      };
#define LANDED(nb, np) do { if (busy) WAIT_V(nb); else WAIT_V(np); } while (0)
      bool flow = false;
#pragma unroll 1
      for (int n = 0;; ++n) {
        if (!flow) {
          int kn;
          while ((kn = job_of(n)) < 0) __builtin_amdgcn_s_sleep(1);
          if (kn >= nj) { lds_max(misc + 26, 1u + (kn - nj >= LATEF)); break; }
          decode(kn);
          read(0);
          read(1);
        }
        if (LOOK == 2 && lane == 0) lds_inc(misc + 17);
        dc = dn; sc = sn;
        LANDED(1, 2);
        if (flow && lane == 0) lds_inc(misc + 320 + n - 1);
        write(0); read(2);
        LANDED(2, 3);
        write(1); read(3);
        const int kn = job_of(n + 1);
        flow = kn >= 0 && kn < nj;
        if (flow) decode(kn);
        if (LOOK == 1 && lane == 0) lds_inc(misc + 17);
        LANDED(2, 3);
        write(2);
        if (flow) read(0);
        if (flow) LANDED(2, 3); else WAIT_V(1);
        if (kn >= nj) lds_max(misc + 26, 1u + (kn - nj >= LATEF));
        write(3);
        if (flow) read(1);
        else { WAIT_V(0); if (lane == 0) lds_inc(misc + 320 + n); }
      }
    } else if (wv == 5) {
#pragma unroll 1
      for (int n = 0;; ++n) {
        int kk;
        while ((kk = job_of(n)) < 0) __builtin_amdgcn_s_sleep(1);
        if (kk >= nj) break;
        const int k = kk < 2 * nu ? kk >> 1 : kk - nu;
        const unsigned tn = __builtin_amdgcn_readlane(ttab, max(k - ni, 0)), sum = tn >> 20;
        char* const f = k < ni ? me + item_flag(__builtin_amdgcn_readlane(qtab, min(k, 59)), far)
                      : sum ? P.ws[(pair0 + sum) & 7] + OFF_TF + 4 + (((rank >> 2) * 4 + ((rank >> 1) & 1)) * 16 + (tn & 15u)) * 96u + rh * 32u
                            : me + OFF_TF + ((tn >> 10) & 1023u) * 32u;
        while (UNI(lds_get(misc + 320 + n)) < 4) __builtin_amdgcn_s_sleep(1);
        if (lane == 0) {
          if (!HALF2) __hip_atomic_fetch_add((unsigned*)f, kk < 2 * nu ? 1u : 2u, __ATOMIC_RELAXED, __HIP_MEMORY_SCOPE_SYSTEM);
          else if (kk >= 2 * nu || (__hip_atomic_fetch_add((unsigned*)(me + OFF_HC) + idx * 128 + k, 1u, __ATOMIC_RELAXED, __HIP_MEMORY_SCOPE_AGENT) & 1))
            __hip_atomic_store((unsigned*)f, 2u * call, __ATOMIC_RELAXED, __HIP_MEMORY_SCOPE_SYSTEM);
        }
      }
    } else if (wv == 7) {
      const unsigned sum = (unsigned)ttab >> 20;
      const unsigned* fin = (const unsigned*)(me + OFF_AF) + far * 64 + qtab;
      const unsigned* ft0 = (const unsigned*)(me + OFF_TF + 4 + (sum ? T_HAND + (sum - 2) * 8 + (ttab & 15) : (ttab >> 10) & 1023) * 32);
      const unsigned* ft1 = (const unsigned*)(me + OFF_TF + (ttab & 1023) * 32);
      u64 seen_in = 0, seen_t = 0;
#pragma unroll 1
      for (int n = 0;; ++n) {
        unsigned kk = (busy ? 16 * NP : 0) + n * 8 + xcd;
        if (n >= 2) {
          while ((int)UNI(__hip_atomic_load(misc + 17, __ATOMIC_RELAXED, __HIP_MEMORY_SCOPE_WORKGROUP)) < 4 * (n - 1) - SLACK) __builtin_amdgcn_s_sleep(1);
          if (lane == 0) kk = __hip_atomic_fetch_add((unsigned*)(me + OFF_LU + idx * 128), 1u, __ATOMIC_RELAXED, __HIP_MEMORY_SCOPE_AGENT);
          kk = UNI(kk) + (busy ? 16u * NP + 16u : 16u) - (call - 1u) * (nj - (busy ? 16 * NP + 8 : 8));
        }
        if ((int)kk < nj) {
          const int k = (int)kk < 2 * nu ? kk >> 1 : kk - nu;
          const u64 t0 = __builtin_amdgcn_s_memrealtime();
          while (!((k < ni ? seen_in >> k : seen_t >> (k - ni)) & 1)) {
            const unsigned a = __hip_atomic_load(k < ni ? fin : ft0, __ATOMIC_RELAXED, __HIP_MEMORY_SCOPE_SYSTEM);
            unsigned c = a;
            if (!busy && sum) c = __hip_atomic_load(ft1, __ATOMIC_RELAXED, __HIP_MEMORY_SCOPE_SYSTEM);
            const u64 ok = __ballot((int)(a - 2u * call) >= 0 && (int)(c - 2u * call) >= 0);
            if (k < ni) seen_in = ok; else seen_t = ok;
            if ((ok >> (k < ni ? k : k - ni)) & 1) break;
            if (expired(t0, tmo, call, lane)) break;
            __builtin_amdgcn_s_sleep(SLP);
          }
        }
        if (lane == 0) __hip_atomic_store(jobs + n, kk + 1u, __ATOMIC_RELAXED, __HIP_MEMORY_SCOPE_WORKGROUP);
        if ((int)kk >= nj) break;
      }
    }
  } else if (t >= PT0 && t < PT0 + 16) {
    f32x4 va[8], vb[8];
    u32x4 fa[4], fb[4];
    const int e = (t - PT0) * 8 + xcd, q = e >> 1, hf = e & 1;
    const int fd = q < 32 ? hf * 4 + (lane & 3) : hf * 2 + (lane & 2) * 2 + (lane & 1);
    u64 fa4 = (u64)P.ws[fd] + OFF_AF + (rank * 64 + q) * 4;
    draw_call();
    const bool soon = q < 32 ? !(q & 2) || q >> 2 == (rank ^ 1) : q < 44;
    while (__builtin_amdgcn_s_memrealtime() - tstart < (soon ? LDELAY : LDELAY2)) __builtin_amdgcn_s_sleep(2);
    load_item(va, q, hf * 6, 0);
    load_item(vb, q, hf * 6, 1);
    convert_item(fa, va, q);
    convert_item(fb, vb, q);
    const unsigned sb = item_base(q, 8 + hf * 2);
#pragma unroll
    for (int u = 0; u < 4; ++u) {
      st128(fa[u], rme, lane * 16, sb + frag_off(q, u));
      st128(fb[u], rme, lane * 16, sb + (1u << 20) + frag_off(q, u));
    }
    WAIT_V(0);
    BAR;
    call = UNI(misc[16]);
    if (tid < 4) __hip_atomic_store((gu32*)fa4, 2u * call, __ATOMIC_RELAXED, __HIP_MEMORY_SCOPE_SYSTEM);
    if (tid < 2) __builtin_amdgcn_raw_buffer_store_b32(2u * call, rme, tid * 8, item_flag(q, 8 + hf * 2), AUX_SYS);
  } else {
    if (WARM & 1 && t == (PT0 ? 1 : 16)) {
      const char* c = (const char*)pc0 + tid * 128;
      if (tid < CODE_LINES) __builtin_amdgcn_global_load_lds((const unsigned*)c, (unsigned*)(lds + wv * 256), 4, 0, 0);
    }
    draw_call();
    BAR;
    call = UNI(misc[16]);
  }
  STAMP_AT(1);

  f32x4 acc[4][4];
  const int ks = wv >> 2, wm = (wv >> 1) & 1, wn = wv & 1;
  const bool has_brick = t >= 14 || t == 12 + (~xcd & 1);
  const int e = (t >= 14 ? t - 14 : 11) * 8 + xcd;
  const bool second = t == 0 || (t < 14 && orel >= 2 && !((xcd ^ orel) & 1));
  const char* lbw = lds + 16384 + ks * 8192 + wn * 4096 + lane * 16;
  const char* lbx = lds + (64 * wm + l15) * 128 + (((4 * ks + q4) ^ (lane & 7)) * 16);
  const int job1 = idx < 7 ? 0 : second ? 3 : 2;
  if (t >= 0) {
    const unsigned hold = has_brick ? BDELAY : !second && t != 7 ? ODELAY : 0;
    while (__builtin_amdgcn_s_memrealtime() - tstart < hold) __builtin_amdgcn_s_sleep(8);
  }
#pragma unroll 1
  for (int job = has_brick ? 0 : 1; job < job1;) {
    const bool brick = job == 0;
    const int owner = (pair0 + orel) & 7, rb = 2 * owner + ((xcd ^ orel) & 1);
    const int quad = 1 + (e >> 5);
    const int trb = brick ? 2 * rank + (e & 1) : rb, tcb = brick ? (e >> 1) & 15 : cb;
    const int blk = brick ? (rank >> 1) ^ quad : rank >> 1;
    unsigned todo = brick ? 0xfu : orel >= 2 ? 0xffffu & ~(3u << 2 * orel) : 0xffffu;
    if (!brick) STAMP_AT(9);
#pragma unroll
    for (int mi = 0; mi < 4; ++mi)
#pragma unroll
      for (int u = 0; u < 4; ++u) acc[mi][u] = f32x4{0.f, 0.f, 0.f, 0.f};
    const unsigned sw = l15 < 4 ? 16 + (lane < 32 ? blk & 2 : (blk & 1) * 2) * 2 + l15 : (2 * pair0 + lane) & 15;
    const char* fp = me + (lane < 32 ? OFF_XF + trb * 128 : OFF_WF + tcb * 128) + sw * 4;
    const unsigned tab = (sw >> 1) * (16u * 65536u) + (sw & 1u) * 32768u;
    const char* gx = me + OFF_XS + trb * 65536 + wv * 2048 + lane * 16;
    const char* gw = me + OFF_WS + tcb * 65536 + wv * 2048 + lane * 16;
    unsigned ox = 0, ow = 0;
    auto stage1 = [&](int buf, int part) INL {
      char* l = lds + buf * STAGE_B + wv * 2048;
      const char* g = (part & 2) ? gw + ow : gx + ox;
      __builtin_amdgcn_global_load_lds((const unsigned*)(g + (buf & 1) * 16384 + (part & 1) * 1024),
                                       (unsigned*)(l + (part >> 1) * 16384 + (part & 1) * 1024), 16, 0, AUX_SYS);
    };
    bf16x8 xa[4] = {}, wa[4] = {}, xb[4] = {}, wb[4] = {};
    auto step = [&](bf16x8* xc, bf16x8* wc, bf16x8* xn, bf16x8* wx, int buf, bool full, bool mf, bool rd, bool ld) INL {
      const int bo = ((buf + 1) & 3) * STAGE_B;
      if (full) {
#pragma unroll
        for (int mi = 0; mi < 4; ++mi) {
          acc[mi][0] = __builtin_amdgcn_mfma_f32_16x16x32_bf16(wc[0], xc[mi], acc[mi][0], 0, 0, 0);
          wx[mi] = *(const bf16x8*)(lbw + bo + mi * 1024);
          xn[mi] = *(const bf16x8*)(lbx + bo + mi * 2048);
          SB;
#pragma unroll
          for (int u = 1; u < 4; ++u)
            acc[mi][u] = __builtin_amdgcn_mfma_f32_16x16x32_bf16(wc[u], xc[mi], acc[mi][u], 0, 0, 0);
          stage1(buf, mi);
          SB;
        }
      } else {
        if (ld) {
#pragma unroll
          for (int part = 0; part < 4; ++part) stage1(buf, part);
        }
        if (rd) {
#pragma unroll
          for (int mi = 0; mi < 4; ++mi) {
            wx[mi] = *(const bf16x8*)(lbw + bo + mi * 1024);
            xn[mi] = *(const bf16x8*)(lbx + bo + mi * 2048);
          }
        }
        SB;
        if (mf) {
#pragma unroll
          for (int mi = 0; mi < 4; ++mi)
#pragma unroll
            for (int u = 0; u < 4; ++u)
              acc[mi][u] = __builtin_amdgcn_mfma_f32_16x16x32_bf16(wc[u], xc[mi], acc[mi][u], 0, 0, 0);
        }
      }
    };
    __builtin_amdgcn_s_waitcnt(0x0F70);
    if (wv == PWV) __builtin_amdgcn_global_load_lds((const unsigned*)fp, misc + 384 + 64, 4, 0, AUX_SYS);
    const u64 t0 = __builtin_amdgcn_s_memrealtime();
    unsigned ready = 0;
    bool h0 = false, h1 = false;
    int pb = 0;
    auto round = [&](bool full) INL {
      const unsigned avail = ready & todo;
      const bool h2 = full || avail != 0;
      if (h2) {
        const int sl = __builtin_ctz(avail);
        todo &= ~(1u << sl);
        ox = __builtin_amdgcn_readlane(tab, sl);
        ow = __builtin_amdgcn_readlane(tab, sl + 32);
      } else if (!h0 && !h1 && wv == PWV) {
        if (expired(t0, tmo, call, lane) && lane == 0) misc[24] = 0xffffu;
        __builtin_amdgcn_s_sleep(SLP);
      }
      if (full || h1) WAIT_V(8); else WAIT_V(0);
      BAR;
      const bool poll = wv == PWV && (!full || (__popc(ready & todo) <= PTH && (todo & ~ready)));
      if (poll) __builtin_amdgcn_global_load_lds((const unsigned*)fp, misc + 384 + pb * 32, 4, 0, AUX_SYS);
      const unsigned fl = misc[384 + (pb ^ 2) * 32 + lane], force = full ? 0u : misc[24];
      step(xa, wa, xb, wb, pb, full, h0, h0, h2);
      if (!h2) WAIT_V(4); else if (poll) WAIT_V(9); else WAIT_V(8);
      BAR;
      const u64 ok = __ballot((int)(fl - 2u * call) >= 0);
      ready |= ((unsigned)ok & (unsigned)(ok >> 32)) | UNI(force);
      step(xb, wb, xa, wa, pb + 1, full, h0, h1, h2);
      h0 = h1; h1 = h2; pb ^= 2;
    };
#pragma unroll 1
    do {
      if (h0 && h1) {
#pragma unroll 1
        while (ready & todo) round(true);
      }
      round(false);
    } while (todo | h0 | h1);

    f32x4 fin[2][4];
    {
      char* xb0 = lds + lane * 16;
      char* lo = xb0 + wv * 8192;
      const char* li = xb0 + (wv ^ 4) * 8192;
#pragma unroll
      for (int mi = 0; mi < 2; ++mi)
#pragma unroll
        for (int u = 0; u < 4; ++u) *(f32x4*)(lo + (mi * 4 + u) * 1024) = ks ? acc[mi][u] : acc[2 + mi][u];
      BAR;
#pragma unroll
      for (int mi = 0; mi < 2; ++mi)
#pragma unroll
        for (int u = 0; u < 4; ++u) fin[mi][u] = (ks ? acc[2 + mi][u] : acc[mi][u]) + *(const f32x4*)(li + (mi * 4 + u) * 1024);
    }
    if (!brick) STAMP_AT(3);
    const int pos = (((blk >> 1) * 2 + (trb & 1)) * 2 + (blk & 1)) * 16 + tcb;
    const int ti = brick ? pos * 3 + 2 : orel == rh ? pos * 3 + rh : T_PARK + orel * 32 + (rb & 1) * 16 + cb;
    const unsigned sp = OFF_TS + (unsigned)ti * 32768u + (unsigned)(((4 * wm + 2 * ks) * 2 + wn) * 2048);
#pragma unroll
    for (int mi = 0; mi < 2; ++mi)
#pragma unroll
      for (int s = 0; s < 2; ++s) {
        const u32x4 o = {pack2(fin[mi][0][2 * s], fin[mi][1][2 * s]), pack2(fin[mi][2][2 * s], fin[mi][3][2 * s]),
                         pack2(fin[mi][0][2 * s + 1], fin[mi][1][2 * s + 1]),
                         pack2(fin[mi][2][2 * s + 1], fin[mi][3][2 * s + 1])};
        st128(o, rme, lane * 16, sp + mi * 4096 + s * 1024);
      }
    WAIT_V(0);
    BAR;
    {
      const bool early = !brick && orel >= 2 && !(rb & 1) && cb < EC, give = early && giver(orel, cb) == rh;
      const bool here = brick || orel == rh || (early && !give);
      const auto rf = rsrc_of(uniform_ptr(P.ws[here ? rank : give ? rank ^ 1 : owner]), WS_NEED);
      const unsigned fo = here ? ti * 32 : 4 + (give ? T_HAND + (orel - 2) * 8 + cb : pos * 3 + rh) * 32;
      if (tid == 0) __builtin_amdgcn_raw_buffer_store_b32(2u * call, rf, fo, OFF_TF, AUX_SYS);
    }
    if (!brick) STAMP_AT(4);
    if (++job == 2) {
      STAMP_AT(12);
      if (t >= 1 && t < 7) cb = 12 + (xcd >> 1);
      else { cb = 4 * (t == 0 ? 3 : (orel - 2) >> 1) + (xcd >> 1); orel = rh; }
    }
  }
  STAMP_AT(5);

  float y[16];
  {
    const int sblk = (rpos >> 6) * 2 + ((rpos >> 4) & 1);
    const bool foreign = sblk != (rank >> 1);
    unsigned mask = foreign ? 7u : 3u;
    if (foreign && !((rpos >> 5) & 1) && (rpos & 15) < EC) mask = 4u | (2u >> giver((rank - 2 * sblk) & 7, rpos & 15));
    const unsigned* tf = (const unsigned*)(me + OFF_TF + rpos * 96 + (lane & 3) * 32);
    if (wv == 6 && lane < 8) *(u64*)(misc + 32 + 2 * lane) = (u64)(P.ws[lane] + OFF_AG) + rank * 16;
    u32x4 p[3][2] = {};
    if (idx < 7) {
      mask &= ~(1u << rhalf);
      if (wv == 6) {
        wait_words(tf + 1, lane == rhalf, 2u * call, tmo, call, lane);
        lds_max(misc + 25, 1u);
        do {
          unsigned v = 2u * call;
          if (lane < 3 && (mask >> lane & 1)) v = __hip_atomic_load(tf, __ATOMIC_RELAXED, __HIP_MEMORY_SCOPE_SYSTEM);
          if (__all((int)(v - 2u * call) >= 0)) { lds_max(misc + 27, 1u); break; }
        } while (!UNI(lds_get(misc + 26)));
      }
      while (!(UNI(lds_get(misc + 25)) && UNI(lds_get(misc + 26)))) __builtin_amdgcn_s_sleep(1);
      const unsigned sr = OFF_TS + (T_PARK + ((rank - 2 * sblk) & 7) * 32 + ((rpos >> 5) & 1) * 16 + (rpos & 15)) * 32768u + rhalf * 16384 + wv * 2048;
      const auto rr = rsrc_of(uniform_ptr(P.ws[2 * sblk + rhalf]), WS_NEED);
      p[0][0] = __builtin_amdgcn_raw_buffer_load_b128(rr, lane * 16, sr, AUX_SYS);
      if (UNI(lds_get(misc + 26)) == 1) WAIT_V(0);
      p[0][1] = __builtin_amdgcn_raw_buffer_load_b128(rr, lane * 16, sr + 1024, AUX_SYS);
    }
    if (wv == 6 && !UNI(lds_get(misc + 27))) {
      wait_words(tf, lane < 3 && (mask >> lane & 1), 2u * call, tmo, call, lane);
      lds_max(misc + 27, 1u);
    }
    if (idx < 7) {
      while (!UNI(lds_get(misc + 27))) __builtin_amdgcn_s_sleep(1);
    } else {
      BAR;
    }
    STAMP_AT(6);
    const unsigned so = OFF_TS + rpos * 3 * 32768 + rhalf * 16384 + wv * 2048;
    if (idx >= 7) {
      const unsigned vo = (mask & 1) ? lane * 16 : 0x7ffffff0u;
      p[0][0] = __builtin_amdgcn_raw_buffer_load_b128(rme, vo, so, AUX_SYS);
      p[0][1] = __builtin_amdgcn_raw_buffer_load_b128(rme, vo, so + 1024, AUX_SYS);
    }
#pragma unroll
    for (int t = 1; t < 3; ++t) {
      const int sl = t == 1 && idx < 7 ? rhalf ^ 1 : t;
      const unsigned vo = (mask >> sl & 1) ? lane * 16 : 0x7ffffff0u;
      p[t][0] = __builtin_amdgcn_raw_buffer_load_b128(rme, vo, so + sl * 32768, AUX_SYS);
      p[t][1] = __builtin_amdgcn_raw_buffer_load_b128(rme, vo, so + sl * 32768 + 1024, AUX_SYS);
    }
    const u64 known = __hip_atomic_load((const u64*)(me + OFF_AG + xcd * 128 + (lane & 7) * 16), __ATOMIC_RELAXED, __HIP_MEMORY_SCOPE_SYSTEM);
#pragma unroll
    for (int s = 0; s < 2; ++s)
#pragma unroll
      for (int d = 0; d < 4; ++d) {
        y[8 * s + 2 * d] = bf_lo(p[0][s][d]) + bf_lo(p[1][s][d]) + bf_lo(p[2][s][d]);
        y[8 * s + 2 * d + 1] = bf_hi(p[0][s][d]) + bf_hi(p[1][s][d]) + bf_hi(p[2][s][d]);
      }
    unsigned am = 0;
#pragma unroll
    for (int t = 0; t < 16; ++t) am = max(am, __builtin_bit_cast(unsigned, y[t]) & 0x7fffffffu);
    am = wave_max(am);
    STAMP_AT(10);
    unsigned tk = 0;
    if (lane == 0) {
      __hip_atomic_fetch_max(misc + 20, am, __ATOMIC_RELAXED, __HIP_MEMORY_SCOPE_WORKGROUP);
      tk = __hip_atomic_fetch_add(misc + 19, 1u, __ATOMIC_RELAXED, __HIP_MEMORY_SCOPE_WORKGROUP);
    }
    if (UNI(tk) == 7) {
      const unsigned wmax = UNI(__hip_atomic_load(misc + 20, __ATOMIC_RELAXED, __HIP_MEMORY_SCOPE_WORKGROUP));
      const u64 entry = *(const u64*)(misc + 32 + 2 * (lane & 7)) + (lane >> 3) * 128;
      if (!__any((unsigned)(known >> 32) == call && (unsigned)known >= wmax)) {
        __hip_atomic_fetch_max((gu64*)entry, ((u64)call << 32) | wmax, __ATOMIC_RELAXED, __HIP_MEMORY_SCOPE_SYSTEM);
        WAIT_V(0);
      }
      unsigned n = 0;
      if (lane == 0) n = __hip_atomic_fetch_add((unsigned*)(me + OFF_ACT), 1u, __ATOMIC_RELAXED, __HIP_MEMORY_SCOPE_AGENT);
      if (UNI(n) + 1u == call * NWG) __hip_atomic_store((gu32*)(entry + 8), call, __ATOMIC_RELAXED, __HIP_MEMORY_SCOPE_SYSTEM);
    }
  }
  {
    const auto ro = rsrc_of(P.out, 512u * NN);
    const unsigned oo = (unsigned)((128 * (rpos >> 5) + 16 * (rhalf * 4 + (wv >> 1)) + l15) * NN + 128 * (rpos & 31) + 64 * (wv & 1) + 16 * q4);
    const u64 t0 = __builtin_amdgcn_s_memrealtime();
    unsigned used = 0xffffffffu;
    for (int ph = 0;; ph ^= 2) {
      if (wv == 0) {
        unsigned cur, fin;
        for (;;) {
          asm volatile("" ::: "memory");
          const u32x4 g = __builtin_amdgcn_raw_buffer_load_b128(rme, (lane & 7) * 16, OFF_AG + xcd * 128, AUX_SYS);
          cur = wave_max(g[1] == call ? g[0] : 0u);
          fin = __all(g[2] == call) || expired(t0, tmo, call, lane);
          if (fin || (cur != used)) break;
          __builtin_amdgcn_s_sleep(1);
        }
        if (lane == 0) { misc[60 + ph] = cur; misc[61 + ph] = fin; }
      }
      BAR;
      const unsigned cur = UNI(misc[60 + ph]), fin = UNI(misc[61 + ph]);
      if (cur != used) {
        const float inv = 448.0f / __builtin_bit_cast(float, cur);
        u32x4 o;
#pragma unroll
        for (int d = 0; d < 4; ++d) {
          int r = __builtin_amdgcn_cvt_pk_fp8_f32(y[4 * d] * inv, y[4 * d + 1] * inv, 0, false);
          r = __builtin_amdgcn_cvt_pk_fp8_f32(y[4 * d + 2] * inv, y[4 * d + 3] * inv, r, true);
          o[d] = (unsigned)r;
        }
        WAIT_V(0);
        __builtin_amdgcn_raw_buffer_store_b128(o, ro, oo, 0, OAUX);
        used = cur;
      }
      if (fin) break;
    }
    STAMP_AT(7);
    if (__builtin_amdgcn_s_memrealtime() - tstart > TMO_LOOK &&
        __hip_atomic_load(tmo, __ATOMIC_RELAXED, __HIP_MEMORY_SCOPE_AGENT) == call) {
      WAIT_V(0);
      __builtin_amdgcn_raw_buffer_store_b128(u32x4{0x7f7f7f7fu, 0x7f7f7f7fu, 0x7f7f7f7fu, 0x7f7f7f7fu}, ro, oo, 0, OAUX);
    }
  }
  STAMP_AT(8);
}

}

extern "C" void kernel_launch(int rank, int nranks, ncclComm_t comm, hipStream_t stream,
                              void* const* d_in, const int* in_sizes, int n_in, void* d_out, int out_size,
                              const int* in_dtypes, int out_dtype, void* const* d_out_all,
                              void* const* d_ws_all, size_t ws_bytes) {
  (void)comm; (void)in_dtypes; (void)out_dtype; (void)d_out_all;
  if (nranks != NR || (unsigned)rank >= (unsigned)NR || n_in < 2 || ws_bytes < WS_NEED ||
      in_sizes[0] != 4096 * KS || in_sizes[1] != KS * NN || out_size != 512 * NN) {
    fprintf(stderr, "unexpected problem shape\n");
    return;
  }
  Params p;
  p.x = (const float*)d_in[0];
  p.w = (const float*)d_in[1];
  p.out = (unsigned char*)d_out;
  for (int r = 0; r < NR; ++r) p.ws[r] = (char*)d_ws_all[r];
  p.rank = rank;
  p.me = p.ws[rank];
  gemm_rs_kernel<<<NWG, NTHR, 0, stream>>>(p);
  HIP_CHECK(hipGetLastError());
}
```

```cpp
#include <hip/hip_runtime.h>
#include <rccl/rccl.h>
#include <cstdio>

#define HIP_CHECK(expr) do { hipError_t _e = (expr); if (_e != hipSuccess) { \
    fprintf(stderr, "HIP error %s at %s:%d\n", hipGetErrorString(_e), __FILE__, __LINE__); \
    return; } } while (0)

namespace {

constexpr int NR = 8;
constexpr int NN = 4096;
constexpr int KS = 512;
constexpr int NWG = 256;
constexpr int NTHR = 512;

#define EC 4
constexpr int LDELAY = 80;
constexpr int LDELAY2 = 250;
constexpr int LATEF = 4;
constexpr int CODE_LINES = 144;
constexpr int SLP = 2;

constexpr unsigned OFF_CALL = 0;
constexpr unsigned OFF_TMO = 32768;
constexpr unsigned OFF_LU = OFF_TMO + 128;
constexpr unsigned OFF_ACT = OFF_TMO + 1024;
constexpr unsigned OFF_XF = OFF_TMO + 4096;
constexpr unsigned OFF_WF = OFF_XF + 2048;
constexpr unsigned OFF_AG = OFF_TMO + 8192;
[[maybe_unused]] constexpr unsigned OFF_DBG = 65536;
constexpr unsigned OFF_AF = OFF_TMO + 2048;
constexpr unsigned OFF_TF = 131072;
constexpr unsigned OFF_XS = 1u << 20;
constexpr unsigned OFF_WS = OFF_XS + (12u << 20);
constexpr unsigned OFF_TS = OFF_WS + (12u << 20);
constexpr int T_HAND = 384;
constexpr int T_PARK = T_HAND + 48;
constexpr unsigned WS_NEED = OFF_TS + (T_PARK + 256) * 32768u;

constexpr unsigned long long DEADLINE = 100000000ull;
constexpr unsigned long long TMO_LOOK = 20000;
constexpr int AUX_SYS = 17;

constexpr int STAGE_B = 32768;
constexpr int LDS_MISC = 4 * STAGE_B;
constexpr int LDS_B = LDS_MISC + 2048;

typedef __bf16 bf16x8 __attribute__((ext_vector_type(8)));
typedef __bf16 bf16x2 __attribute__((ext_vector_type(2)));
typedef float f32x2 __attribute__((ext_vector_type(2)));
typedef float f32x4 __attribute__((ext_vector_type(4)));
typedef unsigned u32x4 __attribute__((ext_vector_type(4)));
typedef unsigned long long u64;
typedef __attribute__((address_space(1))) unsigned long long gu64;
typedef __attribute__((address_space(1))) unsigned gu32;

struct Params {
  const float* x;
  const float* w;
  char* me;
  long rank;
  unsigned char* out;
  char* ws[NR];
};

__device__ __forceinline__ unsigned pack2(float lo, float hi) {
  f32x2 v = {lo, hi};
  return __builtin_bit_cast(unsigned, __builtin_convertvector(v, bf16x2));
}
__device__ __forceinline__ float bf_lo(unsigned u) { return __builtin_bit_cast(float, u << 16); }
__device__ __forceinline__ float bf_hi(unsigned u) { return __builtin_bit_cast(float, u & 0xffff0000u); }

__device__ __forceinline__ __amdgpu_buffer_rsrc_t rsrc_of(const void* p, unsigned bytes) {
  return __builtin_amdgcn_make_buffer_rsrc(const_cast<void*>(p), 0, bytes, 0x00020000);
}
__device__ __forceinline__ void st128(u32x4 v, __amdgpu_buffer_rsrc_t r, unsigned vo, unsigned so) {
  __builtin_amdgcn_raw_buffer_store_b128(v, r, vo, so, AUX_SYS);
  asm volatile("s_nop 1" :: "v"(v) : "memory");
}
__device__ __forceinline__ unsigned lds_addr(const void* p) { return (unsigned)(size_t)(const __attribute__((address_space(3))) char*)p; }
__device__ __forceinline__ unsigned lds_get(const unsigned* p) {
  unsigned r;
  asm volatile("ds_read_b32 %0, %1\n\ts_waitcnt lgkmcnt(0)" : "=&v"(r) : "v"(lds_addr(p)) : "memory");
  return r;
}
__device__ __forceinline__ unsigned lds_inc(unsigned* p) {
  unsigned r;
  asm volatile("ds_add_rtn_u32 %0, %1, %2\n\ts_waitcnt lgkmcnt(0)" : "=&v"(r) : "v"(lds_addr(p)), "v"(1u) : "memory");
  return r;
}
__device__ __forceinline__ void lds_max(unsigned* p, unsigned v) {
  asm volatile("ds_max_u32 %0, %1\n\ts_waitcnt lgkmcnt(0)" :: "v"(lds_addr(p)), "v"(v) : "memory");
}
__device__ __forceinline__ unsigned wave_max(unsigned v) {
  v = max(v, (unsigned)__builtin_amdgcn_mov_dpp((int)v, 0xB1, 0xf, 0xf, true));
  v = max(v, (unsigned)__builtin_amdgcn_mov_dpp((int)v, 0x4E, 0xf, 0xf, true));
  v = max(v, (unsigned)__builtin_amdgcn_mov_dpp((int)v, 0x141, 0xf, 0xf, true));
  v = max(v, (unsigned)__builtin_amdgcn_mov_dpp((int)v, 0x140, 0xf, 0xf, true));
  const unsigned a = __builtin_amdgcn_readlane(v, 0), b = __builtin_amdgcn_readlane(v, 16);
  const unsigned c = __builtin_amdgcn_readlane(v, 32), d = __builtin_amdgcn_readlane(v, 48);
  return max(max(a, b), max(c, d));
}
__device__ __forceinline__ char* uniform_ptr(char* p) {
  const u64 a = (u64)p;
  const unsigned lo = __builtin_amdgcn_readfirstlane((unsigned)a), hi = __builtin_amdgcn_readfirstlane((unsigned)(a >> 32));
  return (char*)(((u64)hi << 32) | lo);
}
#define INL __attribute__((always_inline))
#define UNI(x) __builtin_amdgcn_readfirstlane(x)
#define WAIT_V_(n) asm volatile("s_waitcnt vmcnt(" #n ")" ::: "memory")
#define WAIT_V(n) WAIT_V_(n)
#define SB __builtin_amdgcn_sched_barrier(0)
#define BAR do { asm volatile("" ::: "memory"); __builtin_amdgcn_s_waitcnt(0xC07F);   \
                 __builtin_amdgcn_s_barrier(); asm volatile("" ::: "memory"); SB; } while (0)

#define STAMP_AT(k) do { } while (0)

__device__ __forceinline__ bool expired(u64 t0, unsigned* tmo, unsigned call, int lane) {
  const u64 dt = __builtin_amdgcn_s_memrealtime() - t0;
  if (dt > TMO_LOOK) {
    if (UNI(__hip_atomic_load(tmo, __ATOMIC_RELAXED, __HIP_MEMORY_SCOPE_AGENT)) == call) return true;
    if (dt > DEADLINE) {
      if (lane == 0) __hip_atomic_store(tmo, call, __ATOMIC_RELAXED, __HIP_MEMORY_SCOPE_AGENT);
      return true;
    }
  }
  return false;
}
__device__ __forceinline__ void wait_words(const unsigned* p, bool need, unsigned want, unsigned* tmo,
                                           unsigned call, int lane) {
  const u64 t0 = __builtin_amdgcn_s_memrealtime();
  for (;;) {
    unsigned v = want;
    if (need) v = __hip_atomic_load(p, __ATOMIC_RELAXED, __HIP_MEMORY_SCOPE_SYSTEM);
    if (__all((int)(v - want) >= 0)) break;
    if (expired(t0, tmo, call, lane)) break;
    __builtin_amdgcn_s_sleep(SLP);
  }
}

__global__ void __launch_bounds__(NTHR, 2) gemm_rs_kernel(Params P) {
  __shared__ __attribute__((aligned(16))) char lds[LDS_B];
  const int tid = threadIdx.x;
  const int lane = tid & 63;
  const int wv = UNI(tid >> 6);
  const int l15 = lane & 15, q4 = lane >> 4;
  u64 pc0;
  asm volatile("s_getpc_b64 %0" : "=s"(pc0));
  char* const px = (char*)P.x;
  char* const pw = (char*)P.w;
  char* const me = P.me;
  const int rank = (int)P.rank;
  asm volatile("" :: "s"(px), "s"(pw), "s"(me), "s"(rank));
  const int rh = rank & 1, pair0 = rank & 6;
  unsigned* const tmo = (unsigned*)(me + OFF_TMO);
  unsigned* const misc = (unsigned*)(lds + LDS_MISC);

  const int b = UNI(blockIdx.x), xcd = b & 7, idx = b >> 3;
  STAMP_AT(0);
  const u64 tstart = __builtin_amdgcn_s_memrealtime();
  const int t = idx - 7;
  int orel, cq;
  if (t < 22) { const int ro = t % 7; orel = ro ? ro + 1 : rh ^ 1; cq = t / 7; }
  else { orel = 2 + 2 * (t - 22) + (~xcd & 1); cq = 3; }
  int cb = 4 * cq + (xcd >> 1);
  int rpos, rhalf;
  {
    const int u = b - 56, far = idx < 6 ? (pair0 + 2 + idx) & 7 : idx == 6 ? rank ^ 1 : rank;
    const bool last = idx < 7 || u >= 192;
    const int blk = last ? far >> 1 : (u >> 2) & 3, ra = last ? xcd & 1 : (u >> 1) & 1, rc = last ? 12 + (xcd >> 1) : u >> 4;
    rpos = (((blk >> 1) * 2 + ra) * 2 + (blk & 1)) * 16 + rc;
    rhalf = (last ? far : u) & 1;
  }
  auto giver = [](int rel, int c) { return ((c >> 2) ^ c ^ (rel >> 1)) & 1; };

  const unsigned lx = (unsigned)((16 * wv + (lane >> 3)) * KS + 4 * ((lane ^ (lane >> 3)) & 7)) * 4u;
  const unsigned lw = (unsigned)((64 * (wv >> 2) + 16 * ((wv >> 1) & 1) + 4 * q4) * NN + 64 * (wv & 1) + 4 * l15) * 4u;
  struct Src { __amdgpu_buffer_rsrc_t rs; unsigned so, s0, s1, s2, vo; };
  auto src_of = [&](int q, int dst, int hh) INL {
    const int blk = (q & 31) >> 1, kb = 256 * hh + 128 * (q & 1);
    const bool isx = !(q & 32);
    Src r;
    r.rs = rsrc_of(uniform_ptr(pw + (isx ? px - pw : 0)), 4096u * KS * 4u);
    r.so = UNI(isx ? (unsigned)((512 * (blk >> 1) + 256 * (dst >> 2) + 128 * (blk & 1)) * KS + kb) * 4u
                   : (unsigned)(kb * NN + 2048 * ((dst >> 1) & 1) + 128 * blk) * 4u);
    r.s0 = UNI(isx ? 128u : NN * 4u);
    r.s1 = UNI(isx ? 8u * KS * 4u : 2u * NN * 4u);
    r.s2 = UNI(isx ? 256u : 32u * NN * 4u);
    r.vo = lw + (isx ? lx - lw : 0u);
    return r;
  };
  auto load1 = [&](const Src& r, int t) INL {
    return __builtin_bit_cast(f32x4, __builtin_amdgcn_raw_buffer_load_b128(r.rs, r.vo, r.so + (t & 1) * r.s0 + ((t >> 1) & 1) * r.s1 + (t >> 2) * r.s2, 2  ));
  };
  auto load_item = [&](f32x4* v, int q, int dst, int hh) INL {
    const Src r = src_of(q, dst, hh);
#pragma unroll
    for (int t = 0; t < 8; ++t) v[t] = load1(r, t);
  };
  auto convert_item = [&](u32x4* f, const f32x4* v, int q) INL {
    if (q < 32) {
#pragma unroll
      for (int u = 0; u < 4; ++u)
        f[u] = u32x4{pack2(v[2 * u][0], v[2 * u][1]), pack2(v[2 * u][2], v[2 * u][3]),
                     pack2(v[2 * u + 1][0], v[2 * u + 1][1]), pack2(v[2 * u + 1][2], v[2 * u + 1][3])};
    } else {
#pragma unroll
      for (int u = 0; u < 4; ++u)
        f[u] = u32x4{pack2(v[0][u], v[1][u]), pack2(v[2][u], v[3][u]), pack2(v[4][u], v[5][u]), pack2(v[6][u], v[7][u])};
    }
#pragma unroll
    for (int u = 0; u < 4; ++u) asm volatile("" : "+v"(f[u]));
  };
  auto item_base = [&](int q, int chunk) -> unsigned {
    const unsigned pb = (q < 32 ? OFF_XS : OFF_WS) + (unsigned)(chunk * 16 + ((q & 31) >> 1)) * 65536u + (q & 1) * 32768u;
    return pb + (q < 32 ? wv * 2048u : wv * 4096u);
  };
  auto frag_off = [](int q, int u) INL { return (u & 1) * 1024u + (u >> 1) * (q < 32 ? 16384u : 2048u); };
  auto item_flag = [&](int q, int chunk) -> unsigned {
    return (q < 32 ? OFF_XF : OFF_WF) + ((q & 31) >> 1) * 128u + chunk * 8u + (q & 1) * 4u;
  };
  auto draw_call = [&]() {
    if (tid != 16) misc[tid] = 0;
    if (tid == NTHR - 64)
      misc[16] = __hip_atomic_fetch_add((unsigned*)(me + OFF_CALL + b * 128), 1u, __ATOMIC_RELAXED,
                                        __HIP_MEMORY_SCOPE_AGENT) + 1u;
  };
  const auto rme = rsrc_of(uniform_ptr(me), WS_NEED);
  unsigned call;
  if (idx < 7) {
    const bool busy = idx < 6;
    const int lrel = busy ? 2 + idx : rh ^ 1, far = (pair0 + lrel) & 7;
    const int ni = busy ? 60 : 0, nu = busy ? 84 - EC / 2 : 24 + 3 * EC;
    const int nj = 2 * nu + 8;
    char* const pfar = uniform_ptr(P.ws[far]);
    unsigned* const jobs = misc + 128;
    const auto rd = rsrc_of(pfar, WS_NEED);
    const int q0 = (xcd & 1 ? 32 : (far ^ 1) * 4) + (xcd & 2) + (xcd >> 2);
    f32x4 v[8];
    load_item(v, q0, far, far & 1);
    asm volatile("" ::: "memory");
    SB;
    int qtab, ttab;
    {
      int k = lane < 52 ? lane : lane < 56 ? lane + 4 : lane - 4;
      const int j = k < 32 ? (k >> 3) * 2 + ((k >> 1) & 1) : k < 40 ? 8 + (k & 3) : (k >> 3) * 4 - 12 + (k & 3);
      const int kh = (k >> 2) & 1;
      const bool isw = k < 32 ? k & 1 : k >= 40;
      const int second_blk = j >= 7, jt = second_blk ? 11 - j : j - 2;
      const int en = jt + (jt + 2 >= lrel);
      const int row = j < 2 ? (rank ^ 1) * 2 + j : ((pair0 + 2 + en) & 7) * 2 + second_blk;
      qtab = k >= 56 ? rank * 4 + (k & 3) : isw ? 32 + j * 2 + kh : row * 2 + kh;
      k = lane;
      const int mrel = (rank - (far & 6)) & 7;
      int to = mrel, ta, tc, sum = 0;
      if (busy) {
        int i = k - (8 - EC);
        const int fpos = (mrel - 2) / 2 * 8;
        const bool one = i >= fpos && i < fpos + EC / 2;
        if (i >= fpos + EC / 2) i -= EC / 2;
        if (i < 0) { ta = 0; tc = 8 + i; }
        else if (one) { ta = 0; tc = 2 * (i - fpos); tc += giver(to, tc) == (far & 1); }
        else if (i < 8) { ta = 1; tc = i; }
        else { ta = i & 1; tc = (4 + (i >> 1)) & 15; }
      } else if (k >= 8 && k < 8 + 3 * EC) {
        to = 2 + (k - 8) / (EC + !EC) * 2 + (k & 1); tc = (k - 8) % (EC + !EC) & ~1;
        ta = 0; tc += giver(to, tc) == rh; sum = to;
      } else {
        const int i = k < 8 ? k : k - 3 * EC;
        ta = i & 1; tc = (i >> 1) & 15;
      }
      const int there = T_PARK + to * 32 + ta * 16 + tc;
      ttab = there | (sum ? there : ((((far >> 2) * 2 + ta) * 2 + ((far >> 1) & 1)) * 16 + tc) * 3 + (far & 1)) << 10 | sum << 20;
    }
    if (busy) {
      u32x4 f[4];
      draw_call();
      convert_item(f, v, q0);
      const unsigned sb = item_base(q0, rank);
#pragma unroll
      for (int u = 0; u < 4; ++u) {
        st128(f[u], rd, lane * 16, sb + frag_off(q0, u));
        if (u == 0) { BAR; call = UNI(misc[16]); }
        if (u == 3 || wv < 7) WAIT_V(0);
      }
      unsigned tk = 0;
      if (lane == 0) tk = __hip_atomic_fetch_add(misc + 28, 1u, __ATOMIC_RELAXED, __HIP_MEMORY_SCOPE_WORKGROUP);
      if (UNI(tk) == 7 && lane == 0) __builtin_amdgcn_raw_buffer_store_b32(2u * call, rd, item_flag(q0, rank), 0, AUX_SYS);
    } else {
      draw_call();
      BAR;
      call = UNI(misc[16]);
    }
    auto job_of = [&](int n) INL { return (int)UNI(lds_get(jobs + n)) - 1; };
    if (wv < 4) {
      char* const ring = lds + wv * 8192;
      unsigned so = 0, dn = 0, dc = 0;
      bool sn = false, sc = false;
      auto decode = [&](int kk) INL {
        const int k = kk < 2 * nu ? kk >> 1 : kk - nu;
        const unsigned wo = ((kk < 2 * nu ? kk : ~far) & 1) * 16384u + wv * 4096u;
        const int q = __builtin_amdgcn_readlane(qtab, min(k, 59));
        const unsigned tn = __builtin_amdgcn_readlane(ttab, max(k - ni, 0));
        const unsigned o = (q < 32 ? OFF_XS : OFF_WS) + (q & 31) * 32768u + wo;
        so = k < ni ? o + (unsigned)(8 + (q < 32 ? rank >> 2 : (rank >> 1) & 1) * 2 + rh) * (1u << 20) : OFF_TS + (tn & 1023u) * 32768u + wo;
        dn = k < ni ? o + (unsigned)far * (1u << 20) : OFF_TS + ((tn >> 10) & 1023u) * 32768u + wo;
        sn = tn >> 20;
      };
      auto read = [&](int u) INL {
        __builtin_amdgcn_global_load_lds((const unsigned*)(pfar + so + u * 1024 + lane * 16), (unsigned*)(ring + u * 1024), 16, 0, AUX_SYS);
        if (!busy) __builtin_amdgcn_global_load_lds((const unsigned*)(me + so + u * 1024 + lane * 16), (unsigned*)(ring + 4096 + u * 1024), 16, 0, AUX_SYS);
      };
      auto write = [&](int u) INL {
        u32x4 x = *(const u32x4*)(ring + u * 1024 + lane * 16);
        if (sc) {
          const u32x4 y = *(const u32x4*)(ring + 4096 + u * 1024 + lane * 16);
#pragma unroll
          for (int d = 0; d < 4; ++d) x[d] = pack2(bf_lo(x[d]) + bf_lo(y[d]), bf_hi(x[d]) + bf_hi(y[d]));
        }
        st128(x, rme, lane * 16, dc + u * 1024);
      };
#define LANDED(nb, np) do { if (busy) WAIT_V(nb); else WAIT_V(np); } while (0)
      bool flow = false;
#pragma unroll 1
      for (int n = 0;; ++n) {
        if (!flow) {
          int kn;
          while ((kn = job_of(n)) < 0) __builtin_amdgcn_s_sleep(1);
          if (kn >= nj) { lds_max(misc + 26, 1u + (kn - nj >= LATEF)); break; }
          decode(kn);
          read(0);
          read(1);
        }
        if (lane == 0) lds_inc(misc + 17);
        dc = dn; sc = sn;
        LANDED(1, 2);
        if (flow && lane == 0) lds_inc(misc + 320 + n - 1);
        write(0); read(2);
        LANDED(2, 3);
        write(1); read(3);
        const int kn = job_of(n + 1);
        flow = kn >= 0 && kn < nj;
        if (flow) decode(kn);
        LANDED(2, 3);
        write(2);
        if (flow) read(0);
        if (flow) LANDED(2, 3); else WAIT_V(1);
        if (kn >= nj) lds_max(misc + 26, 1u + (kn - nj >= LATEF));
        write(3);
        if (flow) read(1);
        else { WAIT_V(0); if (lane == 0) lds_inc(misc + 320 + n); }
      }
    } else if (wv == 5) {
#pragma unroll 1
      for (int n = 0;; ++n) {
        int kk;
        while ((kk = job_of(n)) < 0) __builtin_amdgcn_s_sleep(1);
        if (kk >= nj) break;
        const int k = kk < 2 * nu ? kk >> 1 : kk - nu;
        const unsigned tn = __builtin_amdgcn_readlane(ttab, max(k - ni, 0)), sum = tn >> 20;
        char* const f = k < ni ? me + item_flag(__builtin_amdgcn_readlane(qtab, min(k, 59)), far)
                      : sum ? P.ws[(pair0 + sum) & 7] + OFF_TF + 4 + (((rank >> 2) * 4 + ((rank >> 1) & 1)) * 16 + (tn & 15u)) * 96u + rh * 32u
                            : me + OFF_TF + ((tn >> 10) & 1023u) * 32u;
        while (UNI(lds_get(misc + 320 + n)) < 4) __builtin_amdgcn_s_sleep(1);
        if (lane == 0) __hip_atomic_fetch_add((unsigned*)f, kk < 2 * nu ? 1u : 2u, __ATOMIC_RELAXED, __HIP_MEMORY_SCOPE_SYSTEM);
      }
    } else if (wv == 7) {
      const unsigned sum = (unsigned)ttab >> 20;
      const unsigned* fin = (const unsigned*)(me + OFF_AF) + far * 64 + qtab;
      const unsigned* ft0 = (const unsigned*)(me + OFF_TF + 4 + (sum ? T_HAND + (sum - 2) * 8 + (ttab & 15) : (ttab >> 10) & 1023) * 32);
      const unsigned* ft1 = (const unsigned*)(me + OFF_TF + (ttab & 1023) * 32);
      u64 seen_in = 0, seen_t = 0;
#pragma unroll 1
      for (int n = 0;; ++n) {
        unsigned kk = (busy ? 16 : 0) + n * 8 + xcd;
        if (n >= 2) {
          while ((int)UNI(__hip_atomic_load(misc + 17, __ATOMIC_RELAXED, __HIP_MEMORY_SCOPE_WORKGROUP)) < 4 * (n - 1)) __builtin_amdgcn_s_sleep(1);
          if (lane == 0) kk = __hip_atomic_fetch_add((unsigned*)(me + OFF_LU + idx * 128), 1u, __ATOMIC_RELAXED, __HIP_MEMORY_SCOPE_AGENT);
          kk = min(UNI(kk) + (busy ? 32u : 16u) - (call - 1u) * (nj - (busy ? 24 : 8)), (unsigned)nj + 7u);
        }
        if ((int)kk < nj) {
          const int k = (int)kk < 2 * nu ? kk >> 1 : kk - nu;
          const u64 t0 = __builtin_amdgcn_s_memrealtime();
          while (!((k < ni ? seen_in >> k : seen_t >> (k - ni)) & 1)) {
            const unsigned a = __hip_atomic_load(k < ni ? fin : ft0, __ATOMIC_RELAXED, __HIP_MEMORY_SCOPE_SYSTEM);
            unsigned c = a;
            if (!busy && sum) c = __hip_atomic_load(ft1, __ATOMIC_RELAXED, __HIP_MEMORY_SCOPE_SYSTEM);
            const u64 ok = __ballot((int)(a - 2u * call) >= 0 && (int)(c - 2u * call) >= 0);
            if (k < ni) seen_in = ok; else seen_t = ok;
            if ((ok >> (k < ni ? k : k - ni)) & 1) break;
            if (expired(t0, tmo, call, lane)) break;
            __builtin_amdgcn_s_sleep(SLP);
          }
        }
        if (lane == 0) __hip_atomic_store(jobs + n, kk + 1u, __ATOMIC_RELAXED, __HIP_MEMORY_SCOPE_WORKGROUP);
        if ((int)kk >= nj) break;
      }
    }
  } else if (t >= 9) {
    f32x4 va[8], vb[8];
    u32x4 fa[4], fb[4];
    const int e = (t - 9) * 8 + xcd, q = e >> 1, hf = e & 1;
    const int fd = q < 32 ? hf * 4 + (lane & 3) : hf * 2 + (lane & 2) * 2 + (lane & 1);
    u64 fa4 = (u64)P.ws[fd] + OFF_AF + (rank * 64 + q) * 4;
    draw_call();
    const bool soon = q < 32 ? !(q & 2) || q >> 2 == (rank ^ 1) : q < 44;
    while (__builtin_amdgcn_s_memrealtime() - tstart < (soon ? LDELAY : LDELAY2)) __builtin_amdgcn_s_sleep(2);
    load_item(va, q, hf * 6, 0);
    load_item(vb, q, hf * 6, 1);
    convert_item(fa, va, q);
    convert_item(fb, vb, q);
    const unsigned sb = item_base(q, 8 + hf * 2);
#pragma unroll
    for (int u = 0; u < 4; ++u) {
      st128(fa[u], rme, lane * 16, sb + frag_off(q, u));
      st128(fb[u], rme, lane * 16, sb + (1u << 20) + frag_off(q, u));
    }
    WAIT_V(0);
    BAR;
    call = UNI(misc[16]);
    if (tid < 4) __hip_atomic_store((gu32*)fa4, 2u * call, __ATOMIC_RELAXED, __HIP_MEMORY_SCOPE_SYSTEM);
    if (tid < 2) __builtin_amdgcn_raw_buffer_store_b32(2u * call, rme, tid * 8, item_flag(q, 8 + hf * 2), AUX_SYS);
  } else {
    if (t == 1) {
      const char* c = (const char*)pc0 + tid * 128;
      if (tid < CODE_LINES) __builtin_amdgcn_global_load_lds((const unsigned*)c, (unsigned*)(lds + wv * 256), 4, 0, 0);
    }
    draw_call();
    BAR;
    call = UNI(misc[16]);
  }
  STAMP_AT(1);

  f32x4 acc[4][4];
  const int ks = wv >> 2, wm = (wv >> 1) & 1, wn = wv & 1;
  const bool has_brick = t >= 14 || t == 12 + (~xcd & 1);
  const int e = (t >= 14 ? t - 14 : 11) * 8 + xcd;
  const bool second = t == 0 || (t < 14 && orel >= 2 && !((xcd ^ orel) & 1));
  const char* lbw = lds + 16384 + ks * 8192 + wn * 4096 + lane * 16;
  const char* lbx = lds + (64 * wm + l15) * 128 + (((4 * ks + q4) ^ (lane & 7)) * 16);
  const int job1 = idx < 7 ? 0 : second ? 3 : 2;
#pragma unroll 1
  for (int job = has_brick ? 0 : 1; job < job1;) {
    const bool brick = job == 0;
    const int owner = (pair0 + orel) & 7, rb = 2 * owner + ((xcd ^ orel) & 1);
    const int quad = 1 + (e >> 5);
    const int trb = brick ? 2 * rank + (e & 1) : rb, tcb = brick ? (e >> 1) & 15 : cb;
    const int blk = brick ? (rank >> 1) ^ quad : rank >> 1;
    unsigned todo = brick ? 0xfu : orel >= 2 ? 0xffffu & ~(3u << 2 * orel) : 0xffffu;
    if (!brick) STAMP_AT(9);
#pragma unroll
    for (int mi = 0; mi < 4; ++mi)
#pragma unroll
      for (int u = 0; u < 4; ++u) acc[mi][u] = f32x4{0.f, 0.f, 0.f, 0.f};
    const unsigned sw = l15 < 4 ? 16 + (lane < 32 ? blk & 2 : (blk & 1) * 2) * 2 + l15 : (2 * pair0 + lane) & 15;
    const char* fp = me + (lane < 32 ? OFF_XF + trb * 128 : OFF_WF + tcb * 128) + sw * 4;
    const unsigned tab = (sw >> 1) * (16u * 65536u) + (sw & 1u) * 32768u;
    const char* gx = me + OFF_XS + trb * 65536 + wv * 2048 + lane * 16;
    const char* gw = me + OFF_WS + tcb * 65536 + wv * 2048 + lane * 16;
    unsigned ox = 0, ow = 0;
    auto stage1 = [&](int buf, int part) INL {
      char* l = lds + buf * STAGE_B + wv * 2048;
      const char* g = (part & 2) ? gw + ow : gx + ox;
      __builtin_amdgcn_global_load_lds((const unsigned*)(g + (buf & 1) * 16384 + (part & 1) * 1024),
                                       (unsigned*)(l + (part >> 1) * 16384 + (part & 1) * 1024), 16, 0, AUX_SYS);
    };
    bf16x8 xa[4] = {}, wa[4] = {}, xb[4] = {}, wb[4] = {};
    auto step = [&](bf16x8* xc, bf16x8* wc, bf16x8* xn, bf16x8* wx, int buf, bool full, bool mf, bool rd, bool ld) INL {
      const int bo = ((buf + 1) & 3) * STAGE_B;
      if (full) {
#pragma unroll
        for (int mi = 0; mi < 4; ++mi) {
          acc[mi][0] = __builtin_amdgcn_mfma_f32_16x16x32_bf16(wc[0], xc[mi], acc[mi][0], 0, 0, 0);
          wx[mi] = *(const bf16x8*)(lbw + bo + mi * 1024);
          xn[mi] = *(const bf16x8*)(lbx + bo + mi * 2048);
          SB;
#pragma unroll
          for (int u = 1; u < 4; ++u)
            acc[mi][u] = __builtin_amdgcn_mfma_f32_16x16x32_bf16(wc[u], xc[mi], acc[mi][u], 0, 0, 0);
          stage1(buf, mi);
          SB;
        }
      } else {
        if (ld) {
#pragma unroll
          for (int part = 0; part < 4; ++part) stage1(buf, part);
        }
        if (rd) {
#pragma unroll
          for (int mi = 0; mi < 4; ++mi) {
            wx[mi] = *(const bf16x8*)(lbw + bo + mi * 1024);
            xn[mi] = *(const bf16x8*)(lbx + bo + mi * 2048);
          }
        }
        SB;
        if (mf) {
#pragma unroll
          for (int mi = 0; mi < 4; ++mi)
#pragma unroll
            for (int u = 0; u < 4; ++u)
              acc[mi][u] = __builtin_amdgcn_mfma_f32_16x16x32_bf16(wc[u], xc[mi], acc[mi][u], 0, 0, 0);
        }
      }
    };
    __builtin_amdgcn_s_waitcnt(0x0F70);
    if (wv == 0) __builtin_amdgcn_global_load_lds((const unsigned*)fp, misc + 384 + 64, 4, 0, AUX_SYS);
    const u64 t0 = __builtin_amdgcn_s_memrealtime();
    unsigned ready = 0;
    bool h0 = false, h1 = false;
    int pb = 0;
    auto round = [&](bool full) INL {
      const unsigned avail = ready & todo;
      const bool h2 = full || avail != 0;
      if (h2) {
        const int sl = __builtin_ctz(avail);
        todo &= ~(1u << sl);
        ox = __builtin_amdgcn_readlane(tab, sl);
        ow = __builtin_amdgcn_readlane(tab, sl + 32);
      } else if (!h0 && !h1 && wv == 0) {
        if (expired(t0, tmo, call, lane) && lane == 0) misc[24] = 0xffffu;
        __builtin_amdgcn_s_sleep(SLP);
      }
      if (full || h1) WAIT_V(8); else WAIT_V(0);
      BAR;
      const bool poll = wv == 0 && (!full || (todo & ~ready));
      if (poll) __builtin_amdgcn_global_load_lds((const unsigned*)fp, misc + 384 + pb * 32, 4, 0, AUX_SYS);
      const unsigned fl = misc[384 + (pb ^ 2) * 32 + lane], force = full ? 0u : misc[24];
      step(xa, wa, xb, wb, pb, full, h0, h0, h2);
      if (!h2) WAIT_V(4); else if (poll) WAIT_V(9); else WAIT_V(8);
      BAR;
      const u64 ok = __ballot((int)(fl - 2u * call) >= 0);
      ready |= ((unsigned)ok & (unsigned)(ok >> 32)) | UNI(force);
      step(xb, wb, xa, wa, pb + 1, full, h0, h1, h2);
      h0 = h1; h1 = h2; pb ^= 2;
    };
#pragma unroll 1
    do {
      if (h0 && h1) {
#pragma unroll 1
        while (ready & todo) round(true);
      }
      round(false);
    } while (todo | h0 | h1);

    f32x4 fin[2][4];
    {
      char* xb0 = lds + lane * 16;
      char* lo = xb0 + wv * 8192;
      const char* li = xb0 + (wv ^ 4) * 8192;
#pragma unroll
      for (int mi = 0; mi < 2; ++mi)
#pragma unroll
        for (int u = 0; u < 4; ++u) *(f32x4*)(lo + (mi * 4 + u) * 1024) = ks ? acc[mi][u] : acc[2 + mi][u];
      BAR;
#pragma unroll
      for (int mi = 0; mi < 2; ++mi)
#pragma unroll
        for (int u = 0; u < 4; ++u) fin[mi][u] = (ks ? acc[2 + mi][u] : acc[mi][u]) + *(const f32x4*)(li + (mi * 4 + u) * 1024);
    }
    if (!brick) STAMP_AT(3);
    const int pos = (((blk >> 1) * 2 + (trb & 1)) * 2 + (blk & 1)) * 16 + tcb;
    const int ti = brick ? pos * 3 + 2 : orel == rh ? pos * 3 + rh : T_PARK + orel * 32 + (rb & 1) * 16 + cb;
    const unsigned sp = OFF_TS + (unsigned)ti * 32768u + (unsigned)(((4 * wm + 2 * ks) * 2 + wn) * 2048);
#pragma unroll
    for (int mi = 0; mi < 2; ++mi)
#pragma unroll
      for (int s = 0; s < 2; ++s) {
        const u32x4 o = {pack2(fin[mi][0][2 * s], fin[mi][1][2 * s]), pack2(fin[mi][2][2 * s], fin[mi][3][2 * s]),
                         pack2(fin[mi][0][2 * s + 1], fin[mi][1][2 * s + 1]),
                         pack2(fin[mi][2][2 * s + 1], fin[mi][3][2 * s + 1])};
        st128(o, rme, lane * 16, sp + mi * 4096 + s * 1024);
      }
    WAIT_V(0);
    BAR;
    {
      const bool early = !brick && orel >= 2 && !(rb & 1) && cb < EC, give = early && giver(orel, cb) == rh;
      const bool here = brick || orel == rh || (early && !give);
      const auto rf = rsrc_of(uniform_ptr(P.ws[here ? rank : give ? rank ^ 1 : owner]), WS_NEED);
      const unsigned fo = here ? ti * 32 : 4 + (give ? T_HAND + (orel - 2) * 8 + cb : pos * 3 + rh) * 32;
      if (tid == 0) __builtin_amdgcn_raw_buffer_store_b32(2u * call, rf, fo, OFF_TF, AUX_SYS);
    }
    if (!brick) STAMP_AT(4);
    if (++job == 2) {
      STAMP_AT(12);
      if (t >= 1 && t < 7) cb = 12 + (xcd >> 1);
      else { cb = 4 * (t == 0 ? 3 : (orel - 2) >> 1) + (xcd >> 1); orel = rh; }
    }
  }
  STAMP_AT(5);

  float y[16];
  {
    const int sblk = (rpos >> 6) * 2 + ((rpos >> 4) & 1);
    const bool foreign = sblk != (rank >> 1);
    unsigned mask = foreign ? 7u : 3u;
    if (foreign && !((rpos >> 5) & 1) && (rpos & 15) < EC) mask = 4u | (2u >> giver((rank - 2 * sblk) & 7, rpos & 15));
    const unsigned* tf = (const unsigned*)(me + OFF_TF + rpos * 96 + (lane & 3) * 32);
    if (wv == 6 && lane < 8) *(u64*)(misc + 32 + 2 * lane) = (u64)(P.ws[lane] + OFF_AG) + rank * 16;
    u32x4 p[4][2] = {};
    if (idx < 7) {
      mask &= ~(1u << rhalf);
      const unsigned sr = OFF_TS + (T_PARK + ((rank - 2 * sblk) & 7) * 32 + ((rpos >> 5) & 1) * 16 + (rpos & 15)) * 32768u + rhalf * 16384 + wv * 2048;
      char* const pr = uniform_ptr(P.ws[2 * sblk + rhalf]);
      asm volatile("" :: "s"(sr), "s"(pr));
      const auto rr = rsrc_of(pr, WS_NEED);
      if (wv == 6) {
        wait_words(tf + 1, lane == rhalf, 2u * call, tmo, call, lane);
        lds_max(misc + 25, 1u);
        do {
          unsigned v = 2u * call;
          if (lane < 3 && (mask >> lane & 1)) v = __hip_atomic_load(tf, __ATOMIC_RELAXED, __HIP_MEMORY_SCOPE_SYSTEM);
          if (__all((int)(v - 2u * call) >= 0)) { lds_max(misc + 27, 1u); break; }
        } while (!UNI(lds_get(misc + 26)));
      }
      unsigned go;
      while (!(UNI(lds_get(misc + 25)) && (go = UNI(lds_get(misc + 26))))) __builtin_amdgcn_s_sleep(1);
      p[3][0] = __builtin_amdgcn_raw_buffer_load_b128(rr, lane * 16, sr, AUX_SYS);
      if (go == 1) WAIT_V(0);
      p[3][1] = __builtin_amdgcn_raw_buffer_load_b128(rr, lane * 16, sr + 1024, AUX_SYS);
    }
    if (wv == 6 && !UNI(lds_get(misc + 27))) {
      wait_words(tf, lane < 3 && (mask >> lane & 1), 2u * call, tmo, call, lane);
      lds_max(misc + 27, 1u);
    }
    if (idx < 7) {
      while (!UNI(lds_get(misc + 27))) __builtin_amdgcn_s_sleep(1);
    } else {
      BAR;
    }
    STAMP_AT(6);
    const u64 known = __hip_atomic_load((const u64*)(me + OFF_AG + xcd * 128 + (lane & 7) * 16), __ATOMIC_RELAXED, __HIP_MEMORY_SCOPE_SYSTEM);
    asm volatile("" ::: "memory");
    SB;
    const unsigned so = OFF_TS + rpos * 3 * 32768 + rhalf * 16384 + wv * 2048;
#pragma unroll
    for (int t = 0; t < 3; ++t) {
      const unsigned vo = (mask >> t & 1) ? lane * 16 : 0x7ffffff0u;
      p[t][0] = __builtin_amdgcn_raw_buffer_load_b128(rme, vo, so + t * 32768, AUX_SYS);
      p[t][1] = __builtin_amdgcn_raw_buffer_load_b128(rme, vo, so + t * 32768 + 1024, AUX_SYS);
    }
    SB;
#pragma unroll
    for (int s = 0; s < 2; ++s)
#pragma unroll
      for (int d = 0; d < 4; ++d) {
        y[8 * s + 2 * d] = (bf_lo(p[0][s][d]) + bf_lo(p[1][s][d])) + (bf_lo(p[2][s][d]) + bf_lo(p[3][s][d]));
        y[8 * s + 2 * d + 1] = (bf_hi(p[0][s][d]) + bf_hi(p[1][s][d])) + (bf_hi(p[2][s][d]) + bf_hi(p[3][s][d]));
      }
    unsigned am = 0;
#pragma unroll
    for (int t = 0; t < 16; ++t) am = max(am, __builtin_bit_cast(unsigned, y[t]) & 0x7fffffffu);
    am = wave_max(am);
    STAMP_AT(10);
    unsigned tk = 0;
    if (lane == 0) {
      __hip_atomic_fetch_max(misc + 20, am, __ATOMIC_RELAXED, __HIP_MEMORY_SCOPE_WORKGROUP);
      tk = __hip_atomic_fetch_add(misc + 19, 1u, __ATOMIC_RELAXED, __HIP_MEMORY_SCOPE_WORKGROUP);
    }
    if (UNI(tk) == 7) {
      const unsigned wmax = UNI(__hip_atomic_load(misc + 20, __ATOMIC_RELAXED, __HIP_MEMORY_SCOPE_WORKGROUP));
      const u64 entry = *(const u64*)(misc + 32 + 2 * (lane & 7)) + (lane >> 3) * 128;
      if (!__any((unsigned)(known >> 32) == call && (unsigned)known >= wmax)) {
        __hip_atomic_fetch_max((gu64*)entry, ((u64)call << 32) | wmax, __ATOMIC_RELAXED, __HIP_MEMORY_SCOPE_SYSTEM);
        WAIT_V(0);
      }
      unsigned n = 0;
      if (lane == 0) n = __hip_atomic_fetch_add((unsigned*)(me + OFF_ACT), 1u, __ATOMIC_RELAXED, __HIP_MEMORY_SCOPE_AGENT);
      if (UNI(n) + 1u == call * NWG) __hip_atomic_store((gu32*)(entry + 8), call, __ATOMIC_RELAXED, __HIP_MEMORY_SCOPE_SYSTEM);
    }
  }
  {
    const auto ro = rsrc_of(P.out, 512u * NN);
    const unsigned oo = (unsigned)((128 * (rpos >> 5) + 16 * (rhalf * 4 + (wv >> 1)) + l15) * NN + 128 * (rpos & 31) + 64 * (wv & 1) + 16 * q4);
    const u64 t0 = __builtin_amdgcn_s_memrealtime();
    unsigned used = 0xffffffffu;
    for (int ph = 0;; ph ^= 2) {
      if (wv == 0) {
        unsigned cur, fin;
        for (;;) {
          asm volatile("" ::: "memory");
          const u32x4 g = __builtin_amdgcn_raw_buffer_load_b128(rme, (lane & 7) * 16, OFF_AG + xcd * 128, AUX_SYS);
          cur = wave_max(g[1] == call ? g[0] : 0u);
          fin = __all(g[2] == call) || expired(t0, tmo, call, lane);
          if (fin || (cur != used)) break;
          __builtin_amdgcn_s_sleep(1);
        }
        if (lane == 0) { misc[60 + ph] = cur; misc[61 + ph] = fin; }
      }
      BAR;
      const unsigned cur = UNI(misc[60 + ph]), fin = UNI(misc[61 + ph]);
      if (cur != used) {
        const float inv = 448.0f / __builtin_bit_cast(float, cur);
        u32x4 o;
#pragma unroll
        for (int d = 0; d < 4; ++d) {
          int r = __builtin_amdgcn_cvt_pk_fp8_f32(y[4 * d] * inv, y[4 * d + 1] * inv, 0, false);
          r = __builtin_amdgcn_cvt_pk_fp8_f32(y[4 * d + 2] * inv, y[4 * d + 3] * inv, r, true);
          o[d] = (unsigned)r;
        }
        WAIT_V(0);
        __builtin_amdgcn_raw_buffer_store_b128(o, ro, oo, 0, AUX_SYS);
        used = cur;
      }
      if (fin) break;
    }
    STAMP_AT(7);
    if (__builtin_amdgcn_s_memrealtime() - tstart > TMO_LOOK &&
        __hip_atomic_load(tmo, __ATOMIC_RELAXED, __HIP_MEMORY_SCOPE_AGENT) == call) {
      WAIT_V(0);
      __builtin_amdgcn_raw_buffer_store_b128(u32x4{0x7f7f7f7fu, 0x7f7f7f7fu, 0x7f7f7f7fu, 0x7f7f7f7fu}, ro, oo, 0, AUX_SYS);
    }
  }
  STAMP_AT(8);
}

}

extern "C" void kernel_launch(int rank, int nranks, ncclComm_t comm, hipStream_t stream,
                              void* const* d_in, const int* in_sizes, int n_in, void* d_out, int out_size,
                              const int* in_dtypes, int out_dtype, void* const* d_out_all,
                              void* const* d_ws_all, size_t ws_bytes) {
  (void)comm; (void)in_dtypes; (void)out_dtype; (void)d_out_all;
  if (nranks != NR || (unsigned)rank >= (unsigned)NR || n_in < 2 || ws_bytes < WS_NEED ||
      in_sizes[0] != 4096 * KS || in_sizes[1] != KS * NN || out_size != 512 * NN) {
    fprintf(stderr, "unexpected problem shape\n");
    return;
  }
  Params p;
  p.x = (const float*)d_in[0];
  p.w = (const float*)d_in[1];
  p.out = (unsigned char*)d_out;
  for (int r = 0; r < NR; ++r) p.ws[r] = (char*)d_ws_all[r];
  p.rank = rank;
  p.me = p.ws[rank];
  gemm_rs_kernel<<<NWG, NTHR, 0, stream>>>(p);
  HIP_CHECK(hipGetLastError());
}
```
